```python
import numpy as np

kernel_name = "gpu_gemm_a2a_m4096_k4096_n32768_f32_relu"


ABSMAX_RTOL = 0.001

COMPUTE_DTYPE = "float32"
EPILOGUE = "relu"
IN_QUANT = None
OUT_QUANT = None
IN_DTYPES = {"x": "float32", "w_mat": "float32"}
WORLD_SIZE = 8

def setup_inputs():

    x = np.random.randn(4096, 4096).astype(np.float32)
    w_mat = (np.random.randn(4096, 32768) / np.sqrt(4096)).astype(np.float32)
    return {"x": x, "w_mat": w_mat}

def _epilogue(y):
    return np.maximum(y, 0.0).astype(np.float32)

def reference(x, w_mat):
    y = (x @ w_mat).astype(np.float32)
    return _epilogue(y)
```

```cpp
#include <hip/hip_runtime.h>
#include <rccl/rccl.h>
#include <cstdio>

#define HIP_CHECK(expr) do { hipError_t _e = (expr); if (_e != hipSuccess) { \
    fprintf(stderr, "HIP error %s at %s:%d\n", hipGetErrorString(_e), __FILE__, __LINE__); \
    return; } } while (0)

namespace {

typedef _Float16 h8 __attribute__((ext_vector_type(8)));
typedef _Float16 h4 __attribute__((ext_vector_type(4)));
typedef float f32x4 __attribute__((ext_vector_type(4)));
typedef unsigned u32x4 __attribute__((ext_vector_type(4)));
typedef unsigned u32x2 __attribute__((ext_vector_type(2)));
#define LAS __attribute__((address_space(3)))

constexpr int NRANKS = 8;
constexpr int M_ = 4096, K_ = 4096, NLOC = 4096, NFULL = 32768, MLOC = 512;
constexpr int NKT = K_ / 64;
constexpr int NIT = NKT / 2;
#define WLOOK 5
constexpr int D = WLOOK;
constexpr size_t UNIT = 16384;
constexpr size_t WKT_BYTES = 32 * UNIT;
constexpr size_t XKT_BYTES = 4 * UNIT;
constexpr size_t X16_BYTES = (size_t)NKT * XKT_BYTES;
constexpr size_t W16_BYTES = (size_t)NKT * WKT_BYTES;
constexpr size_t WS_TMO = 0;
constexpr size_t WS_CTR = 4096;
constexpr size_t WS_XFLAG = WS_CTR + 256 * 128;
constexpr int WFP = NIT + 1;
constexpr size_t WS_WFLAG = WS_XFLAG + 8 * 256 * 4;
constexpr size_t WS_X16 = 1u << 20;
constexpr size_t WS_W16 = WS_X16 + 2 * X16_BYTES;
constexpr size_t WS_NEEDED = WS_W16 + W16_BYTES + 2 * WKT_BYTES;
static_assert(WS_WFLAG + 16 * WFP * 128 * 4 <= WS_X16, "control area");
constexpr unsigned long long DEADLINE_TICKS = 100000000ull;
constexpr int SCR = 8 * 16384;
constexpr int LDS_BYTES = SCR + 16384;

struct Params {
    const float* x;
    const float* w;
    float* out;
    char* ws[NRANKS];
    int rank;
};

__device__ __forceinline__ h8 cvt8(f32x4 a, f32x4 b) {
    h8 r;
    r[0] = (_Float16)a[0]; r[1] = (_Float16)a[1]; r[2] = (_Float16)a[2]; r[3] = (_Float16)a[3];
    r[4] = (_Float16)b[0]; r[5] = (_Float16)b[1]; r[6] = (_Float16)b[2]; r[7] = (_Float16)b[3];
    return r;
}

#define WAIT_V(n) asm volatile("s_waitcnt vmcnt(%0)" :: "n"(n) : "memory")
#define WAIT_L(n) asm volatile("s_waitcnt lgkmcnt(%0)" :: "n"(n) : "memory")
#define BAR __builtin_amdgcn_s_barrier()
#define SCHED __builtin_amdgcn_sched_barrier(0)

__device__ __forceinline__ void spin_flags2(const unsigned* lane_ptr, unsigned call, unsigned* tmo) {
    const unsigned long long t0 = __builtin_amdgcn_s_memrealtime();
    for (;;) {
        u32x2 v;
        asm volatile("global_load_dwordx2 %0, %1, off sc0 sc1\n\ts_waitcnt vmcnt(0)" : "=&v"(v) : "v"(lane_ptr) : "memory");
        if (__all(v[0] >= call && v[1] >= call)) return;
        if (__builtin_amdgcn_readfirstlane(__hip_atomic_load(tmo, __ATOMIC_RELAXED, __HIP_MEMORY_SCOPE_AGENT)) == call
            || __builtin_amdgcn_s_memrealtime() - t0 > DEADLINE_TICKS) {
            __hip_atomic_store(tmo, call, __ATOMIC_RELAXED, __HIP_MEMORY_SCOPE_AGENT);
            return;
        }
        __builtin_amdgcn_s_sleep(2);
    }
}

__global__ __launch_bounds__(512, 2) void fused_kernel(Params p) {
    extern __shared__ __attribute__((aligned(16))) unsigned char shm[];
    LAS unsigned char* lds = (LAS unsigned char*)shm;
    const int tid = threadIdx.x;
    const int wid = __builtin_amdgcn_readfirstlane(tid >> 6), lane = tid & 63;
    char* myws = p.ws[p.rank];
    const int bid = blockIdx.x;
    const int xg = bid & 7, xi = bid >> 3;
    const int src = (p.rank + xg) & (NRANKS - 1);
    const int mt = src * 2 + (xi & 1), nt = xi >> 1;
    unsigned* tmo = (unsigned*)(myws + WS_TMO);

#define STAMP(idx) do {} while (0)
    STAMP(0);
    if (wid == 0) asm volatile("buffer_inv sc0 sc1" ::: "memory");
    unsigned* ctr = (unsigned*)(myws + WS_CTR + bid * 128 + wid * 4);
    unsigned callv;
    asm volatile("global_load_dword %0, %1, off sc1" : "=v"(callv) : "v"(ctr) : "memory");
    const int xkt = bid >> 2, xmh = bid & 3;
    const float* xsrc = p.x + (size_t)(xmh * 128 + wid * 16 + (lane & 15)) * K_ + xkt * 64 + (lane >> 4) * 8;
    f32x4 xa0, xa1, xb0, xb1;
    asm volatile("global_load_dwordx4 %0, %4, off\n\tglobal_load_dwordx4 %1, %4, off offset:16\n\t"
                 "global_load_dwordx4 %2, %4, off offset:128\n\tglobal_load_dwordx4 %3, %4, off offset:144"
                 : "=&v"(xa0), "=&v"(xa1), "=&v"(xb0), "=&v"(xb1) : "v"(xsrc) : "memory");
    const int wr_ = lane >> 3, wj = lane & 7;
    const float* wsrc = p.w + (size_t)((mt >> 3) * 64 + (mt & 7) * 8 + wr_) * NFULL + p.rank * NLOC + nt * 256 + wid * 32 + ((wj ^ ((wr_ >> 2) << 2)) << 2);
    constexpr size_t WPAIR = (size_t)128 * NFULL;
#pragma unroll
    for (int j = 0; j < D; ++j)
        __builtin_amdgcn_global_load_lds((const unsigned*)(wsrc + j * WPAIR), (LAS unsigned*)(lds + (j * 8 + wid) * 1024), 16, 0, 0);
    asm volatile("s_waitcnt vmcnt(%5)" : "+v"(callv), "+v"(xa0), "+v"(xa1), "+v"(xb0), "+v"(xb1) : "n"(D) : "memory");
    const unsigned call = __builtin_amdgcn_readfirstlane(callv) + 1u;
    STAMP(1);
    const size_t xslot = WS_X16 + (size_t)(call & 1u) * X16_BYTES;
    {
        char* xd = myws + xslot + (size_t)xkt * XKT_BYTES + xmh * UNIT + wid * 2048 + lane * 16;
        const h8 v0 = cvt8(xa0, xa1), v1 = cvt8(xb0, xb1);
        asm volatile("global_store_dwordx4 %0, %1, off sc0 sc1\n\ts_nop 1" :: "v"(xd), "v"(v0) : "memory");
        asm volatile("global_store_dwordx4 %0, %1, off offset:1024 sc0 sc1\n\ts_nop 1" :: "v"(xd), "v"(v1) : "memory");
    }
    const int ccol = lane >> 1, ckh = lane & 1;
    const int crd = ckh * 512 + (((ccol >> 2) ^ (ckh << 2)) << 4) + ((ccol & 3) << 2);
    const int nl = wid * 32 + ccol;
    char* wdst = myws + WS_W16 + (size_t)(mt >> 3) * WKT_BYTES + (size_t)(nt * 2 + (nl >> 7)) * UNIT
                 + ((((nl >> 4) & 7) * 2 + ((mt >> 2) & 1)) << 10) + ((((mt & 3) << 4) + (nl & 15)) << 4) + ckh * 8;
    unsigned* wflag_mine = (unsigned*)(myws + WS_WFLAG) + (size_t)nt * WFP * 128 + mt * 8 + wid;
    const unsigned* wflag_poll = (const unsigned*)(myws + WS_WFLAG) + (size_t)nt * WFP * 128 + lane * 2;
#define WCONV(scroff, pair) do { \
        const LAS float* _s = (const LAS float*)(lds + (scroff) + crd); \
        h4 _h; _h[0] = (_Float16)_s[0]; _h[1] = (_Float16)_s[32]; _h[2] = (_Float16)_s[64]; _h[3] = (_Float16)_s[96]; \
        char* _d = wdst + (size_t)(pair) * (2 * WKT_BYTES); \
        asm volatile("global_store_dwordx2 %0, %1, off sc1" :: "v"(_d), "v"(_h) : "memory"); } while (0)
    WAIT_V(2);
#pragma unroll
    for (int j = 0; j < D; ++j) WCONV((j * 8 + wid) * 1024, j);
    WAIT_V(D);
    BAR;
    STAMP(2);
    if (wid == 0 && lane < NRANKS) {
        unsigned* f = (unsigned*)(p.ws[lane] + WS_XFLAG) + p.rank * 256 + xmh * 64 + xkt;
        asm volatile("global_store_dword %0, %1, off sc0 sc1" :: "v"(f), "v"(call) : "memory");
    }
    WAIT_V(0);
    if (lane < D) {
        unsigned* f = wflag_mine + lane * 128;
        asm volatile("global_store_dword %0, %1, off sc1" :: "v"(f), "v"(call) : "memory");
    }
    if (lane == 0) asm volatile("global_store_dword %0, %1, off sc1" :: "v"(ctr), "v"(call) : "memory");
    STAMP(3);
    spin_flags2((const unsigned*)(myws + WS_XFLAG) + src * 256 + (mt & 1) * 128 + lane * 2, call, tmo);
    STAMP(4);
    spin_flags2(wflag_poll, call, tmo);
    spin_flags2(wflag_poll + 128, call, tmo);

    STAMP(5);
    const int wr = wid >> 2, wc = wid & 3;
    const char* gW = myws + WS_W16 + (size_t)(nt * 2) * UNIT + tid * 16;
    const char* gX = p.ws[src] + xslot + (size_t)((mt & 1) * 2) * UNIT + tid * 16;
    const unsigned ldsw = (unsigned)wid * 1024u;
    const int aoff = wr * 8192 + lane * 16, boff = wc * 4096 + lane * 16;
#define SA(b, h) (((b) * 2 + (h)) * 16384)
#define SB(b, h) ((4 + (b) * 2 + (h)) * 16384)
#define STAGE(bufoff, g, ktb, h, kt) do { _Pragma("unroll") for (int _i = 0; _i < 2; ++_i) \
        __builtin_amdgcn_global_load_lds((const unsigned*)((g) + (size_t)(kt) * (ktb) + (h) * UNIT + _i * 8192), \
            (LAS unsigned*)(lds + (bufoff) + ldsw + _i * 8192), 16, 0, 0); } while (0)
#define STW(b, h, kt) STAGE(SA(b, h), gW, WKT_BYTES, h, kt)
#define STX(b, h, kt) STAGE(SB(b, h), gX, XKT_BYTES, h, kt)
#define LDA(dst, b, h) do { _Pragma("unroll") for (int m = 0; m < 4; ++m) _Pragma("unroll") for (int k = 0; k < 2; ++k) \
        dst[m][k] = *(const LAS h8*)(lds + SA(b, h) + aoff + m * 2048 + k * 1024); } while (0)
#define LDB(dst, b, h) do { _Pragma("unroll") for (int n = 0; n < 2; ++n) _Pragma("unroll") for (int k = 0; k < 2; ++k) \
        dst[n][k] = *(const LAS h8*)(lds + SB(b, h) + boff + n * 2048 + k * 1024); } while (0)
#define MMA(ai, bj, At, Bt) do { __builtin_amdgcn_s_setprio(1); \
        _Pragma("unroll") for (int m = 0; m < 4; ++m) _Pragma("unroll") for (int n = 0; n < 2; ++n) _Pragma("unroll") for (int k = 0; k < 2; ++k) \
            acc[ai][bj][m][n] = __builtin_amdgcn_mfma_f32_16x16x32_f16(At[m][k], Bt[n][k], acc[ai][bj][m][n], 0, 0, 0); \
        __builtin_amdgcn_s_setprio(0); } while (0)

    f32x4 acc[2][2][4][2];
#pragma unroll
    for (int a = 0; a < 2; ++a)
#pragma unroll
        for (int b = 0; b < 2; ++b)
#pragma unroll
            for (int m = 0; m < 4; ++m)
#pragma unroll
                for (int n = 0; n < 2; ++n) acc[a][b][m][n] = (f32x4){0.f, 0.f, 0.f, 0.f};
    h8 At[4][2], B0[2][2], B1[2][2];

    STX(0, 0, 0); STW(0, 0, 0); STX(0, 1, 0); STW(0, 1, 0);
    if (wr == 1) BAR;
    WAIT_V(4); BAR;
    STX(1, 0, 1); STW(1, 0, 1); STX(1, 1, 1);
    WAIT_V(6); BAR;
    for (int it = 0; it < NIT; ++it) {
        const int t = it * 2, t1 = t + 1;
        const int t2 = (it + 1 < NIT) ? t + 2 : 0;
        const int t3 = t2 + 1;
        const int pG = (it + D < NIT) ? it + D : NIT - 1;
        const int pC = (it >= 1 && it + D - 1 < NIT) ? it + D - 1 : NIT;
        const int pF = (it + D - 2 < NIT) ? it + D - 2 : NIT;
        const int pP = (it + 2 < NIT) ? it + 2 : NIT - 1;
        u32x2 pollv;
        STAMP(8 + it);
        LDB(B0, 0, 0); SCHED; LDA(At, 0, 0); STW(1, 1, t1);
        WAIT_L(8); BAR; WAIT_L(0); MMA(0, 0, At, B0); BAR; SCHED;
        LDB(B1, 0, 1); STX(0, 0, t2);
        __builtin_amdgcn_global_load_lds((const unsigned*)(wsrc + (size_t)pG * WPAIR), (LAS unsigned*)(lds + SCR + (it & 1) * 8192 + ldsw), 16, 0, 0);
        BAR; WAIT_L(0); MMA(0, 1, At, B1); BAR;
        LDA(At, 0, 1); STW(0, 0, t2);
        BAR; WAIT_L(0); MMA(1, 0, At, B0); BAR; SCHED;
        STX(0, 1, t2);
        WAIT_V(7); BAR; MMA(1, 1, At, B1); BAR;
        if (lane == 0) {
            unsigned* f = wflag_mine + pF * 128;
            asm volatile("global_store_dword %0, %1, off sc1" :: "v"(f), "v"(call) : "memory");
        }
        asm volatile("global_load_dwordx2 %0, %1, off sc1" : "=v"(pollv) : "v"(wflag_poll + pP * 128) : "memory");
        LDB(B0, 1, 0); SCHED; LDA(At, 1, 0); STW(0, 1, t2);
        WAIT_L(8); BAR; WAIT_L(0); MMA(0, 0, At, B0); BAR; SCHED;
        LDB(B1, 1, 1); STX(1, 0, t3);
        WCONV(SCR + ((it + 1) & 1) * 8192 + ldsw, pC);
        BAR; WAIT_L(0); MMA(0, 1, At, B1); BAR;
        LDA(At, 1, 1); STW(1, 0, t3);
        BAR; WAIT_L(0); MMA(1, 0, At, B0); BAR; SCHED;
        STX(1, 1, t3);
        asm volatile("s_waitcnt vmcnt(7)" : "+v"(pollv) :: "memory");
        if (!__all(pollv[0] >= call && pollv[1] >= call)) spin_flags2(wflag_poll + pP * 128, call, tmo);
        BAR; MMA(1, 1, At, B1); BAR;
    }
    STAMP(40);
    {
        int tz = threadIdx.x; asm volatile("" : "+v"(tz));
        const int ln = tz & 63, fr = ln & 15, fq = ln >> 4;
        const float qnan = __builtin_nanf("");
        const bool bad = __builtin_amdgcn_readfirstlane(__hip_atomic_load(tmo, __ATOMIC_RELAXED, __HIP_MEMORY_SCOPE_AGENT)) == call;
        float* obase = p.out + (size_t)(mt * 256 + wc * 32 + fr) * NLOC + nt * 256 + wr * 64 + fq * 4;
#pragma unroll
        for (int bj = 0; bj < 2; ++bj)
#pragma unroll
            for (int n = 0; n < 2; ++n)
#pragma unroll
                for (int ai = 0; ai < 2; ++ai)
#pragma unroll
                    for (int m = 0; m < 4; ++m) {
                        f32x4 v = acc[ai][bj][m][n];
                        v[0] = fmaxf(v[0], 0.f); v[1] = fmaxf(v[1], 0.f); v[2] = fmaxf(v[2], 0.f); v[3] = fmaxf(v[3], 0.f);
                        if (bad) v = (f32x4){qnan, qnan, qnan, qnan};
                        *(f32x4*)(obase + (size_t)(bj * 128 + n * 16) * NLOC + ai * 128 + m * 16) = v;
                    }
    }
    STAMP(41);
    WAIT_V(0);
    STAMP(42);
    if (wr == 0) BAR;
}

}

extern "C" void kernel_launch(int rank, int nranks, ncclComm_t comm,
                              hipStream_t stream,
                              void* const* d_in, const int* in_sizes, int n_in,
                              void* d_out, int out_size,
                              const int* in_dtypes, int out_dtype,
                              void* const* d_out_all,
                              void* const* d_ws_all, size_t ws_bytes) {
    if (nranks != NRANKS || (unsigned)rank >= (unsigned)NRANKS || ws_bytes < WS_NEEDED || n_in < 2 ||
        in_sizes[0] != MLOC * K_ || in_sizes[1] != K_ * NFULL || out_size != M_ * NLOC) {
        fprintf(stderr, "kernel_launch: unexpected problem shape\n");
        return;
    }
    HIP_CHECK(hipFuncSetAttribute((const void*)fused_kernel, hipFuncAttributeMaxDynamicSharedMemorySize, LDS_BYTES));
    Params p;
    p.x = (const float*)d_in[0];
    p.w = (const float*)d_in[1];
    p.out = (float*)d_out;
    for (int q = 0; q < NRANKS; ++q) p.ws[q] = (char*)d_ws_all[q];
    p.rank = rank;
    hipLaunchKernelGGL(fused_kernel, dim3(256), dim3(512), LDS_BYTES, stream, p);
    HIP_CHECK(hipGetLastError());
}
```

```cpp
#include <hip/hip_runtime.h>
#include <rccl/rccl.h>
#include <cstdio>

#define HIP_CHECK(expr) do { hipError_t _e = (expr); if (_e != hipSuccess) { \
    fprintf(stderr, "HIP error %s at %s:%d\n", hipGetErrorString(_e), __FILE__, __LINE__); \
    return; } } while (0)

namespace {

typedef _Float16 h8 __attribute__((ext_vector_type(8)));
typedef _Float16 h4 __attribute__((ext_vector_type(4)));
typedef float f32x4 __attribute__((ext_vector_type(4)));
typedef unsigned u32x2 __attribute__((ext_vector_type(2)));
#define LAS __attribute__((address_space(3)))

constexpr int NRANKS = 8;
constexpr int M_ = 4096, K_ = 4096, NLOC = 4096, NFULL = 32768, MLOC = 512;
constexpr int NKT = K_ / 64;
constexpr int NIT = NKT / 2;
#define WLOOK 6
constexpr int D = WLOOK;
constexpr size_t UNIT = 16384;
constexpr size_t WKT_BYTES = 32 * UNIT;
constexpr size_t XKT_BYTES = 4 * UNIT;
constexpr size_t X16_BYTES = (size_t)NKT * XKT_BYTES;
constexpr size_t W16_BYTES = (size_t)NKT * WKT_BYTES;
constexpr size_t WS_TMO = 0;
constexpr size_t WS_CTR = 4096;
constexpr size_t WS_XCNT = WS_CTR + 256 * 128;
constexpr size_t WS_MBOX = WS_XCNT + 8 * 128;
constexpr int WFP = NIT + 1;
constexpr size_t WS_WFLAG = WS_MBOX + 256 * 128;
constexpr size_t WS_X16 = 1u << 20;
constexpr size_t WS_W16 = WS_X16 + 2 * X16_BYTES;
constexpr size_t WS_NEEDED = WS_W16 + W16_BYTES + 2 * WKT_BYTES;
static_assert(WS_WFLAG + 16 * WFP * 16 * 4 <= WS_X16, "control area");
constexpr unsigned long long DEADLINE_TICKS = 100000000ull;
constexpr int SCR = 8 * 16384;
constexpr int POLL = SCR + 16384;
constexpr int LDS_BYTES = POLL + 256 + 16;

struct Params {
    const float* x;
    const float* w;
    float* out;
    char* ws[NRANKS];
    int rank;
    unsigned srcmap;
    unsigned grpmap;
    unsigned invmap;
};

__device__ __forceinline__ h8 cvt8(f32x4 a, f32x4 b) {
    h8 r;
    r[0] = (_Float16)a[0]; r[1] = (_Float16)a[1]; r[2] = (_Float16)a[2]; r[3] = (_Float16)a[3];
    r[4] = (_Float16)b[0]; r[5] = (_Float16)b[1]; r[6] = (_Float16)b[2]; r[7] = (_Float16)b[3];
    return r;
}

#define WAIT_V(n) asm volatile("s_waitcnt vmcnt(%0)" :: "n"(n) : "memory")
#define WAIT_L(n) asm volatile("s_waitcnt lgkmcnt(%0)" :: "n"(n) : "memory")
#define WAIT_VX(n, w) asm volatile("s_waitcnt vmcnt(%0)\n\ts_cmp_eq_u32 %2, %3\n\ts_cbranch_scc1 1f\n\ts_waitcnt vmcnt(%1)\n1:" \
        :: "n"((n) + 1), "n"(n), "s"(wid), "n"(w) : "memory", "scc")
#define BAR __builtin_amdgcn_s_barrier()
#define SCHED __builtin_amdgcn_sched_barrier(0)

__device__ __forceinline__ void spin_flags(const unsigned* lane_ptr, unsigned need, unsigned call, unsigned* tmo) {
    const unsigned long long t0 = __builtin_amdgcn_s_memrealtime();
    for (unsigned n = 1;; ++n) {
        unsigned v;
        asm volatile("global_load_dword %0, %1, off sc0 sc1\n\ts_waitcnt vmcnt(0)" : "=&v"(v) : "v"(lane_ptr) : "memory");
        if (__all(v >= need)) return;
        if ((n & 15u) == 0u) {
            if (__builtin_amdgcn_readfirstlane(__hip_atomic_load(tmo, __ATOMIC_RELAXED, __HIP_MEMORY_SCOPE_AGENT)) == call
                || __builtin_amdgcn_s_memrealtime() - t0 > DEADLINE_TICKS) {
                __hip_atomic_store(tmo, call, __ATOMIC_RELAXED, __HIP_MEMORY_SCOPE_AGENT);
                return;
            }
        }
        __builtin_amdgcn_s_sleep(1);
    }
}

__global__ __launch_bounds__(512, 2) void fused_kernel(Params p) {
    extern __shared__ __attribute__((aligned(16))) unsigned char shm[];
    LAS unsigned char* lds = (LAS unsigned char*)shm;
    const int tid = threadIdx.x;
    const int wid = __builtin_amdgcn_readfirstlane(tid >> 6), lane = tid & 63;
    char* myws = p.ws[p.rank];
    const int bid = blockIdx.x;
    const int xg = bid & 7, xi = bid >> 3;
    const int src = (p.srcmap >> (xg * 4)) & (NRANKS - 1);
    const int mt = src * 2 + (xi & 1), nt = xi >> 1;
    unsigned* tmo = (unsigned*)(myws + WS_TMO);

#define STAMP(idx) do {} while (0)
#define CSTAMP(idx) do {} while (0)
    STAMP(0);
    unsigned* ctr = (unsigned*)(myws + WS_CTR + bid * 128 + wid * 4);
    unsigned callv;
    const int xkt = bid >> 2, xmh = bid & 3;
    const int wr_ = lane >> 3, wj = lane & 7;
    const float* wsrc0 = p.w + (size_t)((mt >> 3) * 64 + (mt & 7) * 8 + wr_) * NFULL + p.rank * NLOC + nt * 256 + ((wj ^ ((wr_ >> 2) << 2)) << 2);
    constexpr size_t WPAIR = (size_t)128 * NFULL;
    const int ccol = lane >> 1, ckh = lane & 1;
    const int crd = ckh * 512 + (((ccol >> 2) ^ (ckh << 2)) << 4) + ((ccol & 3) << 2);
    char* wdst0 = myws + WS_W16 + (size_t)(mt >> 3) * WKT_BYTES + (size_t)(nt * 2) * UNIT
                  + (((ccol >> 4) * 2 + ((mt >> 2) & 1)) << 10) + ((((mt & 3) << 4) + (ccol & 15)) << 4) + ckh * 8;
    unsigned* wflag_row = (unsigned*)(myws + WS_WFLAG) + (size_t)nt * WFP * 16;
    const unsigned* mbox = (const unsigned*)(myws + WS_MBOX + bid * 128);
#define WCONV(scroff, dstp) do { \
        const LAS float* _s = (const LAS float*)(lds + (scroff) + crd); \
        h4 _h; _h[0] = (_Float16)_s[0]; _h[1] = (_Float16)_s[32]; _h[2] = (_Float16)_s[64]; _h[3] = (_Float16)_s[96]; \
        asm volatile("global_store_dwordx2 %0, %1, off " WSC :: "v"(dstp), "v"(_h) : "memory"); } while (0)
#define XSC "sc0 sc1"
#define WDELAY 32
#define WAUX 2
#define XLD ""
#define OSC "sc1"
#define WSC "sc1"
    unsigned call;
    LAS unsigned* arr = (LAS unsigned*)(lds + POLL + 256);
    if (tid < 2) arr[tid] = 0u;
    WAIT_L(0); BAR;
    if (wid < 4) {
        const float* xsrc = p.x + (size_t)(xmh * 128 + wid * 32 + (lane & 15)) * K_ + xkt * 64 + (lane >> 4) * 8;
        const float* xsrc2 = xsrc + 16 * K_;
        f32x4 a0, a1, a2, a3, b0, b1, b2, b3;
        asm volatile("global_load_dwordx4 %0, %9, off" XLD "\n\tglobal_load_dwordx4 %1, %9, off offset:16" XLD "\n\t"
                     "global_load_dwordx4 %2, %9, off offset:128" XLD "\n\tglobal_load_dwordx4 %3, %9, off offset:144" XLD "\n\t"
                     "global_load_dwordx4 %4, %10, off" XLD "\n\tglobal_load_dwordx4 %5, %10, off offset:16" XLD "\n\t"
                     "global_load_dwordx4 %6, %10, off offset:128" XLD "\n\tglobal_load_dwordx4 %7, %10, off offset:144" XLD "\n\t"
                     "s_load_dword %8, %11, 0x0\n\t"
                     "s_waitcnt vmcnt(0) lgkmcnt(0)"
                     : "=&v"(a0), "=&v"(a1), "=&v"(a2), "=&v"(a3), "=&v"(b0), "=&v"(b1), "=&v"(b2), "=&v"(b3), "=&s"(callv)
                     : "v"(xsrc), "v"(xsrc2), "s"(ctr) : "memory");
        call = callv + 1u;
        STAMP(1);
        char* xd = myws + WS_X16 + (size_t)(call & 1u) * X16_BYTES + (size_t)xkt * XKT_BYTES + xmh * UNIT + wid * 4096 + lane * 16;
        const h8 v0 = cvt8(a0, a1), v1 = cvt8(a2, a3), v2 = cvt8(b0, b1), v3 = cvt8(b2, b3);
        asm volatile("global_store_dwordx4 %0, %1, off " XSC "\n\tglobal_store_dwordx4 %0, %2, off offset:1024 " XSC "\n\t"
                     "global_store_dwordx4 %0, %3, off offset:2048 " XSC "\n\tglobal_store_dwordx4 %0, %4, off offset:3072 " XSC "\n\t"
                     "s_waitcnt vmcnt(0)"
                     :: "v"(xd), "v"(v0), "v"(v1), "v"(v2), "v"(v3) : "memory");
        STAMP(2);
        unsigned old = 0;
        if (lane == 0) old = __hip_atomic_fetch_add(arr, 1u, __ATOMIC_RELAXED, __HIP_MEMORY_SCOPE_WORKGROUP);
        if (__builtin_amdgcn_readfirstlane(old) == 3u) {
            const int shard = bid >> 5;
            unsigned cnt = 0;
            if (lane == 0) cnt = __hip_atomic_fetch_add((unsigned*)(myws + WS_XCNT + shard * 128), 1u, __ATOMIC_RELAXED, __HIP_MEMORY_SCOPE_AGENT);
            if (__builtin_amdgcn_readfirstlane(cnt) + 1u == 32u * call) {
#pragma unroll
                for (int i = 0; i < 4; ++i) {
                    const int q0 = 2 * i, q1 = 2 * i + 1;
                    char* w0 = p.ws[q0] + (((p.grpmap >> (4 * q0)) & 7u) << 7);
                    char* w1 = p.ws[q1] + (((p.grpmap >> (4 * q1)) & 7u) << 7);
                    char* f = (lane < 32 ? w0 : w1) + WS_MBOX + (lane & 31) * 1024 + shard * 4;
                    asm volatile("global_store_dword %0, %1, off sc0 sc1" :: "v"(f), "v"(call) : "memory");
                }
            }
        }
        STAMP(3);
        if (wid == 0) spin_flags(mbox + (lane & 31), (lane & 31) == 0 || ((lane & 31) >= 8 && (lane & 31) < 24) ? call : 0u, call, tmo);
        STAMP(4);
    } else if (wid < 6) {
        const int j = wid - 4;
#pragma unroll
        for (int c = 0; c < 8; ++c)
            __builtin_amdgcn_global_load_lds((const unsigned*)(wsrc0 + j * WPAIR + c * 32), (LAS unsigned*)(lds + (j * 8 + c) * 1024), 16, 0, WAUX);
        asm volatile("s_load_dword %0, %1, 0x0\n\ts_waitcnt lgkmcnt(0)" : "=&s"(callv) : "s"(ctr) : "memory");
        call = callv + 1u;
        char* wd = wdst0 + (size_t)j * (2 * WKT_BYTES);
#pragma unroll
        for (int c = 0; c < 8; ++c) {
            WAIT_V(7);
            WCONV((j * 8 + c) * 1024, wd + (c >> 2) * UNIT + (c & 3) * 4096);
        }
        STAMP(1);
        WAIT_V(0);
        STAMP(2);
        unsigned old = 0;
        if (lane == 0) old = __hip_atomic_fetch_add(arr + 1, 1u, __ATOMIC_RELAXED, __HIP_MEMORY_SCOPE_WORKGROUP);
        if (__builtin_amdgcn_readfirstlane(old) == 1u && lane < 16) {
            const int cb = ((nt * 2 + (lane & 1)) << 3) + (int)((p.invmap >> (4 * (lane >> 1))) & 7u);
            char* f = myws + WS_MBOX + cb * 128 + (8 + mt) * 4;
            asm volatile("global_store_dword %0, %1, off sc1" :: "v"(f), "v"(call) : "memory");
        }
    } else {
        const int c0 = (wid - 6) * 4;
        __builtin_amdgcn_s_sleep(WDELAY);
#pragma unroll
        for (int j = 2; j < D; ++j)
#pragma unroll
            for (int c = 0; c < 4; ++c)
                __builtin_amdgcn_global_load_lds((const unsigned*)(wsrc0 + j * WPAIR + (c0 + c) * 32), (LAS unsigned*)(lds + (j * 8 + c0 + c) * 1024), 16, 0, WAUX);
        asm volatile("s_load_dword %0, %1, 0x0\n\ts_waitcnt lgkmcnt(0)" : "=&s"(callv) : "s"(ctr) : "memory");
        call = callv + 1u;
        char* wd = wdst0 + (c0 >> 2) * UNIT;
#pragma unroll
        for (int j = 2; j < D; ++j)
#pragma unroll
            for (int c = 0; c < 4; ++c) {
                WAIT_V(4 * (D - 2) - 1);
                WCONV((j * 8 + c0 + c) * 1024, wd + (size_t)j * (2 * WKT_BYTES) + c * 4096);
            }
    }
    STAMP(6);
    BAR;
    const size_t xslot = WS_X16 + (size_t)(call & 1u) * X16_BYTES;
    const float* wsrc = wsrc0 + wid * 32;
    char* wdst = wdst0 + (wid >> 2) * UNIT + (wid & 3) * 4096;
    STAMP(5);
    const int wr = wid >> 2, wc = wid & 3;
    const char* gW = myws + WS_W16 + (size_t)(nt * 2) * UNIT + tid * 16;
    const char* gX = p.ws[src] + xslot + (size_t)((mt & 1) * 2) * UNIT + tid * 16;
    const unsigned ldsw = (unsigned)wid * 1024u;
    const int aoff = wr * 8192 + lane * 16, boff = wc * 4096 + lane * 16;
#define SA(b, h) (((b) * 2 + (h)) * 16384)
#define SB(b, h) ((4 + (b) * 2 + (h)) * 16384)
#define STAGE(bufoff, g, ktb, h, kt) do { _Pragma("unroll") for (int _i = 0; _i < 2; ++_i) \
        __builtin_amdgcn_global_load_lds((const unsigned*)((g) + (size_t)(kt) * (ktb) + (h) * UNIT + _i * 8192), \
            (LAS unsigned*)(lds + (bufoff) + ldsw + _i * 8192), 16, 0, 0); } while (0)
#define STW(b, h, kt) STAGE(SA(b, h), gW, WKT_BYTES, h, kt)
#define STX(b, h, kt) STAGE(SB(b, h), gX, XKT_BYTES, h, kt)
#define LDA(dst, b, h) do { _Pragma("unroll") for (int m = 0; m < 4; ++m) _Pragma("unroll") for (int k = 0; k < 2; ++k) \
        dst[m][k] = *(const LAS h8*)(lds + SA(b, h) + aoff + m * 2048 + k * 1024); } while (0)
#define LDB(dst, b, h) do { _Pragma("unroll") for (int n = 0; n < 2; ++n) _Pragma("unroll") for (int k = 0; k < 2; ++k) \
        dst[n][k] = *(const LAS h8*)(lds + SB(b, h) + boff + n * 2048 + k * 1024); } while (0)
#define MMA(ai, bj, At, Bt) do { __builtin_amdgcn_s_setprio(1); \
        _Pragma("unroll") for (int m = 0; m < 4; ++m) _Pragma("unroll") for (int n = 0; n < 2; ++n) _Pragma("unroll") for (int k = 0; k < 2; ++k) \
            acc[ai][bj][m][n] = __builtin_amdgcn_mfma_f32_16x16x32_f16(At[m][k], Bt[n][k], acc[ai][bj][m][n], 0, 0, 0); \
        __builtin_amdgcn_s_setprio(0); } while (0)

    f32x4 acc[2][2][4][2];
#pragma unroll
    for (int a = 0; a < 2; ++a)
#pragma unroll
        for (int b = 0; b < 2; ++b)
#pragma unroll
            for (int m = 0; m < 4; ++m)
#pragma unroll
                for (int n = 0; n < 2; ++n) acc[a][b][m][n] = (f32x4){0.f, 0.f, 0.f, 0.f};
    h8 At[4][2], B0[2][2], B1[2][2];

    const int pl = (lane >= 24) ? (lane & 15) : lane;
    if (wid == 0) *(volatile LAS unsigned*)(lds + POLL + lane * 4) = call;
    STX(0, 0, 0); STW(0, 0, 0); STX(0, 1, 0); STW(0, 1, 0);
    STX(1, 0, 1); STW(1, 0, 1); STX(1, 1, 1);
    WAIT_V(6); BAR;
    if (wid == 2 && lane < D - 2) asm volatile("global_store_dword %0, %1, off sc1" :: "v"(wflag_row + (lane + 2) * 16 + mt), "v"(call) : "memory");
    if (wr == 1) BAR;
    STAMP(8); CSTAMP(50);
    for (int it = 0; it < NIT - 1; ++it) {
        const int t = it * 2, t1 = t + 1, t2 = t + 2, t3 = t + 3;
        const int pG = (it + D < NIT) ? it + D : NIT - 1;
        const int pC = (it >= 1 && it + D - 1 < NIT) ? it + D - 1 : NIT;
        const int pF = (it + D - 2 < NIT) ? it + D - 2 : NIT;
        const int pP = (it + 2 < NIT) ? it + 2 : NIT - 1;
        unsigned pv = *(const volatile LAS unsigned*)(lds + POLL + lane * 4);
        SCHED;
        LDB(B0, 0, 0); SCHED; LDA(At, 0, 0); STW(1, 1, t1);
        WAIT_L(8); BAR; WAIT_L(0); MMA(0, 0, At, B0); BAR; SCHED;
        asm volatile("" : "+v"(pv));
        {
            const unsigned need = (pl < 16 || pl - 16 <= ((it + 2) >> 2)) ? call : 0u;
            if (__builtin_expect(!__all(pv >= need), 0)) if (wid == 0) spin_flags(pl < 16 ? wflag_row + (it + 1) * 16 + pl : mbox + (pl - 16), need, call, tmo);
        }
        LDB(B1, 0, 1); STX(0, 0, t2);
        __builtin_amdgcn_global_load_lds((const unsigned*)(wsrc + (size_t)pG * WPAIR), (LAS unsigned*)(lds + SCR + (it & 1) * 8192 + ldsw), 16, 0, WAUX);
        BAR; WAIT_L(0); MMA(0, 1, At, B1); BAR;
        LDA(At, 0, 1); STW(0, 0, t2);
        BAR; WAIT_L(0); MMA(1, 0, At, B0); BAR; SCHED;
        if (wid == 0) __builtin_amdgcn_global_load_lds(pl < 16 ? wflag_row + pP * 16 + pl : mbox + (pl - 16), (LAS unsigned*)(lds + POLL), 4, 0, 17);
        STX(0, 1, t2);
        WAIT_VX(7, 0);
        BAR; MMA(1, 1, At, B1); BAR;
        LDB(B0, 1, 0); SCHED; LDA(At, 1, 0); STW(0, 1, t2);
        WAIT_L(8); BAR; WAIT_L(0); MMA(0, 0, At, B0); BAR; SCHED;
        if (wid == 1) asm volatile("global_store_dword %0, %1, off sc1" :: "v"(wflag_row + pF * 16 + mt), "v"(call) : "memory");
        LDB(B1, 1, 1);
        const LAS float* cs = (const LAS float*)(lds + SCR + ((it + 1) & 1) * 8192 + ldsw + crd);
        const float c0_ = cs[0], c1_ = cs[32], c2_ = cs[64], c3_ = cs[96];
        STX(1, 0, t3);
        BAR; WAIT_L(0); MMA(0, 1, At, B1); BAR;
        LDA(At, 1, 1); STW(1, 0, t3);
        BAR; WAIT_L(0); MMA(1, 0, At, B0); BAR; SCHED;
        { h4 hv; hv[0] = (_Float16)c0_; hv[1] = (_Float16)c1_; hv[2] = (_Float16)c2_; hv[3] = (_Float16)c3_;
          asm volatile("global_store_dwordx2 %0, %1, off " WSC :: "v"(wdst + (size_t)pC * (2 * WKT_BYTES)), "v"(hv) : "memory"); }
        STX(1, 1, t3);
        WAIT_VX(7, 1);
        BAR; MMA(1, 1, At, B1); BAR;
    }
    STAMP(39); CSTAMP(51);
    int tz = threadIdx.x; asm volatile("" : "+v"(tz));
    const int ln = tz & 63, fr = ln & 15, fq = ln >> 4;
    float* obase = p.out + (size_t)(mt * 256 + wc * 32 + fr) * NLOC + nt * 256 + wr * 64 + fq * 4;
    const float qnan = __builtin_nanf("");
    unsigned tmov;
#define OUTQ(ai, bj) do { _Pragma("unroll") for (int n = 0; n < 2; ++n) _Pragma("unroll") for (int m = 0; m < 4; ++m) { \
        f32x4 v = acc[ai][bj][m][n]; \
        v[0] = fmaxf(v[0], 0.f); v[1] = fmaxf(v[1], 0.f); v[2] = fmaxf(v[2], 0.f); v[3] = fmaxf(v[3], 0.f); \
        if (bad) v = (f32x4){qnan, qnan, qnan, qnan}; \
        float* dst = obase + (size_t)((bj) * 128 + n * 16) * NLOC + (ai) * 128 + m * 16; \
        asm volatile("global_store_dwordx4 %0, %1, off " OSC "\n\ts_nop 1" :: "v"(dst), "v"(v) : "memory"); } } while (0)
    {
        LDB(B0, 0, 0); SCHED; LDA(At, 0, 0); STW(1, 1, NKT - 1);
        WAIT_L(8); BAR; WAIT_L(0); MMA(0, 0, At, B0); BAR; SCHED;
        LDB(B1, 0, 1);
        BAR; WAIT_L(0); MMA(0, 1, At, B1); BAR;
        LDA(At, 0, 1);
        BAR; WAIT_L(0); MMA(1, 0, At, B0); BAR; SCHED;
        asm volatile("global_load_dword %0, %1, off sc1\n\ts_waitcnt vmcnt(0)" : "=&v"(tmov) : "v"(tmo) : "memory");
        BAR; MMA(1, 1, At, B1); BAR;
        const bool bad = __builtin_amdgcn_readfirstlane(tmov) == call;

        LDB(B0, 1, 0); SCHED; LDA(At, 1, 0);
        BAR; WAIT_L(0); MMA(0, 0, At, B0); BAR; SCHED;
        LDB(B1, 1, 1); OUTQ(0, 0);
        BAR; WAIT_L(0); MMA(0, 1, At, B1); BAR;
        LDA(At, 1, 1); OUTQ(0, 1);
        BAR; WAIT_L(0); MMA(1, 0, At, B0); BAR; SCHED;
        OUTQ(1, 0);
        BAR; MMA(1, 1, At, B1);
        STAMP(40);
        OUTQ(1, 1);
    }
    if (ln == 0) asm volatile("global_store_dword %0, %1, off sc1" :: "v"(ctr), "v"(call) : "memory");
    STAMP(41);
    WAIT_V(0);
    STAMP(42);
    if (wr == 0) BAR;
}

}

extern "C" void kernel_launch(int rank, int nranks, ncclComm_t comm,
                              hipStream_t stream,
                              void* const* d_in, const int* in_sizes, int n_in,
                              void* d_out, int out_size,
                              const int* in_dtypes, int out_dtype,
                              void* const* d_out_all,
                              void* const* d_ws_all, size_t ws_bytes) {
    if (nranks != NRANKS || (unsigned)rank >= (unsigned)NRANKS || ws_bytes < WS_NEEDED || n_in < 2 ||
        in_sizes[0] != MLOC * K_ || in_sizes[1] != K_ * NFULL || out_size != M_ * NLOC) {
        fprintf(stderr, "kernel_launch: unexpected problem shape\n");
        return;
    }
    HIP_CHECK(hipFuncSetAttribute((const void*)fused_kernel, hipFuncAttributeMaxDynamicSharedMemorySize, LDS_BYTES));
    Params p;
    p.x = (const float*)d_in[0];
    p.w = (const float*)d_in[1];
    p.out = (float*)d_out;
    for (int q = 0; q < NRANKS; ++q) p.ws[q] = (char*)d_ws_all[q];
    p.rank = rank;
    {
        const unsigned half_mask = 0x4Du;
        p.srcmap = p.grpmap = p.invmap = 0;
        for (int r = 0; r < NRANKS; ++r) {
            const int mine = (half_mask >> r) & 1;
            int g = 0;
            for (int pass = 0; pass < 2; ++pass)
                for (int d = 0; d < NRANKS; ++d) {
                    const int q = (r + d) & 7;
                    if (((int)((half_mask >> q) & 1) == mine) != (pass == 0)) continue;
                    if (r == rank) { p.srcmap |= (unsigned)q << (4 * g); p.invmap |= (unsigned)g << (4 * q); }
                    if (q == rank) p.grpmap |= (unsigned)g << (4 * r);
                    ++g;
                }
        }
    }
    hipLaunchKernelGGL(fused_kernel, dim3(256), dim3(512), LDS_BYTES, stream, p);
    HIP_CHECK(hipGetLastError());
}
```

```cpp
#include <hip/hip_runtime.h>
#include <rccl/rccl.h>
#include <cstdio>

#define HIP_CHECK(expr) do { hipError_t _e = (expr); if (_e != hipSuccess) { \
    fprintf(stderr, "HIP error %s at %s:%d\n", hipGetErrorString(_e), __FILE__, __LINE__); \
    return; } } while (0)

namespace {

typedef _Float16 h8 __attribute__((ext_vector_type(8)));
typedef _Float16 h4 __attribute__((ext_vector_type(4)));
typedef float f32x4 __attribute__((ext_vector_type(4)));
typedef unsigned u32x4 __attribute__((ext_vector_type(4)));
#define LAS __attribute__((address_space(3)))

constexpr int NRANKS = 8;
constexpr int M_ = 4096, K_ = 4096, NLOC = 4096, NFULL = 32768, MLOC = 512;
constexpr int NKT = K_ / 64;
constexpr int NIT = NKT / 2;
constexpr int NQ = NKT / 4;
constexpr int UNIT = 16384;
constexpr int WKT_BYTES = 32 * UNIT;
constexpr int XKT_BYTES = 4 * UNIT;
constexpr int X16_BYTES = NKT * XKT_BYTES;
constexpr int W16_BYTES = NKT * WKT_BYTES;
constexpr int WROW = NFULL * 4;
constexpr int WPAIR = 128 * WROW;
constexpr int WS_TMO = 0;
constexpr int WS_CTR = 4096;
constexpr int WS_XCNT = WS_CTR + 256 * 128;
constexpr int WS_MBOX = WS_XCNT + 8 * 128;
constexpr int WS_WFLAG = WS_MBOX + 256 * 128;
constexpr int WS_X16 = 1 << 20;
constexpr int WS_W16 = WS_X16 + 2 * X16_BYTES;
constexpr size_t WS_NEEDED = (size_t)WS_W16 + W16_BYTES;
static_assert(WS_WFLAG + 16 * NQ * 16 * 4 <= WS_X16, "control area");
constexpr unsigned long long DEADLINE_TICKS = 100000000ull;
constexpr int SCR = 8 * 16384;
constexpr int POLL = SCR + 16384;
constexpr int ARR = POLL + 256;
constexpr int LDS_BYTES = ARR + 16;

struct Params {
    const float* x;
    const float* w;
    float* out;
    char* ws[NRANKS];
    int rank;
    unsigned srcmap;
    unsigned grpmap;
    unsigned invmap;
};

__device__ __forceinline__ h8 cvt8(f32x4 a, f32x4 b) {
    h8 r;
    r[0] = (_Float16)a[0]; r[1] = (_Float16)a[1]; r[2] = (_Float16)a[2]; r[3] = (_Float16)a[3];
    r[4] = (_Float16)b[0]; r[5] = (_Float16)b[1]; r[6] = (_Float16)b[2]; r[7] = (_Float16)b[3];
    return r;
}

#define WAIT_V(n) asm volatile("s_waitcnt vmcnt(%0)" :: "n"(n) : "memory")
#define WAIT_L(n) do { asm volatile("" ::: "memory"); __builtin_amdgcn_s_waitcnt(0xC07F); asm volatile("" ::: "memory"); } while (0)
#define WAIT_VX(n, sel, w) asm volatile("s_waitcnt vmcnt(%0)\n\ts_cmp_eq_u32 %2, %3\n\ts_cbranch_scc1 1f\n\ts_waitcnt vmcnt(%1)\n1:" \
        :: "n"((n) + 1), "n"(n), "s"(sel), "n"(w) : "memory", "scc")
#define BAR __builtin_amdgcn_s_barrier()
#define SCHED __builtin_amdgcn_sched_barrier(0)
#define RSRC(ptr, bytes) __builtin_amdgcn_make_buffer_rsrc((void*)(ptr), 0, (bytes), 0x00020000)
#define BLDS(rsrc, ldsoff, size, voff, soff, aux) \
        __builtin_amdgcn_raw_ptr_buffer_load_lds(rsrc, (LAS void*)(lds + (ldsoff)), size, voff, soff, 0, aux)

__device__ __forceinline__ void spin_flags(const unsigned* lane_ptr, unsigned need, unsigned call, unsigned* tmo) {
    const unsigned long long t0 = __builtin_amdgcn_s_memrealtime();
    for (unsigned n = 1;; ++n) {
        unsigned v;
        asm volatile("global_load_dword %0, %1, off sc0 sc1\n\ts_waitcnt vmcnt(0)" : "=&v"(v) : "v"(lane_ptr) : "memory");
        if (__all(v >= need)) return;
        if ((n & 15u) == 0u) {
            if (__builtin_amdgcn_readfirstlane(__hip_atomic_load(tmo, __ATOMIC_RELAXED, __HIP_MEMORY_SCOPE_AGENT)) == call
                || __builtin_amdgcn_s_memrealtime() - t0 > DEADLINE_TICKS) {
                __hip_atomic_store(tmo, call, __ATOMIC_RELAXED, __HIP_MEMORY_SCOPE_AGENT);
                return;
            }
        }
        __builtin_amdgcn_s_sleep(1);
    }
}

__global__ __launch_bounds__(512, 2) void fused_kernel(Params p) {
    extern __shared__ __attribute__((aligned(16))) unsigned char shm[];
    LAS unsigned char* lds = (LAS unsigned char*)shm;
    const int tid = threadIdx.x;
    const int wid = __builtin_amdgcn_readfirstlane(tid >> 6), lane = tid & 63;
    char* myws = p.ws[p.rank];
    const int bid = blockIdx.x;
    const int xg = bid & 7, xi = bid >> 3;
    const int src = (p.srcmap >> (xg * 4)) & (NRANKS - 1);
    const int mt = src * 2 + (xi & 1), nt = xi >> 1;
    unsigned* tmo = (unsigned*)(myws + WS_TMO);
#define STAMP(idx) do {} while (0)
#define CSTAMP(idx) do {} while (0)
    STAMP(0);
    const auto rWS = RSRC(myws, (int)WS_NEEDED);
    const auto rW = RSRC(p.w + p.rank * NLOC + nt * 256, (K_ - 1) * WROW + 1024);
    unsigned* ctr = (unsigned*)(myws + WS_CTR + bid * 128 + wid * 4);
    unsigned callv, call;
    const int gvoff = (lane >> 3) * WROW + (((lane & 7) ^ ((lane >> 5) << 2)) << 4);
    const int gsoff = ((mt >> 3) * 64 + (mt & 7) * 8) * WROW;
    const int wsoff = WS_W16 + (mt >> 3) * WKT_BYTES + nt * 2 * UNIT + (((mt >> 2) & 1) << 10) + ((mt & 3) << 8);
    unsigned* wflag_row = (unsigned*)(myws + WS_WFLAG) + nt * NQ * 16;
    const unsigned* mbox = (const unsigned*)(myws + WS_MBOX + bid * 128);
    LAS unsigned* arr = (LAS unsigned*)(lds + ARR);
    if (tid < 2) arr[tid] = 0u;
    WAIT_L(0); BAR;
    if (wid < 4) {
        const int xkt = bid >> 2, xmh = bid & 3;
        const float* xsrc = p.x + (size_t)(xmh * 128 + wid * 32 + (lane & 15)) * K_ + xkt * 64 + (lane >> 4) * 8;
        const float* xsrc2 = xsrc + 16 * K_;
        f32x4 a0, a1, a2, a3, b0, b1, b2, b3;
        asm volatile("global_load_dwordx4 %0, %9, off nt\n\tglobal_load_dwordx4 %1, %9, off offset:16 nt\n\t"
                     "global_load_dwordx4 %2, %9, off offset:128 nt\n\tglobal_load_dwordx4 %3, %9, off offset:144 nt\n\t"
                     "global_load_dwordx4 %4, %10, off nt\n\tglobal_load_dwordx4 %5, %10, off offset:16 nt\n\t"
                     "global_load_dwordx4 %6, %10, off offset:128 nt\n\tglobal_load_dwordx4 %7, %10, off offset:144 nt\n\t"
                     "s_load_dword %8, %11, 0x0\n\t"
                     "s_waitcnt vmcnt(0) lgkmcnt(0)"
                     : "=&v"(a0), "=&v"(a1), "=&v"(a2), "=&v"(a3), "=&v"(b0), "=&v"(b1), "=&v"(b2), "=&v"(b3), "=&s"(callv)
                     : "v"(xsrc), "v"(xsrc2), "s"(ctr) : "memory");
        call = callv + 1u;
        STAMP(1);
        char* xd = myws + WS_X16 + (call & 1u) * X16_BYTES + xkt * XKT_BYTES + xmh * UNIT + wid * 4096 + lane * 16;
        const h8 v0 = cvt8(a0, a1), v1 = cvt8(a2, a3), v2 = cvt8(b0, b1), v3 = cvt8(b2, b3);
        asm volatile("global_store_dwordx4 %0, %1, off sc0 sc1\n\tglobal_store_dwordx4 %0, %2, off offset:1024 sc0 sc1\n\t"
                     "global_store_dwordx4 %0, %3, off offset:2048 sc0 sc1\n\tglobal_store_dwordx4 %0, %4, off offset:3072 sc0 sc1\n\t"
                     "s_waitcnt vmcnt(0)"
                     :: "v"(xd), "v"(v0), "v"(v1), "v"(v2), "v"(v3) : "memory");
        STAMP(2);
        unsigned old = 0;
        if (lane == 0) old = __hip_atomic_fetch_add(arr, 1u, __ATOMIC_RELAXED, __HIP_MEMORY_SCOPE_WORKGROUP);
        if (__builtin_amdgcn_readfirstlane(old) == 3u) {
            const int shard = bid >> 5;
            unsigned cnt = 0;
            if (lane == 0) cnt = __hip_atomic_fetch_add((unsigned*)(myws + WS_XCNT + shard * 128), 1u, __ATOMIC_RELAXED, __HIP_MEMORY_SCOPE_AGENT);
            if (__builtin_amdgcn_readfirstlane(cnt) + 1u == 32u * call) {
#pragma unroll
                for (int i = 0; i < 4; ++i) {
                    const int q0 = 2 * i, q1 = 2 * i + 1;
                    char* w0 = p.ws[q0] + (((p.grpmap >> (4 * q0)) & 7u) << 7);
                    char* w1 = p.ws[q1] + (((p.grpmap >> (4 * q1)) & 7u) << 7);
                    char* f = (lane < 32 ? w0 : w1) + WS_MBOX + (lane & 31) * 1024 + shard * 4;
                    asm volatile("global_store_dword %0, %1, off sc0 sc1" :: "v"(f), "v"(call) : "memory");
                }
            }
        }
        if (wid == 0) spin_flags(mbox + (lane & 31), (lane & 31) == 0 || ((lane & 31) >= 8 && (lane & 31) < 24) ? call : 0u, call, tmo);
    } else {
        const int j = wid - 4;
        if (j >= 2) __builtin_amdgcn_s_sleep(32);
#pragma unroll
        for (int c = 0; c < 8; ++c) BLDS(rW, (j * 8 + c) * 1024, 16, gvoff, gsoff + j * WPAIR + c * 128, 2);
        asm volatile("s_load_dword %0, %1, 0x0\n\ts_waitcnt lgkmcnt(0)" : "=&s"(callv) : "s"(ctr) : "memory");
        call = callv + 1u;
        const int rd = j * 8192 + (lane >> 5) * 1024 + (lane & 31) * 4;
        const int dv = ((lane >> 4) << 11) + ((lane & 15) << 4);
#pragma unroll
        for (int c = 0; c < 4; ++c) {
            if (c == 0) WAIT_V(6); else if (c == 1) WAIT_V(5); else if (c == 2) WAIT_V(4); else WAIT_V(3);
            const LAS float* c0 = (const LAS float*)(lds + rd + c * 2048);
            const LAS float* c1 = (const LAS float*)(lds + (rd ^ 64) + 512 + c * 2048);
            const h8 hv = cvt8((f32x4){c0[0], c0[32], c0[64], c0[96]}, (f32x4){c1[0], c1[32], c1[64], c1[96]});
            __builtin_amdgcn_raw_buffer_store_b128(__builtin_bit_cast(u32x4, hv), rWS, dv, wsoff + j * 2 * WKT_BYTES + c * 8192, 16);
        }
        STAMP(1);
        if (j < 2) {
            WAIT_V(0);
            STAMP(2);
            unsigned old = 0;
            if (lane == 0) old = __hip_atomic_fetch_add(arr + 1, 1u, __ATOMIC_RELAXED, __HIP_MEMORY_SCOPE_WORKGROUP);
            if (__builtin_amdgcn_readfirstlane(old) == 1u && lane < 16) {
                const int cb = ((nt * 2 + (lane & 1)) << 3) + (int)((p.invmap >> (4 * (lane >> 1))) & 7u);
                char* f = myws + WS_MBOX + cb * 128 + (8 + mt) * 4;
                asm volatile("global_store_dword %0, %1, off sc1" :: "v"(f), "v"(call) : "memory");
            }
        }
    }
    STAMP(6);
    BAR;
    STAMP(5);
    const auto rX = RSRC(p.ws[src] + WS_X16 + (call & 1u) * X16_BYTES, X16_BYTES);
    const int wr = wid >> 2, wc = wid & 3;
    const int tv = tid * 16;
    const int xs0 = (mt & 1) * 2 * UNIT, ws0 = WS_W16 + nt * 2 * UNIT;
    const int ldsw = wid * 1024;
    const int aoff = wr * 8192 + lane * 16, boff = wc * 4096 + lane * 16;
    const int qgs = gsoff + wr * WPAIR + wc * 256;
    const int qws = wsoff + wr * 2 * WKT_BYTES + wc * 8192;
    const int qdv = ((lane >> 4) << 11) + ((lane & 15) << 4);
    const int qrd = SCR + wid * 2048 + (lane >> 5) * 1024 + (lane & 31) * 4;
    const int pl = (lane >= 24) ? (lane & 15) : lane;
    const int pthr = (pl < 16) ? 0 : 2 * (pl - 16) - 1;
    const int pinc = (pl < 16) ? 64 : 0;
    int poff = (pl < 16) ? WS_WFLAG + (nt * NQ + 1) * 64 + pl * 4 : WS_MBOX + bid * 128 + (pl - 16) * 4;
#define SA(b, h) (((b) * 2 + (h)) * 16384)
#define SB(b, h) ((4 + (b) * 2 + (h)) * 16384)
#define STX(b, h, kt) do { BLDS(rX, SA(b, h) + ldsw, 16, tv, xs0 + (kt) * XKT_BYTES + (h) * UNIT, 0); \
                           BLDS(rX, SA(b, h) + ldsw + 8192, 16, tv, xs0 + (kt) * XKT_BYTES + (h) * UNIT + 8192, 0); } while (0)
#define STW(b, h, kt) do { BLDS(rWS, SB(b, h) + ldsw, 16, tv, ws0 + (kt) * WKT_BYTES + (h) * UNIT, 0); \
                           BLDS(rWS, SB(b, h) + ldsw + 8192, 16, tv, ws0 + (kt) * WKT_BYTES + (h) * UNIT + 8192, 0); } while (0)
#define LDA(dst, b, h) do { _Pragma("unroll") for (int m = 0; m < 4; ++m) _Pragma("unroll") for (int k = 0; k < 2; ++k) \
        dst[m][k] = *(const LAS h8*)(lds + SA(b, h) + aoff + m * 2048 + k * 1024); } while (0)
#define LDB(dst, b, h) do { _Pragma("unroll") for (int n = 0; n < 2; ++n) _Pragma("unroll") for (int k = 0; k < 2; ++k) \
        dst[n][k] = *(const LAS h8*)(lds + SB(b, h) + boff + n * 2048 + k * 1024); } while (0)
#define MMQ(ai, bj, Bt) asm volatile( \
            "v_mfma_f32_16x16x32_f16 %0, %8, %12, %0\n\t" \
            "v_mfma_f32_16x16x32_f16 %1, %10, %12, %1\n\t" \
            "v_mfma_f32_16x16x32_f16 %2, %8, %14, %2\n\t" \
            "v_mfma_f32_16x16x32_f16 %3, %10, %14, %3\n\t" \
            "v_mfma_f32_16x16x32_f16 %4, %8, %16, %4\n\t" \
            "v_mfma_f32_16x16x32_f16 %5, %10, %16, %5\n\t" \
            "v_mfma_f32_16x16x32_f16 %6, %8, %18, %6\n\t" \
            "v_mfma_f32_16x16x32_f16 %7, %10, %18, %7\n\t" \
            "v_mfma_f32_16x16x32_f16 %0, %9, %13, %0\n\t" \
            "v_mfma_f32_16x16x32_f16 %1, %11, %13, %1\n\t" \
            "v_mfma_f32_16x16x32_f16 %2, %9, %15, %2\n\t" \
            "v_mfma_f32_16x16x32_f16 %3, %11, %15, %3\n\t" \
            "v_mfma_f32_16x16x32_f16 %4, %9, %17, %4\n\t" \
            "v_mfma_f32_16x16x32_f16 %5, %11, %17, %5\n\t" \
            "v_mfma_f32_16x16x32_f16 %6, %9, %19, %6\n\t" \
            "v_mfma_f32_16x16x32_f16 %7, %11, %19, %7\n\t" \
            : "+v"(acc[ai][bj][0][0]), "+v"(acc[ai][bj][0][1]), "+v"(acc[ai][bj][1][0]), "+v"(acc[ai][bj][1][1]), \
              "+v"(acc[ai][bj][2][0]), "+v"(acc[ai][bj][2][1]), "+v"(acc[ai][bj][3][0]), "+v"(acc[ai][bj][3][1]) \
            : "v"(Bt[0][0]), "v"(Bt[0][1]), "v"(Bt[1][0]), "v"(Bt[1][1]), \
              "v"(At[0][0]), "v"(At[0][1]), "v"(At[1][0]), "v"(At[1][1]), "v"(At[2][0]), "v"(At[2][1]), "v"(At[3][0]), "v"(At[3][1]))
#define COMPUTE(ai) do { BAR; SCHED; __builtin_amdgcn_s_setprio(1); MMQ(ai, 0, B0); MMQ(ai, 1, B1); __builtin_amdgcn_s_setprio(0); BAR; SCHED; } while (0)
#define SEGA(b, tn) do { STX((b) ^ 1, 1, tn); LDB(B0, b, 0); LDA(At, b, 0); LDB(B1, b, 1); } while (0)
#define SEGB(b, tn) do { STX(b, 0, tn); LDA(At, b, 1); STW(b, 0, tn); STW(b, 1, tn); } while (0)
#define CHECKQ(q) do { asm volatile("" : "+v"(pv)); \
        if (__builtin_expect(!__all((int)(pv >= call) | (int)(pthr > (q))), 0)) \
            if (wid == 0) spin_flags((const unsigned*)(myws + poff), pthr > (q) ? 0u : call, call, tmo); } while (0)

    f32x4 acc[2][2][4][2];
#pragma unroll
    for (int a = 0; a < 2; ++a)
#pragma unroll
        for (int b = 0; b < 2; ++b)
#pragma unroll
            for (int m = 0; m < 4; ++m)
#pragma unroll
                for (int n = 0; n < 2; ++n) acc[a][b][m][n] = (f32x4){0.f, 0.f, 0.f, 0.f};
    h8 At[4][2], B0[2][2], B1[2][2];

    STX(0, 0, 0); STX(0, 1, 0); STW(0, 0, 0); STW(0, 1, 0);
    STX(1, 0, 1); STW(1, 0, 1); STW(1, 1, 1);
    WAIT_V(6); BAR;
    if (wid == 2 && lane == 0) asm volatile("global_store_dword %0, %1, off sc1" :: "v"(wflag_row + 16 + mt), "v"(call) : "memory");
    if (wr == 1) BAR;
    STAMP(8); CSTAMP(50);
#pragma nounroll
    for (int q = 0; q < NQ - 2; ++q) {
        const int t = q * 4;
        BLDS(rW, SCR + 2 * ldsw, 16, gvoff, qgs + (q + 2) * (2 * WPAIR), 2);
        BLDS(rW, SCR + 2 * ldsw + 1024, 16, gvoff, qgs + (q + 2) * (2 * WPAIR) + 128, 2);
        SEGA(0, t + 1); WAIT_V(10); WAIT_L(0); COMPUTE(0);
        SEGB(0, t + 2);
        if (wid == 0) BLDS(rWS, POLL, 4, poff, 0, 17);
        WAIT_VX(10, wid, 0); WAIT_L(0); COMPUTE(1);
        SEGA(1, t + 2); WAIT_VX(8, wid, 0); WAIT_L(0); COMPUTE(0);
        const LAS float* c0 = (const LAS float*)(lds + qrd);
        const LAS float* c1 = (const LAS float*)(lds + (qrd ^ 64) + 512);
        const float f0 = c0[0], f1 = c0[32], f2 = c0[64], f3 = c0[96], f4 = c1[0], f5 = c1[32], f6 = c1[64], f7 = c1[96];
        SEGB(1, t + 3);
        SCHED;
        { const h8 hv = cvt8((f32x4){f0, f1, f2, f3}, (f32x4){f4, f5, f6, f7});
          __builtin_amdgcn_raw_buffer_store_b128(__builtin_bit_cast(u32x4, hv), rWS, qdv, qws + (q + 2) * (4 * WKT_BYTES), 16); }
        WAIT_V(9); WAIT_L(0); COMPUTE(1);
        unsigned pv = *(const volatile LAS unsigned*)(lds + POLL + lane * 4);
        SEGA(0, t + 3); WAIT_V(9); WAIT_L(0);
        CHECKQ(q);
        poff += pinc;
        COMPUTE(0);
        SEGB(0, t + 4); WAIT_V(8); WAIT_L(0); COMPUTE(1);
        if (wid == 1) { if (lane == 0) __builtin_amdgcn_raw_buffer_store_b32(call, rWS, 0, WS_WFLAG + ((nt * NQ + q + 2) * 16 + mt) * 4, 16); }
        SEGA(1, t + 4); WAIT_VX(8, wid, 1); WAIT_L(0); COMPUTE(0);
        SEGB(1, t + 5); WAIT_VX(8, wid, 1); WAIT_L(0); COMPUTE(1);
    }
#pragma nounroll
    for (int it = NIT - 4; it < NIT - 1; ++it) {
        const int t = it * 2;
        const int sel = __builtin_amdgcn_readfirstlane((it == NIT - 4) ? wid : 1);
        unsigned pv = *(const volatile LAS unsigned*)(lds + POLL + lane * 4);
        SEGA(0, t + 1); WAIT_V(8); WAIT_L(0);
        if (it == NIT - 3) CHECKQ(NQ - 2);
        COMPUTE(0);
        SEGB(0, t + 2);
        if (sel == 0) BLDS(rWS, POLL, 4, poff, 0, 17);
        WAIT_VX(8, sel, 0); WAIT_L(0); COMPUTE(1);
        SEGA(1, t + 2); WAIT_VX(8, sel, 0); WAIT_L(0); COMPUTE(0);
        SEGB(1, t + 3); WAIT_V(8); WAIT_L(0); COMPUTE(1);
    }
    STAMP(39); CSTAMP(51);
    int ln;
    asm volatile("v_mbcnt_lo_u32_b32 %0, -1, 0\n\tv_mbcnt_hi_u32_b32 %0, -1, %0" : "=v"(ln));
    const int fr = ln & 15, fq = ln >> 4;
    float* obase = p.out + (size_t)(mt * 256 + wr * 64 + fr) * NLOC + nt * 256 + wc * 32 + fq * 4;
    const float qnan = __builtin_nanf("");
    unsigned tmov;
#define OUTQ(ai, bj) do { _Pragma("unroll") for (int m = 0; m < 4; ++m) _Pragma("unroll") for (int n = 0; n < 2; ++n) { \
        f32x4 v = acc[ai][bj][m][n]; \
        v[0] = fmaxf(v[0], 0.f); v[1] = fmaxf(v[1], 0.f); v[2] = fmaxf(v[2], 0.f); v[3] = fmaxf(v[3], 0.f); \
        if (bad) v = (f32x4){qnan, qnan, qnan, qnan}; \
        float* dst = obase + (size_t)((ai) * 128 + m * 16) * NLOC + (bj) * 128 + n * 16; \
        asm volatile("global_store_dwordx4 %0, %1, off sc1\n\ts_nop 1" :: "v"(dst), "v"(v) : "memory"); } } while (0)
    {
        SEGA(0, NKT - 1); WAIT_V(8); WAIT_L(0); COMPUTE(0);
        LDA(At, 0, 1); WAIT_V(2); WAIT_L(0); COMPUTE(1);
        LDB(B0, 1, 0); LDA(At, 1, 0); LDB(B1, 1, 1);
        asm volatile("global_load_dword %0, %1, off sc1\n\ts_waitcnt vmcnt(0)" : "=&v"(tmov) : "v"(tmo) : "memory");
        WAIT_L(0);
        COMPUTE(0);
        const bool bad = __builtin_amdgcn_readfirstlane(tmov) == call;
        asm volatile("s_nop 15" ::: "memory"); SCHED;
        LDA(At, 1, 1);
        OUTQ(0, 0); OUTQ(0, 1);
        WAIT_L(0);
        BAR; SCHED; __builtin_amdgcn_s_setprio(1); MMQ(1, 0, B0); MMQ(1, 1, B1); __builtin_amdgcn_s_setprio(0);
        asm volatile("s_nop 15\n\ts_nop 15" ::: "memory"); SCHED;
        STAMP(40);
        OUTQ(1, 0); OUTQ(1, 1);
    }
    if (ln == 0) asm volatile("global_store_dword %0, %1, off sc1" :: "v"(ctr), "v"(call) : "memory");
    STAMP(41);
    WAIT_V(0);
    STAMP(42);
    if (wr == 0) BAR;
}

}

extern "C" void kernel_launch(int rank, int nranks, ncclComm_t comm,
                              hipStream_t stream,
                              void* const* d_in, const int* in_sizes, int n_in,
                              void* d_out, int out_size,
                              const int* in_dtypes, int out_dtype,
                              void* const* d_out_all,
                              void* const* d_ws_all, size_t ws_bytes) {
    if (nranks != NRANKS || (unsigned)rank >= (unsigned)NRANKS || ws_bytes < WS_NEEDED + (8u << 20) || n_in < 2 ||
        in_sizes[0] != MLOC * K_ || in_sizes[1] != K_ * NFULL || out_size != M_ * NLOC) {
        fprintf(stderr, "kernel_launch: unexpected problem shape\n");
        return;
    }
    HIP_CHECK(hipFuncSetAttribute((const void*)fused_kernel, hipFuncAttributeMaxDynamicSharedMemorySize, LDS_BYTES));
    Params p;
    p.x = (const float*)d_in[0];
    p.w = (const float*)d_in[1];
    p.out = (float*)d_out;
    const size_t wsoff = (ws_bytes - WS_NEEDED - (4u << 20)) & ~(size_t)((2u << 20) - 1);
    for (int q = 0; q < NRANKS; ++q) p.ws[q] = (char*)d_ws_all[q] + wsoff;
    p.rank = rank;
    {
        const unsigned half_mask = 0x4Du;
        p.srcmap = p.grpmap = p.invmap = 0;
        for (int r = 0; r < NRANKS; ++r) {
            const int mine = (half_mask >> r) & 1;
            int g = 0;
            for (int pass = 0; pass < 2; ++pass)
                for (int d = 0; d < NRANKS; ++d) {
                    const int q = (r + d) & 7;
                    if (((int)((half_mask >> q) & 1) == mine) != (pass == 0)) continue;
                    if (r == rank) { p.srcmap |= (unsigned)q << (4 * g); p.invmap |= (unsigned)g << (4 * q); }
                    if (q == rank) p.grpmap |= (unsigned)g << (4 * r);
                    ++g;
                }
        }
    }
    hipLaunchKernelGGL(fused_kernel, dim3(256), dim3(512), LDS_BYTES, stream, p);
    HIP_CHECK(hipGetLastError());
}
```

```cpp
#include <hip/hip_runtime.h>
#include <rccl/rccl.h>
#include <cstdio>

#define HIP_CHECK(expr) do { hipError_t _e = (expr); if (_e != hipSuccess) { \
    fprintf(stderr, "HIP error %s at %s:%d\n", hipGetErrorString(_e), __FILE__, __LINE__); \
    return; } } while (0)

namespace {

typedef _Float16 h8 __attribute__((ext_vector_type(8)));
typedef _Float16 h4 __attribute__((ext_vector_type(4)));
typedef float f32x4 __attribute__((ext_vector_type(4)));
typedef unsigned u32x4 __attribute__((ext_vector_type(4)));
#define LAS __attribute__((address_space(3)))

constexpr int NRANKS = 8;
constexpr int M_ = 4096, K_ = 4096, NLOC = 4096, NFULL = 32768, MLOC = 512;
constexpr int NKT = K_ / 64;
constexpr int NIT = NKT / 2;
constexpr int NQ = NKT / 4;
constexpr int UNIT = 16384;
constexpr int WKT_BYTES = 32 * UNIT;
constexpr int XKT_BYTES = 4 * UNIT;
constexpr int X16_BYTES = NKT * XKT_BYTES;
constexpr int W16_BYTES = NKT * WKT_BYTES;
constexpr int WROW = NFULL * 4;
constexpr int WPAIR = 128 * WROW;
constexpr int WS_TMO = 0;
constexpr int WS_CTR = 4096;
constexpr int WS_XCNT = WS_CTR + 256 * 128;
constexpr int WS_MBOX = WS_XCNT + 8 * 128;
constexpr int WS_CLAIM = WS_MBOX + 256 * 128;
constexpr int WS_WFLAG = WS_CLAIM + 32 * 128;
constexpr int WS_X16 = 1 << 20;
constexpr int WS_W16 = WS_X16 + 2 * X16_BYTES;
constexpr size_t WS_NEEDED = (size_t)WS_W16 + W16_BYTES;
static_assert(WS_WFLAG + 16 * NQ * 16 * 4 <= WS_X16, "control area");
constexpr unsigned long long DEADLINE_TICKS = 100000000ull;
constexpr int SCR = 8 * 16384;
constexpr int POLL = SCR + 16384;
constexpr int ARR = POLL + 256;
constexpr int LDS_BYTES = ARR + 16;

struct Params {
    const float* x;
    const float* w;
    float* out;
    char* ws[NRANKS];
    int rank;
    unsigned srcmap;
    unsigned grpmap;
};

__device__ __forceinline__ h8 cvt8(f32x4 a, f32x4 b) {
    h8 r;
    r[0] = (_Float16)a[0]; r[1] = (_Float16)a[1]; r[2] = (_Float16)a[2]; r[3] = (_Float16)a[3];
    r[4] = (_Float16)b[0]; r[5] = (_Float16)b[1]; r[6] = (_Float16)b[2]; r[7] = (_Float16)b[3];
    return r;
}

#define WAIT_V(n) asm volatile("s_waitcnt vmcnt(%0)" :: "n"(n) : "memory")
#define WAIT_L(n) do { asm volatile("" ::: "memory"); __builtin_amdgcn_s_waitcnt(0xC07F); asm volatile("" ::: "memory"); } while (0)
#define WAIT_VX(n, sel, w) asm volatile("s_waitcnt vmcnt(%0)\n\ts_cmp_eq_u32 %2, %3\n\ts_cbranch_scc1 1f\n\ts_waitcnt vmcnt(%1)\n1:" \
        :: "n"((n) + 1), "n"(n), "s"(sel), "n"(w) : "memory", "scc")
#define BAR __builtin_amdgcn_s_barrier()
#define SCHED __builtin_amdgcn_sched_barrier(0)
#define RSRC(ptr, bytes) __builtin_amdgcn_make_buffer_rsrc((void*)(ptr), 0, (bytes), 0x00020000)
#define BLDS(rsrc, ldsoff, size, voff, soff, aux) \
        __builtin_amdgcn_raw_ptr_buffer_load_lds(rsrc, (LAS void*)(lds + (ldsoff)), size, voff, soff, 0, aux)

__device__ __forceinline__ void spin_flags(const unsigned* lane_ptr, unsigned need, unsigned call, unsigned* tmo) {
    const unsigned long long t0 = __builtin_amdgcn_s_memrealtime();
    for (unsigned n = 1;; ++n) {
        unsigned v;
        asm volatile("global_load_dword %0, %1, off sc0 sc1\n\ts_waitcnt vmcnt(0)" : "=&v"(v) : "v"(lane_ptr) : "memory");
        if (__all(v >= need)) return;
        if ((n & 15u) == 0u) {
            if (__builtin_amdgcn_readfirstlane(__hip_atomic_load(tmo, __ATOMIC_RELAXED, __HIP_MEMORY_SCOPE_AGENT)) == call
                || __builtin_amdgcn_s_memrealtime() - t0 > DEADLINE_TICKS) {
                __hip_atomic_store(tmo, call, __ATOMIC_RELAXED, __HIP_MEMORY_SCOPE_AGENT);
                return;
            }
        }
        __builtin_amdgcn_s_sleep(1);
    }
}

__global__ __launch_bounds__(512, 2) void fused_kernel(Params p) {
    extern __shared__ __attribute__((aligned(16))) unsigned char shm[];
    LAS unsigned char* lds = (LAS unsigned char*)shm;
    const int tid = threadIdx.x;
    const int wid = __builtin_amdgcn_readfirstlane(tid >> 6), lane = tid & 63;
    char* myws = p.ws[p.rank];
    const int bid = blockIdx.x;
    const int xi = bid >> 3, nt = xi >> 1;
    const int kg = (bid & 7) * 2 + (xi & 1);
    unsigned* tmo = (unsigned*)(myws + WS_TMO);
#define STAMP(idx) do {} while (0)
#define CSTAMP(idx) do {} while (0)
    STAMP(0);
    const auto rWS = RSRC(myws, (int)WS_NEEDED);
    const auto rW = RSRC(p.w + p.rank * NLOC + nt * 256, (K_ - 1) * WROW + 1024);
    unsigned* ctr = (unsigned*)(myws + WS_CTR + bid * 128 + wid * 4);
    unsigned callv, call;
    const int gvoff = (lane >> 3) * WROW + (((lane & 7) ^ ((lane >> 5) << 2)) << 4);
    const int gsoff = kg * 8 * WROW;
    const int wsoff = WS_W16 + (kg >> 3) * WKT_BYTES + nt * 2 * UNIT + (((kg >> 2) & 1) << 10) + ((kg & 3) << 8);
    unsigned* wflag_row = (unsigned*)(myws + WS_WFLAG) + nt * NQ * 16;
    LAS unsigned* arr = (LAS unsigned*)(lds + ARR);
    if (tid < 3) arr[tid] = 0u;
    WAIT_L(0); BAR;
    if (wid < 4) {
        const int xkt = bid >> 2, xmh = bid & 3;
        const float* xsrc = p.x + (size_t)(xmh * 128 + wid * 32 + (lane & 15)) * K_ + xkt * 64 + (lane >> 4) * 8;
        const float* xsrc2 = xsrc + 16 * K_;
        f32x4 a0, a1, a2, a3, b0, b1, b2, b3;
        asm volatile("global_load_dwordx4 %0, %9, off nt\n\tglobal_load_dwordx4 %1, %9, off offset:16 nt\n\t"
                     "global_load_dwordx4 %2, %9, off offset:128 nt\n\tglobal_load_dwordx4 %3, %9, off offset:144 nt\n\t"
                     "global_load_dwordx4 %4, %10, off nt\n\tglobal_load_dwordx4 %5, %10, off offset:16 nt\n\t"
                     "global_load_dwordx4 %6, %10, off offset:128 nt\n\tglobal_load_dwordx4 %7, %10, off offset:144 nt\n\t"
                     "s_load_dword %8, %11, 0x0\n\t"
                     "s_waitcnt vmcnt(0) lgkmcnt(0)"
                     : "=&v"(a0), "=&v"(a1), "=&v"(a2), "=&v"(a3), "=&v"(b0), "=&v"(b1), "=&v"(b2), "=&v"(b3), "=&s"(callv)
                     : "v"(xsrc), "v"(xsrc2), "s"(ctr) : "memory");
        call = callv + 1u;
        STAMP(1);
        char* xd = myws + WS_X16 + (call & 1u) * X16_BYTES + xkt * XKT_BYTES + xmh * UNIT + wid * 4096 + lane * 16;
        const h8 v0 = cvt8(a0, a1), v1 = cvt8(a2, a3), v2 = cvt8(b0, b1), v3 = cvt8(b2, b3);
        asm volatile("global_store_dwordx4 %0, %1, off sc0 sc1\n\tglobal_store_dwordx4 %0, %2, off offset:1024 sc0 sc1\n\t"
                     "global_store_dwordx4 %0, %3, off offset:2048 sc0 sc1\n\tglobal_store_dwordx4 %0, %4, off offset:3072 sc0 sc1\n\t"
                     "s_waitcnt vmcnt(0)"
                     :: "v"(xd), "v"(v0), "v"(v1), "v"(v2), "v"(v3) : "memory");
        STAMP(2);
        unsigned old = 0;
        if (lane == 0) old = __hip_atomic_fetch_add(arr, 1u, __ATOMIC_RELAXED, __HIP_MEMORY_SCOPE_WORKGROUP);
        if (__builtin_amdgcn_readfirstlane(old) == 3u) {
            const int shard = bid >> 5;
            unsigned cnt = 0;
            if (lane == 0) cnt = __hip_atomic_fetch_add((unsigned*)(myws + WS_XCNT + shard * 128), 1u, __ATOMIC_RELAXED, __HIP_MEMORY_SCOPE_AGENT);
            if (__builtin_amdgcn_readfirstlane(cnt) + 1u == 32u * call) {
#pragma unroll
                for (int i = 0; i < 4; ++i) {
                    const int q0 = 2 * i, q1 = 2 * i + 1;
                    char* w0 = p.ws[q0] + (((p.grpmap >> (4 * q0)) & 7u) << 7);
                    char* w1 = p.ws[q1] + (((p.grpmap >> (4 * q1)) & 7u) << 7);
                    char* f = (lane < 32 ? w0 : w1) + WS_MBOX + (lane & 31) * 1024 + shard * 4;
                    asm volatile("global_store_dword %0, %1, off sc0 sc1" :: "v"(f), "v"(call) : "memory");
                }
            }
        }
        if (wid == 0) {
            unsigned g1;
            while ((g1 = __builtin_amdgcn_readfirstlane(*(const volatile LAS unsigned*)(arr + 2))) == 0u) __builtin_amdgcn_s_sleep(1);
            const unsigned* mbox = (const unsigned*)(myws + WS_MBOX + (xi * 8 + g1 - 1) * 128);
            spin_flags(mbox + (lane & 31), (lane & 31) == 0 || ((lane & 31) >= 8 && (lane & 31) < 24) ? call : 0u, call, tmo);
        }
    } else {
        const int j = wid - 4;
#pragma unroll
        for (int c = 0; c < 8; ++c) BLDS(rW, (j * 8 + c) * 1024, 16, gvoff, gsoff + j * WPAIR + c * 128, 2);
        asm volatile("s_load_dword %0, %1, 0x0\n\ts_waitcnt lgkmcnt(0)" : "=&s"(callv) : "s"(ctr) : "memory");
        call = callv + 1u;
        if (j == 3) {
            unsigned g = (unsigned)__builtin_amdgcn_s_getreg(20 | (3 << 11)) & 7u;
            unsigned* cl = (unsigned*)(myws + WS_CLAIM + xi * 128);
            for (int i = 0; i < 8; ++i) {
                unsigned old = 0;
                if (lane == 0) old = __hip_atomic_fetch_max(cl + g, call, __ATOMIC_RELAXED, __HIP_MEMORY_SCOPE_AGENT);
                if (__builtin_amdgcn_readfirstlane(old) < call) break;
                g = (g + 1u) & 7u;
            }
            if (lane == 0) *(volatile LAS unsigned*)(arr + 2) = g + 1u;
        }
        const int rd = j * 8192 + (lane >> 5) * 1024 + (lane & 31) * 4;
        const int dv = ((lane >> 4) << 11) + ((lane & 15) << 4);
#pragma unroll
        for (int c = 0; c < 4; ++c) {
            if (c == 0) WAIT_V(6); else if (c == 1) WAIT_V(5); else if (c == 2) WAIT_V(4); else WAIT_V(3);
            const LAS float* c0 = (const LAS float*)(lds + rd + c * 2048);
            const LAS float* c1 = (const LAS float*)(lds + (rd ^ 64) + 512 + c * 2048);
            const h8 hv = cvt8((f32x4){c0[0], c0[32], c0[64], c0[96]}, (f32x4){c1[0], c1[32], c1[64], c1[96]});
            __builtin_amdgcn_raw_buffer_store_b128(__builtin_bit_cast(u32x4, hv), rWS, dv, wsoff + j * 2 * WKT_BYTES + c * 8192, 16);
        }
        STAMP(1);
        if (j < 2) {
            WAIT_V(0);
            STAMP(2);
            unsigned old = 0;
            if (lane == 0) old = __hip_atomic_fetch_add(arr + 1, 1u, __ATOMIC_RELAXED, __HIP_MEMORY_SCOPE_WORKGROUP);
            if (__builtin_amdgcn_readfirstlane(old) == 1u && lane < 16) {
                char* f = myws + WS_MBOX + (nt * 16 + lane) * 128 + (8 + kg) * 4;
                asm volatile("global_store_dword %0, %1, off sc1" :: "v"(f), "v"(call) : "memory");
            }
        }
    }
    STAMP(6);
    WAIT_L(0);
    BAR;
    STAMP(5);
    const int xg = (__builtin_amdgcn_readfirstlane(*(const volatile LAS unsigned*)(arr + 2)) - 1) & 7;
    const int src = (p.srcmap >> (xg * 4)) & (NRANKS - 1);
    const int mt = src * 2 + (xi & 1);
    const auto rX = RSRC(p.ws[src] + WS_X16 + (call & 1u) * X16_BYTES, X16_BYTES);
    const int wr = wid >> 2, wc = wid & 3;
    const int tv = tid * 16;
    const int xs0 = (mt & 1) * 2 * UNIT, ws0 = WS_W16 + nt * 2 * UNIT;
    const int ldsw = wid * 1024;
    const int aoff = wr * 8192 + lane * 16, boff = wc * 4096 + lane * 16;
    const int qgs = gsoff + wr * WPAIR + wc * 256;
    const int qws = wsoff + wr * 2 * WKT_BYTES + wc * 8192;
    const int qdv = ((lane >> 4) << 11) + ((lane & 15) << 4);
    const int qrd = SCR + wid * 2048 + (lane >> 5) * 1024 + (lane & 31) * 4;
    const int pl = (lane >= 24) ? (lane & 15) : lane;
    const int pthr = (pl < 16) ? 0 : 2 * (pl - 16) - 1;
    const int pinc = (pl < 16) ? 64 : 0;
    int poff = (pl < 16) ? WS_WFLAG + (nt * NQ + 1) * 64 + pl * 4 : WS_MBOX + (xi * 8 + xg) * 128 + (pl - 16) * 4;
#define SA(b, h) (((b) * 2 + (h)) * 16384)
#define SB(b, h) ((4 + (b) * 2 + (h)) * 16384)
#define STX(b, h, kt) do { BLDS(rX, SA(b, h) + ldsw, 16, tv, xs0 + (kt) * XKT_BYTES + (h) * UNIT, 0); \
                           BLDS(rX, SA(b, h) + ldsw + 8192, 16, tv, xs0 + (kt) * XKT_BYTES + (h) * UNIT + 8192, 0); } while (0)
#define STW(b, h, kt) do { BLDS(rWS, SB(b, h) + ldsw, 16, tv, ws0 + (kt) * WKT_BYTES + (h) * UNIT, 0); \
                           BLDS(rWS, SB(b, h) + ldsw + 8192, 16, tv, ws0 + (kt) * WKT_BYTES + (h) * UNIT + 8192, 0); } while (0)
#define LDA(dst, b, h) do { _Pragma("unroll") for (int m = 0; m < 4; ++m) _Pragma("unroll") for (int k = 0; k < 2; ++k) \
        dst[m][k] = *(const LAS h8*)(lds + SA(b, h) + aoff + m * 2048 + k * 1024); } while (0)
#define LDB(dst, b, h) do { _Pragma("unroll") for (int n = 0; n < 2; ++n) _Pragma("unroll") for (int k = 0; k < 2; ++k) \
        dst[n][k] = *(const LAS h8*)(lds + SB(b, h) + boff + n * 2048 + k * 1024); } while (0)
#define MMH(ai, h) asm volatile( \
            "v_mfma_f32_16x16x32_f16 %0, %8, %16, %0\n\t" \
            "v_mfma_f32_16x16x32_f16 %0, %9, %17, %0\n\t" \
            "v_mfma_f32_16x16x32_f16 %1, %11, %17, %1\n\t" \
            "v_mfma_f32_16x16x32_f16 %1, %10, %16, %1\n\t" \
            "v_mfma_f32_16x16x32_f16 %2, %12, %16, %2\n\t" \
            "v_mfma_f32_16x16x32_f16 %2, %13, %17, %2\n\t" \
            "v_mfma_f32_16x16x32_f16 %3, %15, %17, %3\n\t" \
            "v_mfma_f32_16x16x32_f16 %3, %14, %16, %3\n\t" \
            "v_mfma_f32_16x16x32_f16 %7, %14, %18, %7\n\t" \
            "v_mfma_f32_16x16x32_f16 %7, %15, %19, %7\n\t" \
            "v_mfma_f32_16x16x32_f16 %6, %13, %19, %6\n\t" \
            "v_mfma_f32_16x16x32_f16 %6, %12, %18, %6\n\t" \
            "v_mfma_f32_16x16x32_f16 %5, %10, %18, %5\n\t" \
            "v_mfma_f32_16x16x32_f16 %5, %11, %19, %5\n\t" \
            "v_mfma_f32_16x16x32_f16 %4, %9, %19, %4\n\t" \
            "v_mfma_f32_16x16x32_f16 %4, %8, %18, %4\n\t" \
            : "+v"(acc[ai][0][2*(h)][0]), "+v"(acc[ai][0][2*(h)][1]), "+v"(acc[ai][1][2*(h)][0]), "+v"(acc[ai][1][2*(h)][1]), \
              "+v"(acc[ai][0][2*(h)+1][0]), "+v"(acc[ai][0][2*(h)+1][1]), "+v"(acc[ai][1][2*(h)+1][0]), "+v"(acc[ai][1][2*(h)+1][1]) \
            : "v"(B0[0][0]), "v"(B0[0][1]), "v"(B0[1][0]), "v"(B0[1][1]), "v"(B1[0][0]), "v"(B1[0][1]), "v"(B1[1][0]), "v"(B1[1][1]), \
              "v"(At[2*(h)][0]), "v"(At[2*(h)][1]), "v"(At[2*(h)+1][0]), "v"(At[2*(h)+1][1]))
#define COMPUTE(ai) do { BAR; SCHED; __builtin_amdgcn_s_setprio(1); MMH(ai, 0); MMH(ai, 1); __builtin_amdgcn_s_setprio(0); BAR; SCHED; } while (0)
#define SEGA(b, tn) do { STX((b) ^ 1, 1, tn); LDB(B0, b, 0); LDA(At, b, 0); LDB(B1, b, 1); } while (0)
#define SEGB(b, tn) do { STX(b, 0, tn); LDA(At, b, 1); STW(b, 0, tn); STW(b, 1, tn); } while (0)
#define CHECKQ(q) do { asm volatile("" : "+v"(pv)); \
        if (__builtin_expect(!__all((int)(pv >= call) | (int)(pthr > (q))), 0)) \
            if (wid == 0) spin_flags((const unsigned*)(myws + poff), pthr > (q) ? 0u : call, call, tmo); } while (0)

    f32x4 acc[2][2][4][2];
#pragma unroll
    for (int a = 0; a < 2; ++a)
#pragma unroll
        for (int b = 0; b < 2; ++b)
#pragma unroll
            for (int m = 0; m < 4; ++m)
#pragma unroll
                for (int n = 0; n < 2; ++n) acc[a][b][m][n] = (f32x4){0.f, 0.f, 0.f, 0.f};
    h8 At[4][2], B0[2][2], B1[2][2];

    STX(0, 0, 0); STX(0, 1, 0); STW(0, 0, 0); STW(0, 1, 0);
    STX(1, 0, 1); STW(1, 0, 1); STW(1, 1, 1);
    WAIT_V(6); BAR;
    if (wid == 2 && lane == 0) asm volatile("global_store_dword %0, %1, off sc1" :: "v"(wflag_row + 16 + kg), "v"(call) : "memory");
    if (wr == 1) BAR;
    STAMP(8); CSTAMP(50);
#pragma nounroll
    for (int q = 0; q < NQ - 2; ++q) {
        const int t = q * 4;
        BLDS(rW, SCR + 2 * ldsw, 16, gvoff, qgs + (q + 2) * (2 * WPAIR), 2);
        BLDS(rW, SCR + 2 * ldsw + 1024, 16, gvoff, qgs + (q + 2) * (2 * WPAIR) + 128, 2);
        SEGA(0, t + 1); WAIT_V(10); WAIT_L(0); COMPUTE(0);
        SEGB(0, t + 2);
        if (wid == 0) BLDS(rWS, POLL, 4, poff, 0, 17);
        WAIT_VX(10, wid, 0); WAIT_L(0); COMPUTE(1);
        SEGA(1, t + 2); WAIT_VX(8, wid, 0); WAIT_L(0); COMPUTE(0);
        const LAS float* c0 = (const LAS float*)(lds + qrd);
        const LAS float* c1 = (const LAS float*)(lds + (qrd ^ 64) + 512);
        const float f0 = c0[0], f1 = c0[32], f2 = c0[64], f3 = c0[96], f4 = c1[0], f5 = c1[32], f6 = c1[64], f7 = c1[96];
        SEGB(1, t + 3);
        SCHED;
        { const h8 hv = cvt8((f32x4){f0, f1, f2, f3}, (f32x4){f4, f5, f6, f7});
          __builtin_amdgcn_raw_buffer_store_b128(__builtin_bit_cast(u32x4, hv), rWS, qdv, qws + (q + 2) * (4 * WKT_BYTES), 16); }
        WAIT_V(9); WAIT_L(0); COMPUTE(1);
        unsigned pv = *(const volatile LAS unsigned*)(lds + POLL + lane * 4);
        SEGA(0, t + 3); WAIT_V(9); WAIT_L(0);
        CHECKQ(q);
        poff += pinc;
        COMPUTE(0);
        SEGB(0, t + 4); WAIT_V(8); WAIT_L(0); COMPUTE(1);
        if (wid == 1) { if (lane == 0) __builtin_amdgcn_raw_buffer_store_b32(call, rWS, 0, WS_WFLAG + ((nt * NQ + q + 2) * 16 + kg) * 4, 16); }
        SEGA(1, t + 4); WAIT_VX(8, wid, 1); WAIT_L(0); COMPUTE(0);
        SEGB(1, t + 5); WAIT_VX(8, wid, 1); WAIT_L(0); COMPUTE(1);
    }
#pragma nounroll
    for (int it = NIT - 4; it < NIT - 1; ++it) {
        const int t = it * 2;
        const int sel = __builtin_amdgcn_readfirstlane((it == NIT - 4) ? wid : 1);
        unsigned pv = *(const volatile LAS unsigned*)(lds + POLL + lane * 4);
        SEGA(0, t + 1); WAIT_V(8); WAIT_L(0);
        if (it == NIT - 3) CHECKQ(NQ - 2);
        COMPUTE(0);
        SEGB(0, t + 2);
        if (sel == 0) BLDS(rWS, POLL, 4, poff, 0, 17);
        WAIT_VX(8, sel, 0); WAIT_L(0); COMPUTE(1);
        SEGA(1, t + 2); WAIT_VX(8, sel, 0); WAIT_L(0); COMPUTE(0);
        SEGB(1, t + 3); WAIT_V(8); WAIT_L(0); COMPUTE(1);
    }
    STAMP(39); CSTAMP(51);
    int ln;
    asm volatile("v_mbcnt_lo_u32_b32 %0, -1, 0\n\tv_mbcnt_hi_u32_b32 %0, -1, %0" : "=v"(ln));
    const int fr = ln & 15, fq = ln >> 4;
    float* obase = p.out + (size_t)(mt * 256 + wr * 64 + fr) * NLOC + nt * 256 + wc * 32 + fq * 4;
    const float qnan = __builtin_nanf("");
    unsigned tmov;
#define OUTQ(ai, bj) do { _Pragma("unroll") for (int m = 0; m < 4; ++m) _Pragma("unroll") for (int n = 0; n < 2; ++n) { \
        f32x4 v = acc[ai][bj][m][n]; \
        v[0] = fmaxf(v[0], 0.f); v[1] = fmaxf(v[1], 0.f); v[2] = fmaxf(v[2], 0.f); v[3] = fmaxf(v[3], 0.f); \
        if (bad) v = (f32x4){qnan, qnan, qnan, qnan}; \
        float* dst = obase + (size_t)((ai) * 128 + m * 16) * NLOC + (bj) * 128 + n * 16; \
        asm volatile("global_store_dwordx4 %0, %1, off sc1\n\ts_nop 1" :: "v"(dst), "v"(v) : "memory"); } } while (0)
    {
        SEGA(0, NKT - 1); WAIT_V(8); WAIT_L(0); COMPUTE(0);
        LDA(At, 0, 1); WAIT_V(2); WAIT_L(0); COMPUTE(1);
        LDB(B0, 1, 0); LDA(At, 1, 0); LDB(B1, 1, 1);
        asm volatile("global_load_dword %0, %1, off sc1\n\ts_waitcnt vmcnt(0)" : "=&v"(tmov) : "v"(tmo) : "memory");
        WAIT_L(0);
        COMPUTE(0);
        const bool bad = __builtin_amdgcn_readfirstlane(tmov) == call;
        asm volatile("s_nop 15" ::: "memory"); SCHED;
        LDA(At, 1, 1);
        OUTQ(0, 0); OUTQ(0, 1);
        WAIT_L(0);
        BAR; SCHED; __builtin_amdgcn_s_setprio(1); MMH(1, 0); MMH(1, 1); __builtin_amdgcn_s_setprio(0);
        asm volatile("s_nop 15\n\ts_nop 15" ::: "memory"); SCHED;
        STAMP(40);
        OUTQ(1, 0); OUTQ(1, 1);
    }
    if (ln == 0) asm volatile("global_store_dword %0, %1, off sc1" :: "v"(ctr), "v"(call) : "memory");
    STAMP(41);
    WAIT_V(0);
    STAMP(42);
    if (wr == 0) BAR;
}

}

extern "C" void kernel_launch(int rank, int nranks, ncclComm_t comm,
                              hipStream_t stream,
                              void* const* d_in, const int* in_sizes, int n_in,
                              void* d_out, int out_size,
                              const int* in_dtypes, int out_dtype,
                              void* const* d_out_all,
                              void* const* d_ws_all, size_t ws_bytes) {
    if (nranks != NRANKS || (unsigned)rank >= (unsigned)NRANKS || ws_bytes < WS_NEEDED + (8u << 20) || n_in < 2 ||
        in_sizes[0] != MLOC * K_ || in_sizes[1] != K_ * NFULL || out_size != M_ * NLOC) {
        fprintf(stderr, "kernel_launch: unexpected problem shape\n");
        return;
    }
    HIP_CHECK(hipFuncSetAttribute((const void*)fused_kernel, hipFuncAttributeMaxDynamicSharedMemorySize, LDS_BYTES));
    Params p;
    p.x = (const float*)d_in[0];
    p.w = (const float*)d_in[1];
    p.out = (float*)d_out;
    const size_t wsoff = (ws_bytes - WS_NEEDED - (4u << 20)) & ~(size_t)((2u << 20) - 1);
    for (int q = 0; q < NRANKS; ++q) p.ws[q] = (char*)d_ws_all[q] + wsoff;
    p.rank = rank;
    {
        const unsigned half_mask = 0x4Du;
        p.srcmap = p.grpmap = 0;
        for (int r = 0; r < NRANKS; ++r) {
            const int mine = (half_mask >> r) & 1;
            int g = 0;
            for (int pass = 0; pass < 2; ++pass)
                for (int d = 0; d < NRANKS; ++d) {
                    const int q = (r + d) & 7;
                    if (((int)((half_mask >> q) & 1) == mine) != (pass == 0)) continue;
                    if (r == rank) p.srcmap |= (unsigned)q << (4 * g);
                    if (q == rank) p.grpmap |= (unsigned)g << (4 * r);
                    ++g;
                }
        }
    }
    hipLaunchKernelGGL(fused_kernel, dim3(256), dim3(512), LDS_BYTES, stream, p);
    HIP_CHECK(hipGetLastError());
}
```

```cpp
#include <hip/hip_runtime.h>
#include <rccl/rccl.h>
#include <cstdio>

#define HIP_CHECK(expr) do { hipError_t _e = (expr); if (_e != hipSuccess) { \
    fprintf(stderr, "HIP error %s at %s:%d\n", hipGetErrorString(_e), __FILE__, __LINE__); \
    return; } } while (0)

namespace {

constexpr int SELFG = 1;
static_assert(SELFG & 1, "the helpers of column tile 0 must not run two jobs of their own");

typedef _Float16 h8 __attribute__((ext_vector_type(8)));
typedef _Float16 h4 __attribute__((ext_vector_type(4)));
typedef float f32x4 __attribute__((ext_vector_type(4)));
typedef unsigned u32x4 __attribute__((ext_vector_type(4)));
#define LAS __attribute__((address_space(3)))

constexpr int NRANKS = 8;
constexpr int M_ = 4096, K_ = 4096, NLOC = 4096, NFULL = 32768, MLOC = 512;
constexpr int NKT = K_ / 64;
constexpr int NIT = NKT / 2;
constexpr int NQ = NKT / 4;
constexpr int UNIT = 16384;
constexpr int WKT_BYTES = 32 * UNIT;
constexpr int XKT_BYTES = 4 * UNIT;
constexpr int X16_BYTES = NKT * XKT_BYTES;
constexpr int W16_BYTES = NKT * WKT_BYTES;
constexpr int WROW = NFULL * 4;
constexpr int WPAIR = 128 * WROW;
constexpr int WS_TMO = 0;
constexpr int WS_CTR = 4096;
constexpr int WS_XCNT = WS_CTR + 256 * 128;
constexpr int WS_MBOX = WS_XCNT + 8 * 128;
constexpr int WS_CLAIM = WS_MBOX + 256 * 128;
constexpr int WS_WFLAG = WS_CLAIM + 32 * 128;
constexpr int WS_X16 = 1 << 20;
constexpr int WS_W16 = WS_X16 + 2 * X16_BYTES;
constexpr size_t WS_NEEDED = (size_t)WS_W16 + W16_BYTES;
static_assert(WS_WFLAG + 16 * NQ * 16 * 4 <= WS_X16, "control area");
constexpr unsigned long long DEADLINE_TICKS = 100000000ull;
constexpr int SCR = 8 * 16384;
constexpr int POLL = SCR + 16384;
constexpr int ARR = POLL + 256;
constexpr int LDS_BYTES = ARR + 16;

struct Params {
    const float* x;
    const float* w;
    float* out;
    char* ws[NRANKS];
    int rank;
    unsigned srcmap;
    unsigned grpmap;
};

__device__ __forceinline__ h8 cvt8(f32x4 a, f32x4 b) {
    h8 r;
    r[0] = (_Float16)a[0]; r[1] = (_Float16)a[1]; r[2] = (_Float16)a[2]; r[3] = (_Float16)a[3];
    r[4] = (_Float16)b[0]; r[5] = (_Float16)b[1]; r[6] = (_Float16)b[2]; r[7] = (_Float16)b[3];
    return r;
}

#define WAIT_V(n) asm volatile("s_waitcnt vmcnt(%0)" :: "n"(n) : "memory")
#define WAIT_L(n) do { asm volatile("" ::: "memory"); __builtin_amdgcn_s_waitcnt(0xC07F); asm volatile("" ::: "memory"); } while (0)
#define WAIT_VX(n, sel, w) asm volatile("s_waitcnt vmcnt(%0)\n\ts_cmp_eq_u32 %2, %3\n\ts_cbranch_scc1 1f\n\ts_waitcnt vmcnt(%1)\n1:" \
        :: "n"((n) + 1), "n"(n), "s"(sel), "n"(w) : "memory", "scc")
#define BAR __builtin_amdgcn_s_barrier()
#define SCHED __builtin_amdgcn_sched_barrier(0)
#define RSRC(ptr, bytes) __builtin_amdgcn_make_buffer_rsrc((void*)(ptr), 0, (bytes), 0x00020000)
#define BLDS(rsrc, ldsoff, size, voff, soff, aux) \
        __builtin_amdgcn_raw_ptr_buffer_load_lds(rsrc, (LAS void*)(lds + (ldsoff)), size, voff, soff, 0, aux)

__device__ __forceinline__ void spin_flags(const unsigned* lane_ptr, unsigned need, unsigned call, unsigned* tmo) {
    const unsigned long long t0 = __builtin_amdgcn_s_memrealtime();
    for (unsigned n = 1;; ++n) {
        unsigned v;
        asm volatile("global_load_dword %0, %1, off sc0 sc1\n\ts_waitcnt vmcnt(0)" : "=&v"(v) : "v"(lane_ptr) : "memory");
        if (__all(v >= need)) return;
        if ((n & 15u) == 0u) {
            if (__builtin_amdgcn_readfirstlane(__hip_atomic_load(tmo, __ATOMIC_RELAXED, __HIP_MEMORY_SCOPE_AGENT)) == call
                || __builtin_amdgcn_s_memrealtime() - t0 > DEADLINE_TICKS) {
                __hip_atomic_store(tmo, call, __ATOMIC_RELAXED, __HIP_MEMORY_SCOPE_AGENT);
                return;
            }
        }
        __builtin_amdgcn_s_sleep(1);
    }
}

__global__ __launch_bounds__(512, 2) void fused_kernel(Params p) {
    extern __shared__ __attribute__((aligned(16))) unsigned char shm[];
    LAS unsigned char* lds = (LAS unsigned char*)shm;
    const int tid = threadIdx.x;
    const int wid = __builtin_amdgcn_readfirstlane(tid >> 6), lane = tid & 63;
    char* myws = p.ws[p.rank];
    const int bid = blockIdx.x;
    const int xi = bid >> 3, nt = xi >> 1;
    const int kg = (bid & 7) * 2 + (xi & 1);
    unsigned* tmo = (unsigned*)(myws + WS_TMO);
#define STAMP(idx) do {} while (0)
#define CSTAMP(idx) do {} while (0)
    STAMP(0);
    const auto rWS = RSRC(myws, (int)WS_NEEDED);
    const auto rW = RSRC(p.w + p.rank * NLOC, (K_ - 1) * WROW + NLOC * 4);
    unsigned* ctr = (unsigned*)(myws + WS_CTR + bid * 128 + wid * 4);
    unsigned callv, call;
    const int gvoff = (lane >> 3) * WROW + (((lane & 7) ^ ((lane >> 5) << 2)) << 4);
    const int gsoff = kg * 8 * WROW + nt * 1024;
    const int wsoff = WS_W16 + (kg >> 3) * WKT_BYTES + nt * 2 * UNIT + (((kg >> 2) & 1) << 10) + ((kg & 3) << 8);
    unsigned* wflag_row = (unsigned*)(myws + WS_WFLAG) + nt * NQ * 16;
    LAS unsigned* arr = (LAS unsigned*)(lds + ARR);
    if (tid < 3) arr[tid] = 0u;
    WAIT_L(0); BAR;
    if (wid < 4) {
        const int xkt = bid >> 2, xmh = bid & 3;
        const float* xsrc = p.x + (size_t)(xmh * 128 + wid * 32 + (lane & 15)) * K_ + xkt * 64 + (lane >> 4) * 8;
        const float* xsrc2 = xsrc + 16 * K_;
        f32x4 a0, a1, a2, a3, b0, b1, b2, b3;
        asm volatile("global_load_dwordx4 %0, %9, off nt\n\tglobal_load_dwordx4 %1, %9, off offset:16 nt\n\t"
                     "global_load_dwordx4 %2, %9, off offset:128 nt\n\tglobal_load_dwordx4 %3, %9, off offset:144 nt\n\t"
                     "global_load_dwordx4 %4, %10, off nt\n\tglobal_load_dwordx4 %5, %10, off offset:16 nt\n\t"
                     "global_load_dwordx4 %6, %10, off offset:128 nt\n\tglobal_load_dwordx4 %7, %10, off offset:144 nt\n\t"
                     "s_load_dword %8, %11, 0x0\n\t"
                     "s_waitcnt vmcnt(0) lgkmcnt(0)"
                     : "=&v"(a0), "=&v"(a1), "=&v"(a2), "=&v"(a3), "=&v"(b0), "=&v"(b1), "=&v"(b2), "=&v"(b3), "=&s"(callv)
                     : "v"(xsrc), "v"(xsrc2), "s"(ctr) : "memory");
        call = callv + 1u;
        STAMP(1);
        char* xd = myws + WS_X16 + (call & 1u) * X16_BYTES + xkt * XKT_BYTES + xmh * UNIT + wid * 4096 + lane * 16;
        const h8 v0 = cvt8(a0, a1), v1 = cvt8(a2, a3), v2 = cvt8(b0, b1), v3 = cvt8(b2, b3);
        asm volatile("global_store_dwordx4 %0, %1, off sc0 sc1\n\tglobal_store_dwordx4 %0, %2, off offset:1024 sc0 sc1\n\t"
                     "global_store_dwordx4 %0, %3, off offset:2048 sc0 sc1\n\tglobal_store_dwordx4 %0, %4, off offset:3072 sc0 sc1\n\t"
                     "s_waitcnt vmcnt(0)"
                     :: "v"(xd), "v"(v0), "v"(v1), "v"(v2), "v"(v3) : "memory");
        STAMP(2);
        unsigned old = 0;
        if (lane == 0) old = __hip_atomic_fetch_add(arr, 1u, __ATOMIC_RELAXED, __HIP_MEMORY_SCOPE_WORKGROUP);
        if (__builtin_amdgcn_readfirstlane(old) == 3u) {
            const int shard = bid >> 5;
            unsigned cnt = 0;
            if (lane == 0) cnt = __hip_atomic_fetch_add((unsigned*)(myws + WS_XCNT + shard * 128), 1u, __ATOMIC_RELAXED, __HIP_MEMORY_SCOPE_AGENT);
            if (__builtin_amdgcn_readfirstlane(cnt) + 1u == 32u * call) {
#pragma unroll
                for (int i = 0; i < 4; ++i) {
                    const int q0 = 2 * i, q1 = 2 * i + 1;
                    char* w0 = p.ws[q0] + (((p.grpmap >> (4 * q0)) & 7u) << 7);
                    char* w1 = p.ws[q1] + (((p.grpmap >> (4 * q1)) & 7u) << 7);
                    char* f = (lane < 32 ? w0 : w1) + WS_MBOX + (lane & 31) * 1024 + shard * 4;
                    asm volatile("global_store_dword %0, %1, off sc0 sc1" :: "v"(f), "v"(call) : "memory");
                }
            }
        }
        if (wid == 0) {
            unsigned g1;
            while ((g1 = __builtin_amdgcn_readfirstlane(*(const volatile LAS unsigned*)(arr + 2))) == 0u) __builtin_amdgcn_s_sleep(1);
            const unsigned* mbox = (const unsigned*)(myws + WS_MBOX + (xi * 8 + g1 - 1) * 128);
            spin_flags(mbox + (lane & 31), (lane & 31) == 0 || ((lane & 31) >= 8 && (lane & 31) < 24) ? call : 0u, call, tmo);
        }
    } else {
        const int j = wid - 4;
#pragma unroll
        for (int c = 0; c < 8; ++c) BLDS(rW, (j * 8 + c) * 1024, 16, gvoff, gsoff + j * WPAIR + c * 128, 2);
        asm volatile("s_load_dword %0, %1, 0x0\n\ts_waitcnt lgkmcnt(0)" : "=&s"(callv) : "s"(ctr) : "memory");
        call = callv + 1u;
        if (j == 3) {
            unsigned g = (unsigned)__builtin_amdgcn_s_getreg(20 | (3 << 11)) & 7u;
            unsigned* cl = (unsigned*)(myws + WS_CLAIM + xi * 128);
            for (int i = 0; i < 8; ++i) {
                unsigned old = 0;
                if (lane == 0) old = __hip_atomic_fetch_max(cl + g, call, __ATOMIC_RELAXED, __HIP_MEMORY_SCOPE_AGENT);
                if (__builtin_amdgcn_readfirstlane(old) < call) break;
                g = (g + 1u) & 7u;
            }
            if (lane == 0) *(volatile LAS unsigned*)(arr + 2) = g + 1u;
        }
        const int rd = j * 8192 + (lane >> 5) * 1024 + (lane & 31) * 4;
        const int dv = ((lane >> 4) << 11) + ((lane & 15) << 4);
#pragma unroll
        for (int c = 0; c < 4; ++c) {
            if (c == 0) WAIT_V(6); else if (c == 1) WAIT_V(5); else if (c == 2) WAIT_V(4); else WAIT_V(3);
            const LAS float* c0 = (const LAS float*)(lds + rd + c * 2048);
            const LAS float* c1 = (const LAS float*)(lds + (rd ^ 64) + 512 + c * 2048);
            const h8 hv = cvt8((f32x4){c0[0], c0[32], c0[64], c0[96]}, (f32x4){c1[0], c1[32], c1[64], c1[96]});
            __builtin_amdgcn_raw_buffer_store_b128(__builtin_bit_cast(u32x4, hv), rWS, dv, wsoff + j * 2 * WKT_BYTES + c * 8192, 16);
        }
        STAMP(1);
        if (j < 2) {
            WAIT_V(0);
            STAMP(2);
            unsigned old = 0;
            if (lane == 0) old = __hip_atomic_fetch_add(arr + 1, 1u, __ATOMIC_RELAXED, __HIP_MEMORY_SCOPE_WORKGROUP);
            if (__builtin_amdgcn_readfirstlane(old) == 1u && lane < 16) {
                char* f = myws + WS_MBOX + (nt * 16 + lane) * 128 + (8 + kg) * 4;
                asm volatile("global_store_dword %0, %1, off sc1" :: "v"(f), "v"(call) : "memory");
            }
        }
    }
    STAMP(6);
    WAIT_L(0);
    BAR;
    STAMP(5);
    const int xg = (__builtin_amdgcn_readfirstlane(*(const volatile LAS unsigned*)(arr + 2)) - 1) & 7;
    const int src = (p.srcmap >> (xg * 4)) & (NRANKS - 1);
    const int mt = src * 2 + (xi & 1);
    const auto rX = RSRC(p.ws[src] + WS_X16 + (call & 1u) * X16_BYTES, X16_BYTES);
    const int wr = wid >> 2, wc = wid & 3;
    const int tv = tid * 16;
    const int xs0 = (mt & 1) * 2 * UNIT, ws0 = WS_W16 + nt * 2 * UNIT;
    const int ldsw = wid * 1024;
    const int aoff = wr * 8192 + lane * 16, boff = wc * 4096 + lane * 16;
    const int lj = ((xg >> 1) * 2 + (xi & 1)) * 2 + (xg & 1);
#define JGS(ct, j) ((j) * 8 * WROW + (ct) * 1024 + wr * WPAIR + wc * 256)
#define JWS(ct, j) (WS_W16 + ((j) >> 3) * WKT_BYTES + (ct) * 2 * UNIT + ((((j) >> 2) & 1) << 10) + (((j) & 3) << 8) + wr * 2 * WKT_BYTES + wc * 8192)
#define JFL(ct, j) (WS_WFLAG + ((ct) * NQ * 16 + (j)) * 4)
    int qgs = JGS(nt, lj), qws = JWS(nt, lj), qfl = JFL(nt, lj);
    int nfl = 1;
    const int qdv = ((lane >> 4) << 11) + ((lane & 15) << 4);
    const int qrd = SCR + wid * 2048 + (lane >> 5) * 1024 + (lane & 31) * 4;
    const int pl = (lane >= 24) ? (lane & 15) : lane;
    const int pthr = (pl < 16) ? 0 : 2 * (pl - 16) - 1;
    const int pinc = (pl < 16) ? 64 : 0;
    int poff = (pl < 16) ? WS_WFLAG + (nt * NQ + 1) * 64 + pl * 4 : WS_MBOX + (xi * 8 + xg) * 128 + (pl - 16) * 4;
#define SA(b, h) (((b) * 2 + (h)) * 16384)
#define SB(b, h) ((4 + (b) * 2 + (h)) * 16384)
#define STX(b, h, kt) do { BLDS(rX, SA(b, h) + ldsw, 16, tv, xs0 + (kt) * XKT_BYTES + (h) * UNIT, 0); \
                           BLDS(rX, SA(b, h) + ldsw + 8192, 16, tv, xs0 + (kt) * XKT_BYTES + (h) * UNIT + 8192, 0); } while (0)
#define STW(b, h, kt) do { BLDS(rWS, SB(b, h) + ldsw, 16, tv, ws0 + (kt) * WKT_BYTES + (h) * UNIT, 0); \
                           BLDS(rWS, SB(b, h) + ldsw + 8192, 16, tv, ws0 + (kt) * WKT_BYTES + (h) * UNIT + 8192, 0); } while (0)
#define LDA(dst, b, h) do { _Pragma("unroll") for (int m = 0; m < 4; ++m) _Pragma("unroll") for (int k = 0; k < 2; ++k) \
        dst[m][k] = *(const LAS h8*)(lds + SA(b, h) + aoff + m * 2048 + k * 1024); } while (0)
#define LDB(dst, b, h) do { _Pragma("unroll") for (int n = 0; n < 2; ++n) _Pragma("unroll") for (int k = 0; k < 2; ++k) \
        dst[n][k] = *(const LAS h8*)(lds + SB(b, h) + boff + n * 2048 + k * 1024); } while (0)
#define MMH(ai, h) asm volatile( \
            "v_mfma_f32_16x16x32_f16 %0, %8, %16, %0\n\t" \
            "v_mfma_f32_16x16x32_f16 %0, %9, %17, %0\n\t" \
            "v_mfma_f32_16x16x32_f16 %1, %11, %17, %1\n\t" \
            "v_mfma_f32_16x16x32_f16 %1, %10, %16, %1\n\t" \
            "v_mfma_f32_16x16x32_f16 %2, %12, %16, %2\n\t" \
            "v_mfma_f32_16x16x32_f16 %2, %13, %17, %2\n\t" \
            "v_mfma_f32_16x16x32_f16 %3, %15, %17, %3\n\t" \
            "v_mfma_f32_16x16x32_f16 %3, %14, %16, %3\n\t" \
            "v_mfma_f32_16x16x32_f16 %7, %14, %18, %7\n\t" \
            "v_mfma_f32_16x16x32_f16 %7, %15, %19, %7\n\t" \
            "v_mfma_f32_16x16x32_f16 %6, %13, %19, %6\n\t" \
            "v_mfma_f32_16x16x32_f16 %6, %12, %18, %6\n\t" \
            "v_mfma_f32_16x16x32_f16 %5, %10, %18, %5\n\t" \
            "v_mfma_f32_16x16x32_f16 %5, %11, %19, %5\n\t" \
            "v_mfma_f32_16x16x32_f16 %4, %9, %19, %4\n\t" \
            "v_mfma_f32_16x16x32_f16 %4, %8, %18, %4\n\t" \
            : "+v"(acc[ai][0][2*(h)][0]), "+v"(acc[ai][0][2*(h)][1]), "+v"(acc[ai][1][2*(h)][0]), "+v"(acc[ai][1][2*(h)][1]), \
              "+v"(acc[ai][0][2*(h)+1][0]), "+v"(acc[ai][0][2*(h)+1][1]), "+v"(acc[ai][1][2*(h)+1][0]), "+v"(acc[ai][1][2*(h)+1][1]) \
            : "v"(B0[0][0]), "v"(B0[0][1]), "v"(B0[1][0]), "v"(B0[1][1]), "v"(B1[0][0]), "v"(B1[0][1]), "v"(B1[1][0]), "v"(B1[1][1]), \
              "v"(At[2*(h)][0]), "v"(At[2*(h)][1]), "v"(At[2*(h)+1][0]), "v"(At[2*(h)+1][1]))
#define COMPUTE(ai) do { BAR; SCHED; __builtin_amdgcn_s_setprio(1); MMH(ai, 0); MMH(ai, 1); __builtin_amdgcn_s_setprio(0); BAR; SCHED; } while (0)
#define SEGA(b, tn) do { STX((b) ^ 1, 1, tn); LDB(B0, b, 0); LDA(At, b, 0); LDB(B1, b, 1); } while (0)
#define SEGB(b, tn) do { STX(b, 0, tn); LDA(At, b, 1); STW(b, 0, tn); STW(b, 1, tn); } while (0)
#define CHECKQ(q) do { asm volatile("" : "+v"(pv)); \
        if (__builtin_expect(!__all((int)(pv >= call) | (int)(pthr > (q))), 0)) \
            if (wid == 0) spin_flags((const unsigned*)(myws + poff), pthr > (q) ? 0u : call, call, tmo); } while (0)

    f32x4 acc[2][2][4][2];
#pragma unroll
    for (int a = 0; a < 2; ++a)
#pragma unroll
        for (int b = 0; b < 2; ++b)
#pragma unroll
            for (int m = 0; m < 4; ++m)
#pragma unroll
                for (int n = 0; n < 2; ++n) acc[a][b][m][n] = (f32x4){0.f, 0.f, 0.f, 0.f};
    h8 At[4][2], B0[2][2], B1[2][2];

    STX(0, 0, 0); STX(0, 1, 0); STW(0, 0, 0); STW(0, 1, 0);
    STX(1, 0, 1); STW(1, 0, 1); STW(1, 1, 1);
    WAIT_V(6); BAR;
    if (wid == 2 && lane == 0) asm volatile("global_store_dword %0, %1, off sc1" :: "v"(wflag_row + 16 + kg), "v"(call) : "memory");
    if (wr == 1) BAR;
    STAMP(8); CSTAMP(50);
#define GJOB(gs) do { BLDS(rW, SCR + 2 * ldsw, 16, gvoff, gs, 2); BLDS(rW, SCR + 2 * ldsw + 1024, 16, gvoff, (gs) + 128, 2); } while (0)
#define CREAD const LAS float* c0 = (const LAS float*)(lds + qrd); const LAS float* c1 = (const LAS float*)(lds + (qrd ^ 64) + 512); \
        const float f0 = c0[0], f1 = c0[32], f2 = c0[64], f3 = c0[96], f4 = c1[0], f5 = c1[32], f6 = c1[64], f7 = c1[96]
#define CSTORE(ws) do { SCHED; const h8 hv = cvt8((f32x4){f0, f1, f2, f3}, (f32x4){f4, f5, f6, f7}); \
        __builtin_amdgcn_raw_buffer_store_b128(__builtin_bit_cast(u32x4, hv), rWS, qdv, ws, 16); } while (0)
#define TRIP(J1, J2) do { \
        if (J1) GJOB(qgs + (q + 2) * (2 * WPAIR)); \
        SEGA(0, t + 1); WAIT_V(8 + 2 * J1 + J2); WAIT_L(0); COMPUTE(0); \
        SEGB(0, t + 2); \
        if (wid == 0) BLDS(rWS, POLL, 4, poff, 0, 17);            \
        WAIT_VX(8 + 2 * J1, wid, 0); WAIT_L(0); COMPUTE(1); \
        SEGA(1, t + 2); WAIT_VX(8, wid, 0); WAIT_L(0); COMPUTE(0); \
        if (J1) { CREAD; SEGB(1, t + 3); CSTORE(qws + (q + 2) * (4 * WKT_BYTES)); } else SEGB(1, t + 3); \
        WAIT_V(8 + J1); WAIT_L(0); COMPUTE(1); \
        if (J2) GJOB(qgs + 8 * WROW + (q + 3) * (2 * WPAIR)); \
        unsigned pv = *(const volatile LAS unsigned*)(lds + POLL + lane * 4); \
        SEGA(0, t + 3); WAIT_V(8 + J1 + 2 * J2); WAIT_L(0); \
        CHECKQ(q); \
        poff += pinc; \
        COMPUTE(0); \
        SEGB(0, t + 4); WAIT_V(8 + 2 * J2); WAIT_L(0); COMPUTE(1); \
        if (J1) { if (wid == 1) { if (lane < nfl) __builtin_amdgcn_raw_buffer_store_b32(call, rWS, lane * 4, qfl + (q + 2) * 64, 16); } } \
        SEGA(1, t + 4); if (J1) WAIT_VX(8, wid, 1); else WAIT_V(8); \
        WAIT_L(0); COMPUTE(0); \
        if (J2) { CREAD; SEGB(1, t + 5); CSTORE(qws + 256 + (q + 3) * (4 * WKT_BYTES)); } else SEGB(1, t + 5); \
        if (J1) WAIT_VX(8 + J2, wid, 1); else WAIT_V(8); \
        WAIT_L(0); COMPUTE(1); } while (0)
    {
        const bool two = !(xg & 1) && nt != 0, helper = src == p.rank && xi >= 16;
        const int n1 = two ? NQ - 3 : 0;
        const int n2 = (two || helper) ? NQ - 2 : 1;
        int q = 0;
#pragma nounroll
        for (; q < n1; ++q) { const int t = q * 4; TRIP(1, 1); nfl = 2; }
#pragma nounroll
        for (; q < n2; ++q) { const int t = q * 4; TRIP(1, 0); qgs = JGS(0, xi - 16); qws = JWS(0, xi - 16); qfl = JFL(0, xi - 16); }
#pragma nounroll
        for (; q < NQ - 2; ++q) { const int t = q * 4; TRIP(0, 0); }
    }
#pragma nounroll
    for (int it = NIT - 4; it < NIT - 1; ++it) {
        const int t = it * 2;
        const int sel = __builtin_amdgcn_readfirstlane((it == NIT - 4) ? wid : 1);
        unsigned pv = *(const volatile LAS unsigned*)(lds + POLL + lane * 4);
        SEGA(0, t + 1); WAIT_V(8); WAIT_L(0);
        if (it == NIT - 3) CHECKQ(NQ - 2);
        COMPUTE(0);
        SEGB(0, t + 2);
        if (sel == 0) BLDS(rWS, POLL, 4, poff, 0, 17);
        WAIT_VX(8, sel, 0); WAIT_L(0); COMPUTE(1);
        SEGA(1, t + 2); WAIT_VX(8, sel, 0); WAIT_L(0); COMPUTE(0);
        SEGB(1, t + 3); WAIT_V(8); WAIT_L(0); COMPUTE(1);
    }
    STAMP(39); CSTAMP(51);
    int ln;
    asm volatile("v_mbcnt_lo_u32_b32 %0, -1, 0\n\tv_mbcnt_hi_u32_b32 %0, -1, %0" : "=v"(ln));
    const int fr = ln & 15, fq = ln >> 4;
    float* obase = p.out + (size_t)(mt * 256 + wr * 64 + fr) * NLOC + nt * 256 + wc * 32 + fq * 4;
    const float qnan = __builtin_nanf("");
    unsigned tmov;
#define OUTQ(ai, bj) do { _Pragma("unroll") for (int m = 0; m < 4; ++m) _Pragma("unroll") for (int n = 0; n < 2; ++n) { \
        f32x4 v = acc[ai][bj][m][n]; \
        v[0] = fmaxf(v[0], 0.f); v[1] = fmaxf(v[1], 0.f); v[2] = fmaxf(v[2], 0.f); v[3] = fmaxf(v[3], 0.f); \
        if (bad) v = (f32x4){qnan, qnan, qnan, qnan}; \
        float* dst = obase + (size_t)((ai) * 128 + m * 16) * NLOC + (bj) * 128 + n * 16; \
        asm volatile("global_store_dwordx4 %0, %1, off sc1\n\ts_nop 1" :: "v"(dst), "v"(v) : "memory"); } } while (0)
    {
        SEGA(0, NKT - 1); WAIT_V(8); WAIT_L(0); COMPUTE(0);
        LDA(At, 0, 1); WAIT_V(2); WAIT_L(0); COMPUTE(1);
        LDB(B0, 1, 0); LDA(At, 1, 0); LDB(B1, 1, 1);
        asm volatile("global_load_dword %0, %1, off sc1\n\ts_waitcnt vmcnt(0)" : "=&v"(tmov) : "v"(tmo) : "memory");
        WAIT_L(0);
        COMPUTE(0);
        const bool bad = __builtin_amdgcn_readfirstlane(tmov) == call;
        asm volatile("s_nop 15" ::: "memory"); SCHED;
        LDA(At, 1, 1);
        OUTQ(0, 0); OUTQ(0, 1);
        WAIT_L(0);
        BAR; SCHED; __builtin_amdgcn_s_setprio(1); MMH(1, 0); MMH(1, 1); __builtin_amdgcn_s_setprio(0);
        asm volatile("s_nop 15\n\ts_nop 15" ::: "memory"); SCHED;
        STAMP(40);
        OUTQ(1, 0); OUTQ(1, 1);
    }
    if (ln == 0) asm volatile("global_store_dword %0, %1, off sc1" :: "v"(ctr), "v"(call) : "memory");
    STAMP(41);
    WAIT_V(0);
    STAMP(42);
    if (wr == 0) BAR;
}

}

extern "C" void kernel_launch(int rank, int nranks, ncclComm_t comm,
                              hipStream_t stream,
                              void* const* d_in, const int* in_sizes, int n_in,
                              void* d_out, int out_size,
                              const int* in_dtypes, int out_dtype,
                              void* const* d_out_all,
                              void* const* d_ws_all, size_t ws_bytes) {
    if (nranks != NRANKS || (unsigned)rank >= (unsigned)NRANKS || ws_bytes < WS_NEEDED + (8u << 20) || n_in < 2 ||
        in_sizes[0] != MLOC * K_ || in_sizes[1] != K_ * NFULL || out_size != M_ * NLOC) {
        fprintf(stderr, "kernel_launch: unexpected problem shape\n");
        return;
    }
    HIP_CHECK(hipFuncSetAttribute((const void*)fused_kernel, hipFuncAttributeMaxDynamicSharedMemorySize, LDS_BYTES));
    Params p;
    p.x = (const float*)d_in[0];
    p.w = (const float*)d_in[1];
    p.out = (float*)d_out;
    const size_t wsoff = (ws_bytes - WS_NEEDED - (4u << 20)) & ~(size_t)((2u << 20) - 1);
    for (int q = 0; q < NRANKS; ++q) p.ws[q] = (char*)d_ws_all[q] + wsoff;
    p.rank = rank;
    {
        const unsigned half_mask = 0x4Du;
        p.srcmap = p.grpmap = 0;
        for (int r = 0; r < NRANKS; ++r) {
            const int mine = (half_mask >> r) & 1;
            int g = 0;
            for (int pass = 0; pass < 2; ++pass)
                for (int d = 0; d < NRANKS; ++d) {
                    const int q = (r + d) & 7;
                    if (((int)((half_mask >> q) & 1) == mine) != (pass == 0)) continue;
                    const int gg = (g == 0) ? SELFG : (g == SELFG ? 0 : g);
                    if (r == rank) p.srcmap |= (unsigned)q << (4 * gg);
                    if (q == rank) p.grpmap |= (unsigned)gg << (4 * r);
                    ++g;
                }
        }
    }
    hipLaunchKernelGGL(fused_kernel, dim3(256), dim3(512), LDS_BYTES, stream, p);
    HIP_CHECK(hipGetLastError());
}
```

```cpp
#include <hip/hip_runtime.h>
#include <rccl/rccl.h>
#include <cstdio>

#define HIP_CHECK(expr) do { hipError_t _e = (expr); if (_e != hipSuccess) { \
    fprintf(stderr, "HIP error %s at %s:%d\n", hipGetErrorString(_e), __FILE__, __LINE__); \
    return; } } while (0)

#define XD 110
#define WD0 4
#define WD1 110
#define PD 32
namespace {

constexpr int SELFG = 1;
static_assert(SELFG & 1, "the helpers of column tile 0 must not run two jobs of their own");

typedef _Float16 h8 __attribute__((ext_vector_type(8)));
typedef _Float16 h4 __attribute__((ext_vector_type(4)));
typedef float f32x4 __attribute__((ext_vector_type(4)));
typedef unsigned u32x4 __attribute__((ext_vector_type(4)));
#define LAS __attribute__((address_space(3)))

constexpr int NRANKS = 8;
constexpr int M_ = 4096, K_ = 4096, NLOC = 4096, NFULL = 32768, MLOC = 512;
constexpr int NKT = K_ / 64;
constexpr int NIT = NKT / 2;
constexpr int NQ = NKT / 4;
constexpr int UNIT = 16384;
constexpr int WKT_BYTES = 32 * UNIT;
constexpr int XKT_BYTES = 4 * UNIT;
constexpr int X16_BYTES = NKT * XKT_BYTES;
constexpr int W16_BYTES = NKT * WKT_BYTES;
constexpr int WROW = NFULL * 4;
constexpr int WPAIR = 128 * WROW;
constexpr int WS_TMO = 0;
constexpr int WS_CTR = 4096;
constexpr int WS_XCNT = WS_CTR + 256 * 128;
constexpr int WS_MBOX = WS_XCNT + 24 * 128;
constexpr int WS_CLAIM = WS_MBOX + 256 * 128;
constexpr int WS_WFLAG = WS_CLAIM + 32 * 128;
constexpr int WS_X16 = 1 << 20;
constexpr int WS_W16 = WS_X16 + 2 * X16_BYTES;
constexpr size_t WS_NEEDED = (size_t)WS_W16 + W16_BYTES;
static_assert(WS_WFLAG + 16 * NQ * 16 * 4 <= WS_X16, "control area");
constexpr unsigned long long DEADLINE_TICKS = 100000000ull;
constexpr int SCR = 8 * 16384;
constexpr int POLL = SCR + 16384;
constexpr int ARR = POLL + 256;
constexpr int LDS_BYTES = ARR + 16;

struct Params {
    const float* x;
    const float* w;
    float* out;
    char* ws[NRANKS];
    int rank;
    unsigned srcmap;
    unsigned grpmap;
};

__device__ __forceinline__ h8 cvt8(f32x4 a, f32x4 b) {
    h8 r;
    r[0] = (_Float16)a[0]; r[1] = (_Float16)a[1]; r[2] = (_Float16)a[2]; r[3] = (_Float16)a[3];
    r[4] = (_Float16)b[0]; r[5] = (_Float16)b[1]; r[6] = (_Float16)b[2]; r[7] = (_Float16)b[3];
    return r;
}

#define WAIT_V(n) asm volatile("s_waitcnt vmcnt(%0)" :: "n"(n) : "memory")
#define WAIT_L(n) do { asm volatile("" ::: "memory"); __builtin_amdgcn_s_waitcnt(0xC07F); asm volatile("" ::: "memory"); } while (0)
#define WAIT_VX(n, sel, w) asm volatile("s_waitcnt vmcnt(%0)\n\ts_cmp_eq_u32 %2, %3\n\ts_cbranch_scc1 1f\n\ts_waitcnt vmcnt(%1)\n1:" \
        :: "n"((n) + 1), "n"(n), "s"(sel), "n"(w) : "memory", "scc")
#define BAR __builtin_amdgcn_s_barrier()
#define SCHED __builtin_amdgcn_sched_barrier(0)
#define RSRC(ptr, bytes) __builtin_amdgcn_make_buffer_rsrc((void*)(ptr), 0, (bytes), 0x00020000)
#define BLDS(rsrc, ldsoff, size, voff, soff, aux) \
        __builtin_amdgcn_raw_ptr_buffer_load_lds(rsrc, (LAS void*)(lds + (ldsoff)), size, voff, soff, 0, aux)

__device__ __forceinline__ void spin_flags(const unsigned* lane_ptr, unsigned need, unsigned call, unsigned* tmo) {
    const unsigned long long t0 = __builtin_amdgcn_s_memrealtime();
    for (unsigned n = 1;; ++n) {
        unsigned v;
        asm volatile("global_load_dword %0, %1, off sc0 sc1\n\ts_waitcnt vmcnt(0)" : "=&v"(v) : "v"(lane_ptr) : "memory");
        if (__all(v >= need)) return;
        if ((n & 15u) == 0u) {
            if (__builtin_amdgcn_readfirstlane(__hip_atomic_load(tmo, __ATOMIC_RELAXED, __HIP_MEMORY_SCOPE_AGENT)) == call
                || __builtin_amdgcn_s_memrealtime() - t0 > DEADLINE_TICKS) {
                __hip_atomic_store(tmo, call, __ATOMIC_RELAXED, __HIP_MEMORY_SCOPE_AGENT);
                return;
            }
        }
        __builtin_amdgcn_s_sleep(1);
    }
}

__global__ __launch_bounds__(512, 2) void fused_kernel(Params p) {
    extern __shared__ __attribute__((aligned(16))) unsigned char shm[];
    LAS unsigned char* lds = (LAS unsigned char*)shm;
    const int tid = threadIdx.x;
    const int wid = __builtin_amdgcn_readfirstlane(tid >> 6), lane = tid & 63;
    char* myws = p.ws[p.rank];
    const int bid = blockIdx.x;
    const int xi = bid >> 3, nt = xi >> 1;
    const int kg = (bid & 7) * 2 + (xi & 1);
    unsigned* tmo = (unsigned*)(myws + WS_TMO);
#define STAMP(idx) do {} while (0)
#define CSTAMP(idx) do {} while (0)
    STAMP(0);
    const auto rWS = RSRC(myws, (int)WS_NEEDED);
    const auto rW = RSRC(p.w + p.rank * NLOC, (K_ - 1) * WROW + NLOC * 4);
    unsigned* ctr = (unsigned*)(myws + WS_CTR + bid * 128 + wid * 4);
    unsigned callv, call;
    const int gvoff = (lane >> 3) * WROW + (((lane & 7) ^ ((lane >> 5) << 2)) << 4);
    const int gsoff = kg * 8 * WROW + nt * 1024;
    const int wsoff = WS_W16 + (kg >> 3) * WKT_BYTES + nt * 2 * UNIT + (((kg >> 2) & 1) << 10) + ((kg & 3) << 8);
    unsigned* wflag_row = (unsigned*)(myws + WS_WFLAG) + nt * NQ * 16;
    LAS unsigned* arr = (LAS unsigned*)(lds + ARR);
    if (tid < 3) arr[tid] = 0u;
    WAIT_L(0); BAR;
    if (wid == 2) {
        const int u = bid >> 4, blk = bid & 15;
        const float* xsrc = p.x + (size_t)((u & 3) * 128 + (blk >> 1) * 16 + (lane & 15)) * K_ + (u >> 2) * 64 + (blk & 1) * 32 + (lane >> 4) * 8;
        f32x4 a0, a1;
        asm volatile("global_load_dwordx4 %0, %3, off nt\n\tglobal_load_dwordx4 %1, %3, off offset:16 nt\n\t"
                     "s_load_dword %2, %4, 0x0\n\ts_waitcnt vmcnt(0) lgkmcnt(0)"
                     : "=&v"(a0), "=&v"(a1), "=&s"(callv) : "v"(xsrc), "s"(ctr) : "memory");
        call = callv + 1u;
        STAMP(1);
        char* xd = myws + WS_X16 + (call & 1u) * X16_BYTES + u * UNIT + blk * 1024 + lane * 16;
        const h8 v0 = cvt8(a0, a1);
        asm volatile("global_store_dwordx4 %0, %1, off sc0 sc1\n\ts_waitcnt vmcnt(0)" :: "v"(xd), "v"(v0) : "memory");
        STAMP(2);
        unsigned cnt = 0;
        if (lane == 0) cnt = __hip_atomic_fetch_add((unsigned*)(myws + WS_XCNT + (8 + u) * 128), 1u, __ATOMIC_RELAXED, __HIP_MEMORY_SCOPE_AGENT);
        if (__builtin_amdgcn_readfirstlane(cnt) + 1u == 16u * call) {
#pragma unroll
            for (int i = 0; i < 2; ++i) {
                char* w0 = p.ws[4 * i] + (((p.grpmap >> (16 * i)) & 7u) << 7);
                char* w1 = p.ws[4 * i + 1] + (((p.grpmap >> (16 * i + 4)) & 7u) << 7);
                char* w2 = p.ws[4 * i + 2] + (((p.grpmap >> (16 * i + 8)) & 7u) << 7);
                char* w3 = p.ws[4 * i + 3] + (((p.grpmap >> (16 * i + 12)) & 7u) << 7);
                char* f = (lane < 32 ? (lane < 16 ? w0 : w1) : (lane < 48 ? w2 : w3)) + WS_MBOX
                          + ((lane & 15) * 2 + ((u >> 1) & 1)) * 1024 + (24 + (u >> 2) * 2 + (u & 1)) * 4;
                asm volatile("global_store_dword %0, %1, off sc0 sc1" :: "v"(f), "v"(call) : "memory");
            }
            STAMP(7);
        }
    } else if (wid == 3) {
        asm volatile("s_load_dword %0, %1, 0x0\n\ts_waitcnt lgkmcnt(0)" : "=&s"(callv) : "s"(ctr) : "memory");
        call = callv + 1u;
        unsigned g = (unsigned)__builtin_amdgcn_s_getreg(20 | (3 << 11)) & 7u;
        unsigned* cl = (unsigned*)(myws + WS_CLAIM + xi * 128);
        for (int i = 0; i < 8; ++i) {
            unsigned old = 0;
            if (lane == 0) old = __hip_atomic_fetch_max(cl + g, call, __ATOMIC_RELAXED, __HIP_MEMORY_SCOPE_AGENT);
            if (__builtin_amdgcn_readfirstlane(old) < call) break;
            g = (g + 1u) & 7u;
        }
        if (lane == 0) *(volatile LAS unsigned*)(arr + 2) = g + 1u;
        const unsigned* mbox = (const unsigned*)(myws + WS_MBOX + (xi * 8 + g) * 128);
        __builtin_amdgcn_s_sleep(PD);
        spin_flags(mbox + (lane & 31), (lane & 31) >= 8 ? call : 0u, call, tmo);
    } else if (wid < 2) {
        __builtin_amdgcn_s_sleep(XD);
        const int u = (bid < 240) ? 16 + bid : 255;
        const float* xs0 = p.x + (size_t)((u & 3) * 128 + wid * 64 + (lane & 15)) * K_ + (u >> 2) * 64 + (lane >> 4) * 8;
        const float* xs1 = xs0 + 16 * K_; const float* xs2 = xs0 + 32 * K_; const float* xs3 = xs0 + 48 * K_;
        f32x4 a[16];
#define XL4(i, ptr) asm volatile("global_load_dwordx4 %0, %4, off nt\n\tglobal_load_dwordx4 %1, %4, off offset:16 nt\n\t" \
                     "global_load_dwordx4 %2, %4, off offset:128 nt\n\tglobal_load_dwordx4 %3, %4, off offset:144 nt" \
                     : "=&v"(a[i]), "=&v"(a[i + 1]), "=&v"(a[i + 2]), "=&v"(a[i + 3]) : "v"(ptr) : "memory")
        XL4(0, xs0); XL4(4, xs1); XL4(8, xs2); XL4(12, xs3);
        asm volatile("s_load_dword %16, %17, 0x0\n\ts_waitcnt vmcnt(0) lgkmcnt(0)"
                     : "+v"(a[0]), "+v"(a[1]), "+v"(a[2]), "+v"(a[3]), "+v"(a[4]), "+v"(a[5]), "+v"(a[6]), "+v"(a[7]),
                       "+v"(a[8]), "+v"(a[9]), "+v"(a[10]), "+v"(a[11]), "+v"(a[12]), "+v"(a[13]), "+v"(a[14]), "+v"(a[15]), "=&s"(callv)
                     : "s"(ctr) : "memory");
        call = callv + 1u;
        STAMP(1);
        if (bid < 240) {
            char* xd = myws + WS_X16 + (call & 1u) * X16_BYTES + u * UNIT + wid * 8192 + lane * 16;
#pragma unroll
            for (int i = 0; i < 8; ++i) {
                const h8 v = cvt8(a[2 * i], a[2 * i + 1]);
                asm volatile("global_store_dwordx4 %0, %1, off sc0 sc1\n\ts_nop 1" :: "v"(xd + i * 1024), "v"(v) : "memory");
            }
            WAIT_V(0);
            STAMP(2);
            unsigned old = 0;
            if (lane == 0) old = __hip_atomic_fetch_add(arr, 1u, __ATOMIC_RELAXED, __HIP_MEMORY_SCOPE_WORKGROUP);
            if (__builtin_amdgcn_readfirstlane(old) == 1u) {
                const int shard = u >> 5;
                unsigned cnt = 0;
                if (lane == 0) cnt = __hip_atomic_fetch_add((unsigned*)(myws + WS_XCNT + shard * 128), 1u, __ATOMIC_RELAXED, __HIP_MEMORY_SCOPE_AGENT);
                if (__builtin_amdgcn_readfirstlane(cnt) + 1u == (shard ? 32u : 16u) * call) {
#pragma unroll
                    for (int i = 0; i < 4; ++i) {
                        const int q0 = 2 * i, q1 = 2 * i + 1;
                        char* w0 = p.ws[q0] + (((p.grpmap >> (4 * q0)) & 7u) << 7);
                        char* w1 = p.ws[q1] + (((p.grpmap >> (4 * q1)) & 7u) << 7);
                        char* f = (lane < 32 ? w0 : w1) + WS_MBOX + (lane & 31) * 1024 + shard * 4;
                        asm volatile("global_store_dword %0, %1, off sc0 sc1" :: "v"(f), "v"(call) : "memory");
                    }
                }
            }
        }
    } else {
        const int j = wid - 4;
        if (j < 2) __builtin_amdgcn_s_sleep(WD0); else __builtin_amdgcn_s_sleep(WD1);
#pragma unroll
        for (int c = 0; c < 8; ++c) BLDS(rW, (j * 8 + c) * 1024, 16, gvoff, gsoff + j * WPAIR + c * 128, 2);
        asm volatile("s_load_dword %0, %1, 0x0\n\ts_waitcnt lgkmcnt(0)" : "=&s"(callv) : "s"(ctr) : "memory");
        call = callv + 1u;
        const int rd = j * 8192 + (lane >> 5) * 1024 + (lane & 31) * 4;
        const int dv = ((lane >> 4) << 11) + ((lane & 15) << 4);
#pragma unroll
        for (int c = 0; c < 4; ++c) {
            if (c == 0) WAIT_V(6); else if (c == 1) WAIT_V(5); else if (c == 2) WAIT_V(4); else WAIT_V(3);
            const LAS float* c0 = (const LAS float*)(lds + rd + c * 2048);
            const LAS float* c1 = (const LAS float*)(lds + (rd ^ 64) + 512 + c * 2048);
            const h8 hv = cvt8((f32x4){c0[0], c0[32], c0[64], c0[96]}, (f32x4){c1[0], c1[32], c1[64], c1[96]});
            __builtin_amdgcn_raw_buffer_store_b128(__builtin_bit_cast(u32x4, hv), rWS, dv, wsoff + j * 2 * WKT_BYTES + c * 8192, 16);
        }
        STAMP(1);
        if (j < 2) {
            WAIT_V(0);
            STAMP(2);
            unsigned old = 0;
            if (lane == 0) old = __hip_atomic_fetch_add(arr + 1, 1u, __ATOMIC_RELAXED, __HIP_MEMORY_SCOPE_WORKGROUP);
            if (__builtin_amdgcn_readfirstlane(old) == 1u && lane < 16) {
                char* f = myws + WS_MBOX + (nt * 16 + lane) * 128 + (8 + kg) * 4;
                asm volatile("global_store_dword %0, %1, off sc1" :: "v"(f), "v"(call) : "memory");
            }
        }
    }
    STAMP(6);
    WAIT_L(0);
    BAR;
    STAMP(5);
    const int xg = (__builtin_amdgcn_readfirstlane(*(const volatile LAS unsigned*)(arr + 2)) - 1) & 7;
    const int src = (p.srcmap >> (xg * 4)) & (NRANKS - 1);
    const int mt = src * 2 + (xi & 1);
    const auto rX = RSRC(p.ws[src] + WS_X16 + (call & 1u) * X16_BYTES, X16_BYTES);
    const int wr = wid >> 2, wc = wid & 3;
    const int tv = tid * 16;
    const int xs0 = (mt & 1) * 2 * UNIT, ws0 = WS_W16 + nt * 2 * UNIT;
    const int ldsw = wid * 1024;
    const int aoff = wr * 8192 + lane * 16, boff = wc * 4096 + lane * 16;
    const int lj = ((xg >> 1) * 2 + (xi & 1)) * 2 + (xg & 1);
#define JGS(ct, j) ((j) * 8 * WROW + (ct) * 1024 + wr * WPAIR + wc * 256)
#define JWS(ct, j) (WS_W16 + ((j) >> 3) * WKT_BYTES + (ct) * 2 * UNIT + ((((j) >> 2) & 1) << 10) + (((j) & 3) << 8) + wr * 2 * WKT_BYTES + wc * 8192)
#define JFL(ct, j) (WS_WFLAG + ((ct) * NQ * 16 + (j)) * 4)
    int qgs = JGS(nt, lj), qws = JWS(nt, lj), qfl = JFL(nt, lj);
    int nfl = 1;
    const int qdv = ((lane >> 4) << 11) + ((lane & 15) << 4);
    const int qrd = SCR + wid * 2048 + (lane >> 5) * 1024 + (lane & 31) * 4;
    const int pl = (lane >= 24) ? (lane & 15) : lane;
    const int pthr = (pl < 16) ? 0 : 2 * (pl - 16) - 1;
    const int pinc = (pl < 16) ? 64 : 0;
    int poff = (pl < 16) ? WS_WFLAG + (nt * NQ + 1) * 64 + pl * 4 : WS_MBOX + (xi * 8 + xg) * 128 + (pl - 16) * 4;
#define SA(b, h) (((b) * 2 + (h)) * 16384)
#define SB(b, h) ((4 + (b) * 2 + (h)) * 16384)
#define STX(b, h, kt) do { BLDS(rX, SA(b, h) + ldsw, 16, tv, xs0 + (kt) * XKT_BYTES + (h) * UNIT, 0); \
                           BLDS(rX, SA(b, h) + ldsw + 8192, 16, tv, xs0 + (kt) * XKT_BYTES + (h) * UNIT + 8192, 0); } while (0)
#define STW(b, h, kt) do { BLDS(rWS, SB(b, h) + ldsw, 16, tv, ws0 + (kt) * WKT_BYTES + (h) * UNIT, 0); \
                           BLDS(rWS, SB(b, h) + ldsw + 8192, 16, tv, ws0 + (kt) * WKT_BYTES + (h) * UNIT + 8192, 0); } while (0)
#define LDA(dst, b, h) do { _Pragma("unroll") for (int m = 0; m < 4; ++m) _Pragma("unroll") for (int k = 0; k < 2; ++k) \
        dst[m][k] = *(const LAS h8*)(lds + SA(b, h) + aoff + m * 2048 + k * 1024); } while (0)
#define LDB(dst, b, h) do { _Pragma("unroll") for (int n = 0; n < 2; ++n) _Pragma("unroll") for (int k = 0; k < 2; ++k) \
        dst[n][k] = *(const LAS h8*)(lds + SB(b, h) + boff + n * 2048 + k * 1024); } while (0)
#define MMH(ai, h) asm volatile( \
            "v_mfma_f32_16x16x32_f16 %0, %8, %16, %0\n\t" \
            "v_mfma_f32_16x16x32_f16 %0, %9, %17, %0\n\t" \
            "v_mfma_f32_16x16x32_f16 %1, %11, %17, %1\n\t" \
            "v_mfma_f32_16x16x32_f16 %1, %10, %16, %1\n\t" \
            "v_mfma_f32_16x16x32_f16 %2, %12, %16, %2\n\t" \
            "v_mfma_f32_16x16x32_f16 %2, %13, %17, %2\n\t" \
            "v_mfma_f32_16x16x32_f16 %3, %15, %17, %3\n\t" \
            "v_mfma_f32_16x16x32_f16 %3, %14, %16, %3\n\t" \
            "v_mfma_f32_16x16x32_f16 %7, %14, %18, %7\n\t" \
            "v_mfma_f32_16x16x32_f16 %7, %15, %19, %7\n\t" \
            "v_mfma_f32_16x16x32_f16 %6, %13, %19, %6\n\t" \
            "v_mfma_f32_16x16x32_f16 %6, %12, %18, %6\n\t" \
            "v_mfma_f32_16x16x32_f16 %5, %10, %18, %5\n\t" \
            "v_mfma_f32_16x16x32_f16 %5, %11, %19, %5\n\t" \
            "v_mfma_f32_16x16x32_f16 %4, %9, %19, %4\n\t" \
            "v_mfma_f32_16x16x32_f16 %4, %8, %18, %4\n\t" \
            : "+v"(acc[ai][0][2*(h)][0]), "+v"(acc[ai][0][2*(h)][1]), "+v"(acc[ai][1][2*(h)][0]), "+v"(acc[ai][1][2*(h)][1]), \
              "+v"(acc[ai][0][2*(h)+1][0]), "+v"(acc[ai][0][2*(h)+1][1]), "+v"(acc[ai][1][2*(h)+1][0]), "+v"(acc[ai][1][2*(h)+1][1]) \
            : "v"(B0[0][0]), "v"(B0[0][1]), "v"(B0[1][0]), "v"(B0[1][1]), "v"(B1[0][0]), "v"(B1[0][1]), "v"(B1[1][0]), "v"(B1[1][1]), \
              "v"(At[2*(h)][0]), "v"(At[2*(h)][1]), "v"(At[2*(h)+1][0]), "v"(At[2*(h)+1][1]))
#define COMPUTE(ai) do { BAR; SCHED; __builtin_amdgcn_s_setprio(1); MMH(ai, 0); MMH(ai, 1); __builtin_amdgcn_s_setprio(0); BAR; SCHED; } while (0)
#define SEGA(b, tn) do { STX((b) ^ 1, 1, tn); LDB(B0, b, 0); LDA(At, b, 0); LDB(B1, b, 1); } while (0)
#define SEGB(b, tn) do { STX(b, 0, tn); LDA(At, b, 1); STW(b, 0, tn); STW(b, 1, tn); } while (0)
#define CHECKQ(q) do { asm volatile("" : "+v"(pv)); \
        if (__builtin_expect(!__all((int)(pv >= call) | (int)(pthr > (q))), 0)) \
            if (wid == 0) spin_flags((const unsigned*)(myws + poff), pthr > (q) ? 0u : call, call, tmo); } while (0)

    f32x4 acc[2][2][4][2];
#pragma unroll
    for (int a = 0; a < 2; ++a)
#pragma unroll
        for (int b = 0; b < 2; ++b)
#pragma unroll
            for (int m = 0; m < 4; ++m)
#pragma unroll
                for (int n = 0; n < 2; ++n) acc[a][b][m][n] = (f32x4){0.f, 0.f, 0.f, 0.f};
    h8 At[4][2], B0[2][2], B1[2][2];

    STX(0, 0, 0); STX(0, 1, 0); STW(0, 0, 0); STW(0, 1, 0);
    STX(1, 0, 1); STW(1, 0, 1); STW(1, 1, 1);
    WAIT_V(6); BAR;
    if (wid == 2 && lane == 0) asm volatile("global_store_dword %0, %1, off sc1" :: "v"(wflag_row + 16 + kg), "v"(call) : "memory");
    if (wr == 1) BAR;
    STAMP(8); CSTAMP(50);
#define GJOB(gs) do { BLDS(rW, SCR + 2 * ldsw, 16, gvoff, gs, 2); BLDS(rW, SCR + 2 * ldsw + 1024, 16, gvoff, (gs) + 128, 2); } while (0)
#define CREAD const LAS float* c0 = (const LAS float*)(lds + qrd); const LAS float* c1 = (const LAS float*)(lds + (qrd ^ 64) + 512); \
        const float f0 = c0[0], f1 = c0[32], f2 = c0[64], f3 = c0[96], f4 = c1[0], f5 = c1[32], f6 = c1[64], f7 = c1[96]
#define CSTORE(ws) do { SCHED; const h8 hv = cvt8((f32x4){f0, f1, f2, f3}, (f32x4){f4, f5, f6, f7}); \
        __builtin_amdgcn_raw_buffer_store_b128(__builtin_bit_cast(u32x4, hv), rWS, qdv, ws, 16); } while (0)
#define TRIP(J1, J2) do { \
        if (J1) GJOB(qgs + (q + 2) * (2 * WPAIR)); \
        SEGA(0, t + 1); WAIT_V(8 + 2 * J1 + J2); WAIT_L(0); COMPUTE(0); \
        SEGB(0, t + 2); \
        if (wid == 0) BLDS(rWS, POLL, 4, poff, 0, 17);            \
        WAIT_VX(8 + 2 * J1, wid, 0); WAIT_L(0); COMPUTE(1); \
        SEGA(1, t + 2); WAIT_VX(8, wid, 0); WAIT_L(0); COMPUTE(0); \
        if (J1) { CREAD; SEGB(1, t + 3); CSTORE(qws + (q + 2) * (4 * WKT_BYTES)); } else SEGB(1, t + 3); \
        WAIT_V(8 + J1); WAIT_L(0); COMPUTE(1); \
        if (J2) GJOB(qgs + 8 * WROW + (q + 3) * (2 * WPAIR)); \
        unsigned pv = *(const volatile LAS unsigned*)(lds + POLL + lane * 4); \
        SEGA(0, t + 3); WAIT_V(8 + J1 + 2 * J2); WAIT_L(0); \
        CHECKQ(q); \
        poff += pinc; \
        COMPUTE(0); \
        SEGB(0, t + 4); WAIT_V(8 + 2 * J2); WAIT_L(0); COMPUTE(1); \
        if (J1) { if (wid == 1) { if (lane < nfl) __builtin_amdgcn_raw_buffer_store_b32(call, rWS, lane * 4, qfl + (q + 2) * 64, 16); } } \
        SEGA(1, t + 4); if (J1) WAIT_VX(8, wid, 1); else WAIT_V(8); \
        WAIT_L(0); COMPUTE(0); \
        if (J2) { CREAD; SEGB(1, t + 5); CSTORE(qws + 256 + (q + 3) * (4 * WKT_BYTES)); } else SEGB(1, t + 5); \
        if (J1) WAIT_VX(8 + J2, wid, 1); else WAIT_V(8); \
        WAIT_L(0); COMPUTE(1); } while (0)
    {
        const bool two = !(xg & 1) && nt != 0, helper = src == p.rank && xi >= 16;
        const int n1 = two ? NQ - 3 : 0;
        const int n2 = (two || helper) ? NQ - 2 : 1;
        int q = 0;
#pragma nounroll
        for (; q < n1; ++q) { const int t = q * 4; TRIP(1, 1); nfl = 2; }
#pragma nounroll
        for (; q < n2; ++q) { const int t = q * 4; TRIP(1, 0); qgs = JGS(0, xi - 16); qws = JWS(0, xi - 16); qfl = JFL(0, xi - 16); }
#pragma nounroll
        for (; q < NQ - 2; ++q) { const int t = q * 4; TRIP(0, 0); }
    }
#pragma nounroll
    for (int it = NIT - 4; it < NIT - 1; ++it) {
        const int t = it * 2;
        const int sel = __builtin_amdgcn_readfirstlane((it == NIT - 4) ? wid : 1);
        unsigned pv = *(const volatile LAS unsigned*)(lds + POLL + lane * 4);
        SEGA(0, t + 1); WAIT_V(8); WAIT_L(0);
        if (it == NIT - 3) CHECKQ(NQ - 2);
        COMPUTE(0);
        SEGB(0, t + 2);
        if (sel == 0) BLDS(rWS, POLL, 4, poff, 0, 17);
        WAIT_VX(8, sel, 0); WAIT_L(0); COMPUTE(1);
        SEGA(1, t + 2); WAIT_VX(8, sel, 0); WAIT_L(0); COMPUTE(0);
        SEGB(1, t + 3); WAIT_V(8); WAIT_L(0); COMPUTE(1);
    }
    STAMP(39); CSTAMP(51);
    int ln;
    asm volatile("v_mbcnt_lo_u32_b32 %0, -1, 0\n\tv_mbcnt_hi_u32_b32 %0, -1, %0" : "=v"(ln));
    const int fr = ln & 15, fq = ln >> 4;
    float* obase = p.out + (size_t)(mt * 256 + wr * 64 + fr) * NLOC + nt * 256 + wc * 32 + fq * 4;
    const float qnan = __builtin_nanf("");
    unsigned tmov;
#define OUTQ(ai, bj) do { _Pragma("unroll") for (int m = 0; m < 4; ++m) _Pragma("unroll") for (int n = 0; n < 2; ++n) { \
        f32x4 v = acc[ai][bj][m][n]; \
        v[0] = fmaxf(v[0], 0.f); v[1] = fmaxf(v[1], 0.f); v[2] = fmaxf(v[2], 0.f); v[3] = fmaxf(v[3], 0.f); \
        if (bad) v = (f32x4){qnan, qnan, qnan, qnan}; \
        float* dst = obase + (size_t)((ai) * 128 + m * 16) * NLOC + (bj) * 128 + n * 16; \
        asm volatile("global_store_dwordx4 %0, %1, off sc1\n\ts_nop 1" :: "v"(dst), "v"(v) : "memory"); } } while (0)
    {
        SEGA(0, NKT - 1); WAIT_V(8); WAIT_L(0); COMPUTE(0);
        LDA(At, 0, 1); WAIT_V(2); WAIT_L(0); COMPUTE(1);
        LDB(B0, 1, 0); LDA(At, 1, 0); LDB(B1, 1, 1);
        asm volatile("global_load_dword %0, %1, off sc1\n\ts_waitcnt vmcnt(0)" : "=&v"(tmov) : "v"(tmo) : "memory");
        WAIT_L(0);
        COMPUTE(0);
        const bool bad = __builtin_amdgcn_readfirstlane(tmov) == call;
        asm volatile("s_nop 15" ::: "memory"); SCHED;
        LDA(At, 1, 1);
        OUTQ(0, 0); OUTQ(0, 1);
        WAIT_L(0);
        BAR; SCHED; __builtin_amdgcn_s_setprio(1); MMH(1, 0); MMH(1, 1); __builtin_amdgcn_s_setprio(0);
        asm volatile("s_nop 15\n\ts_nop 15" ::: "memory"); SCHED;
        STAMP(40);
        OUTQ(1, 0); OUTQ(1, 1);
    }
    if (ln == 0) asm volatile("global_store_dword %0, %1, off sc1" :: "v"(ctr), "v"(call) : "memory");
    STAMP(41);
    WAIT_V(0);
    STAMP(42);
    if (wr == 0) BAR;
}

}

extern "C" void kernel_launch(int rank, int nranks, ncclComm_t comm,
                              hipStream_t stream,
                              void* const* d_in, const int* in_sizes, int n_in,
                              void* d_out, int out_size,
                              const int* in_dtypes, int out_dtype,
                              void* const* d_out_all,
                              void* const* d_ws_all, size_t ws_bytes) {
    if (nranks != NRANKS || (unsigned)rank >= (unsigned)NRANKS || ws_bytes < WS_NEEDED + (8u << 20) || n_in < 2 ||
        in_sizes[0] != MLOC * K_ || in_sizes[1] != K_ * NFULL || out_size != M_ * NLOC) {
        fprintf(stderr, "kernel_launch: unexpected problem shape\n");
        return;
    }
    HIP_CHECK(hipFuncSetAttribute((const void*)fused_kernel, hipFuncAttributeMaxDynamicSharedMemorySize, LDS_BYTES));
    Params p;
    p.x = (const float*)d_in[0];
    p.w = (const float*)d_in[1];
    p.out = (float*)d_out;
    const size_t wsoff = (ws_bytes - WS_NEEDED - (4u << 20)) & ~(size_t)((2u << 20) - 1);
    for (int q = 0; q < NRANKS; ++q) p.ws[q] = (char*)d_ws_all[q] + wsoff;
    p.rank = rank;
    {
        const unsigned half_mask = 0x4Du;
        p.srcmap = p.grpmap = 0;
        for (int r = 0; r < NRANKS; ++r) {
            const int mine = (half_mask >> r) & 1;
            int g = 0;
            for (int pass = 0; pass < 2; ++pass)
                for (int d = 0; d < NRANKS; ++d) {
                    const int q = (r + d) & 7;
                    if (((int)((half_mask >> q) & 1) == mine) != (pass == 0)) continue;
                    const int gg = (g == 0) ? SELFG : (g == SELFG ? 0 : g);
                    if (r == rank) p.srcmap |= (unsigned)q << (4 * gg);
                    if (q == rank) p.grpmap |= (unsigned)gg << (4 * r);
                    ++g;
                }
        }
    }
    hipLaunchKernelGGL(fused_kernel, dim3(256), dim3(512), LDS_BYTES, stream, p);
    HIP_CHECK(hipGetLastError());
}
```

```cpp
#include <hip/hip_runtime.h>
#include <rccl/rccl.h>
#include <cstdio>

#define HIP_CHECK(expr) do { hipError_t _e = (expr); if (_e != hipSuccess) { \
    fprintf(stderr, "HIP error %s at %s:%d\n", hipGetErrorString(_e), __FILE__, __LINE__); \
    return; } } while (0)

namespace {

constexpr int SELFG = 1;
static_assert(SELFG & 1, "the helpers of column tile 0 must not run two jobs of their own");

typedef _Float16 h8 __attribute__((ext_vector_type(8)));
typedef _Float16 h4 __attribute__((ext_vector_type(4)));
typedef float f32x4 __attribute__((ext_vector_type(4)));
typedef unsigned u32x4 __attribute__((ext_vector_type(4)));
#define LAS __attribute__((address_space(3)))

constexpr int NRANKS = 8;
constexpr int M_ = 4096, K_ = 4096, NLOC = 4096, NFULL = 32768, MLOC = 512;
constexpr int NKT = K_ / 64;
constexpr int NIT = NKT / 2;
constexpr int NQ = NKT / 4;
constexpr int UNIT = 16384;
constexpr int WKT_BYTES = 32 * UNIT;
constexpr int XKT_BYTES = 4 * UNIT;
constexpr int X16_BYTES = NKT * XKT_BYTES;
constexpr int W16_BYTES = NKT * WKT_BYTES;
constexpr int WROW = NFULL * 4;
constexpr int WPAIR = 128 * WROW;
constexpr int WS_TMO = 0;
constexpr int WS_CTR = 4096;
constexpr int WS_XCNT = WS_CTR + 256 * 128;
constexpr int WS_MBOX = WS_XCNT + 24 * 128;
constexpr int WS_CLAIM = WS_MBOX + 256 * 128;
constexpr int WS_WFLAG = WS_CLAIM + 32 * 128;
constexpr int WS_X16 = 1 << 20;
constexpr int WS_W16 = WS_X16 + 2 * X16_BYTES;
constexpr size_t WS_NEEDED = (size_t)WS_W16 + W16_BYTES;
static_assert(WS_WFLAG + 16 * NQ * 16 * 4 <= WS_X16, "control area");
constexpr unsigned long long DEADLINE_TICKS = 100000000ull;
constexpr int SCR = 8 * 16384;
constexpr int POLL = SCR + 16384;
constexpr int ARR = POLL + 256;
constexpr int LDS_BYTES = ARR + 16;
constexpr int BULK_DELAY = 104;

struct Params {
    const float* x;
    const float* w;
    float* out;
    char* ws[NRANKS];
    int rank;
    unsigned srcmap;
    unsigned grpmap;
};

__device__ __forceinline__ h8 cvt8(f32x4 a, f32x4 b) {
    h8 r;
    r[0] = (_Float16)a[0]; r[1] = (_Float16)a[1]; r[2] = (_Float16)a[2]; r[3] = (_Float16)a[3];
    r[4] = (_Float16)b[0]; r[5] = (_Float16)b[1]; r[6] = (_Float16)b[2]; r[7] = (_Float16)b[3];
    return r;
}

#define WAIT_V(n) asm volatile("s_waitcnt vmcnt(%0)" :: "n"(n) : "memory")
#define WAIT_L(n) do { asm volatile("" ::: "memory"); __builtin_amdgcn_s_waitcnt(0xC07F); asm volatile("" ::: "memory"); } while (0)
#define WAIT_VX(n, sel, w) asm volatile("s_waitcnt vmcnt(%0)\n\ts_cmp_eq_u32 %2, %3\n\ts_cbranch_scc1 1f\n\ts_waitcnt vmcnt(%1)\n1:" \
        :: "n"((n) + 1), "n"(n), "s"(sel), "n"(w) : "memory", "scc")
#define BAR __builtin_amdgcn_s_barrier()
#define SCHED __builtin_amdgcn_sched_barrier(0)
#define RSRC(ptr, bytes) __builtin_amdgcn_make_buffer_rsrc((void*)(ptr), 0, (bytes), 0x00020000)
#define BLDS(rsrc, ldsoff, size, voff, soff, aux) \
        __builtin_amdgcn_raw_ptr_buffer_load_lds(rsrc, (LAS void*)(lds + (ldsoff)), size, voff, soff, 0, aux)

__device__ __forceinline__ void spin_flags(const unsigned* lane_ptr, unsigned need, unsigned call, unsigned* tmo) {
    const unsigned long long t0 = __builtin_amdgcn_s_memrealtime();
    for (unsigned n = 1;; ++n) {
        unsigned v;
        asm volatile("global_load_dword %0, %1, off sc0 sc1\n\ts_waitcnt vmcnt(0)" : "=&v"(v) : "v"(lane_ptr) : "memory");
        if (__all(v >= need)) return;
        if ((n & 15u) == 0u) {
            if (__builtin_amdgcn_readfirstlane(__hip_atomic_load(tmo, __ATOMIC_RELAXED, __HIP_MEMORY_SCOPE_AGENT)) == call
                || __builtin_amdgcn_s_memrealtime() - t0 > DEADLINE_TICKS) {
                __hip_atomic_store(tmo, call, __ATOMIC_RELAXED, __HIP_MEMORY_SCOPE_AGENT);
                return;
            }
        }
        __builtin_amdgcn_s_sleep(1);
    }
}

__global__ __launch_bounds__(512, 2) void fused_kernel(Params p) {
    extern __shared__ __attribute__((aligned(16))) unsigned char shm[];
    LAS unsigned char* lds = (LAS unsigned char*)shm;
    const int tid = threadIdx.x;
    const int wid = __builtin_amdgcn_readfirstlane(tid >> 6), lane = tid & 63;
    char* myws = p.ws[p.rank];
    const int bid = blockIdx.x;
    const int xi = bid >> 3, nt = xi >> 1;
    const int kg = (bid & 7) * 2 + (xi & 1);
    unsigned* tmo = (unsigned*)(myws + WS_TMO);
    const auto rWS = RSRC(myws, (int)WS_NEEDED);
    const auto rW = RSRC(p.w + p.rank * NLOC, (K_ - 1) * WROW + NLOC * 4);
    unsigned* ctr = (unsigned*)(myws + WS_CTR + bid * 128 + wid * 4);
    unsigned callv, call;
    const int gvoff = (lane >> 3) * WROW + (((lane & 7) ^ ((lane >> 5) << 2)) << 4);
    const int gsoff = kg * 8 * WROW + nt * 1024;
    const int wsoff = WS_W16 + (kg >> 3) * WKT_BYTES + nt * 2 * UNIT + (((kg >> 2) & 1) << 10) + ((kg & 3) << 8);
    unsigned* wflag_row = (unsigned*)(myws + WS_WFLAG) + nt * NQ * 16;
    LAS unsigned* arr = (LAS unsigned*)(lds + ARR);
    if (tid < 3) arr[tid] = 0u;
    WAIT_L(0); BAR;
    if (wid == 2) {
        const int u = bid >> 4, blk = bid & 15;
        const float* xsrc = p.x + (size_t)((u & 3) * 128 + (blk >> 1) * 16 + (lane & 15)) * K_ + (u >> 2) * 64 + (blk & 1) * 32 + (lane >> 4) * 8;
        f32x4 a0, a1;
        asm volatile("global_load_dwordx4 %0, %3, off nt\n\tglobal_load_dwordx4 %1, %3, off offset:16 nt\n\t"
                     "s_load_dword %2, %4, 0x0\n\ts_waitcnt vmcnt(0) lgkmcnt(0)"
                     : "=&v"(a0), "=&v"(a1), "=&s"(callv) : "v"(xsrc), "s"(ctr) : "memory");
        call = callv + 1u;
        char* xd = myws + WS_X16 + (call & 1u) * X16_BYTES + u * UNIT + blk * 1024 + lane * 16;
        const h8 v0 = cvt8(a0, a1);
        asm volatile("global_store_dwordx4 %0, %1, off sc0 sc1\n\ts_waitcnt vmcnt(0)" :: "v"(xd), "v"(v0) : "memory");
        unsigned cnt = 0;
        if (lane == 0) cnt = __hip_atomic_fetch_add((unsigned*)(myws + WS_XCNT + (8 + u) * 128), 1u, __ATOMIC_RELAXED, __HIP_MEMORY_SCOPE_AGENT);
        if (__builtin_amdgcn_readfirstlane(cnt) + 1u == 16u * call) {
#pragma unroll
            for (int i = 0; i < 2; ++i) {
                char* w0 = p.ws[4 * i] + (((p.grpmap >> (16 * i)) & 7u) << 7);
                char* w1 = p.ws[4 * i + 1] + (((p.grpmap >> (16 * i + 4)) & 7u) << 7);
                char* w2 = p.ws[4 * i + 2] + (((p.grpmap >> (16 * i + 8)) & 7u) << 7);
                char* w3 = p.ws[4 * i + 3] + (((p.grpmap >> (16 * i + 12)) & 7u) << 7);
                char* f = (lane < 32 ? (lane < 16 ? w0 : w1) : (lane < 48 ? w2 : w3)) + WS_MBOX
                          + ((lane & 15) * 2 + ((u >> 1) & 1)) * 1024 + (24 + (u >> 2) * 2 + (u & 1)) * 4;
                asm volatile("global_store_dword %0, %1, off sc0 sc1" :: "v"(f), "v"(call) : "memory");
            }
        }
    } else if (wid == 3) {
        asm volatile("s_load_dword %0, %1, 0x0\n\ts_waitcnt lgkmcnt(0)" : "=&s"(callv) : "s"(ctr) : "memory");
        call = callv + 1u;
        unsigned g = (unsigned)__builtin_amdgcn_s_getreg(20 | (3 << 11)) & 7u;
        unsigned* cl = (unsigned*)(myws + WS_CLAIM + xi * 128);
        for (int i = 0; i < 8; ++i) {
            unsigned old = 0;
            if (lane == 0) old = __hip_atomic_fetch_max(cl + g, call, __ATOMIC_RELAXED, __HIP_MEMORY_SCOPE_AGENT);
            if (__builtin_amdgcn_readfirstlane(old) < call) break;
            g = (g + 1u) & 7u;
        }
        if (lane == 0) *(volatile LAS unsigned*)(arr + 2) = g + 1u;
        const unsigned* mbox = (const unsigned*)(myws + WS_MBOX + (xi * 8 + g) * 128);
        __builtin_amdgcn_s_sleep(32);
        spin_flags(mbox + (lane & 31), (lane & 31) >= 8 ? call : 0u, call, tmo);
    } else if (wid < 2) {
        __builtin_amdgcn_s_sleep(BULK_DELAY);
        const int u = (bid < 240) ? 16 + bid : 255;
        const float* xs0 = p.x + (size_t)((u & 3) * 128 + wid * 64 + (lane & 15)) * K_ + (u >> 2) * 64 + (lane >> 4) * 8;
        const float* xs1 = xs0 + 16 * K_; const float* xs2 = xs0 + 32 * K_; const float* xs3 = xs0 + 48 * K_;
        f32x4 a[16];
#define XL4(i, ptr) asm volatile("global_load_dwordx4 %0, %4, off nt\n\tglobal_load_dwordx4 %1, %4, off offset:16 nt\n\t" \
                     "global_load_dwordx4 %2, %4, off offset:128 nt\n\tglobal_load_dwordx4 %3, %4, off offset:144 nt" \
                     : "=&v"(a[i]), "=&v"(a[i + 1]), "=&v"(a[i + 2]), "=&v"(a[i + 3]) : "v"(ptr) : "memory")
        XL4(0, xs0); XL4(4, xs1); XL4(8, xs2); XL4(12, xs3);
        asm volatile("s_load_dword %16, %17, 0x0\n\ts_waitcnt vmcnt(0) lgkmcnt(0)"
                     : "+v"(a[0]), "+v"(a[1]), "+v"(a[2]), "+v"(a[3]), "+v"(a[4]), "+v"(a[5]), "+v"(a[6]), "+v"(a[7]),
                       "+v"(a[8]), "+v"(a[9]), "+v"(a[10]), "+v"(a[11]), "+v"(a[12]), "+v"(a[13]), "+v"(a[14]), "+v"(a[15]), "=&s"(callv)
                     : "s"(ctr) : "memory");
        call = callv + 1u;
        if (bid < 240) {
            char* xd = myws + WS_X16 + (call & 1u) * X16_BYTES + u * UNIT + wid * 8192 + lane * 16;
#pragma unroll
            for (int i = 0; i < 8; ++i) {
                const h8 v = cvt8(a[2 * i], a[2 * i + 1]);
                asm volatile("global_store_dwordx4 %0, %1, off sc0 sc1\n\ts_nop 1" :: "v"(xd + i * 1024), "v"(v) : "memory");
            }
            WAIT_V(0);
            unsigned old = 0;
            if (lane == 0) old = __hip_atomic_fetch_add(arr, 1u, __ATOMIC_RELAXED, __HIP_MEMORY_SCOPE_WORKGROUP);
            if (__builtin_amdgcn_readfirstlane(old) == 1u) {
                const int shard = u >> 5;
                unsigned cnt = 0;
                if (lane == 0) cnt = __hip_atomic_fetch_add((unsigned*)(myws + WS_XCNT + shard * 128), 1u, __ATOMIC_RELAXED, __HIP_MEMORY_SCOPE_AGENT);
                if (__builtin_amdgcn_readfirstlane(cnt) + 1u == (shard ? 32u : 16u) * call) {
#pragma unroll
                    for (int i = 0; i < 4; ++i) {
                        const int q0 = 2 * i, q1 = 2 * i + 1;
                        char* w0 = p.ws[q0] + (((p.grpmap >> (4 * q0)) & 7u) << 7);
                        char* w1 = p.ws[q1] + (((p.grpmap >> (4 * q1)) & 7u) << 7);
                        char* f = (lane < 32 ? w0 : w1) + WS_MBOX + (lane & 31) * 1024 + shard * 4;
                        asm volatile("global_store_dword %0, %1, off sc0 sc1" :: "v"(f), "v"(call) : "memory");
                    }
                }
            }
        }
    } else {
        const int j = wid - 4;
        if (j < 2) __builtin_amdgcn_s_sleep(4); else __builtin_amdgcn_s_sleep(BULK_DELAY);
#pragma unroll
        for (int c = 0; c < 8; ++c) BLDS(rW, (j * 8 + c) * 1024, 16, gvoff, gsoff + j * WPAIR + c * 128, 2);
        asm volatile("s_load_dword %0, %1, 0x0\n\ts_waitcnt lgkmcnt(0)" : "=&s"(callv) : "s"(ctr) : "memory");
        call = callv + 1u;
        const int rd = j * 8192 + (lane >> 5) * 1024 + (lane & 31) * 4;
        const int dv = ((lane >> 4) << 11) + ((lane & 15) << 4);
#pragma unroll
        for (int c = 0; c < 4; ++c) {
            if (c == 0) WAIT_V(6); else if (c == 1) WAIT_V(5); else if (c == 2) WAIT_V(4); else WAIT_V(3);
            const LAS float* c0 = (const LAS float*)(lds + rd + c * 2048);
            const LAS float* c1 = (const LAS float*)(lds + (rd ^ 64) + 512 + c * 2048);
            const h8 hv = cvt8((f32x4){c0[0], c0[32], c0[64], c0[96]}, (f32x4){c1[0], c1[32], c1[64], c1[96]});
            __builtin_amdgcn_raw_buffer_store_b128(__builtin_bit_cast(u32x4, hv), rWS, dv, wsoff + j * 2 * WKT_BYTES + c * 8192, 16);
        }
        if (j < 2) {
            WAIT_V(0);
            unsigned old = 0;
            if (lane == 0) old = __hip_atomic_fetch_add(arr + 1, 1u, __ATOMIC_RELAXED, __HIP_MEMORY_SCOPE_WORKGROUP);
            if (__builtin_amdgcn_readfirstlane(old) == 1u && lane < 16) {
                char* f = myws + WS_MBOX + (nt * 16 + lane) * 128 + (8 + kg) * 4;
                asm volatile("global_store_dword %0, %1, off sc1" :: "v"(f), "v"(call) : "memory");
            }
        }
    }
    WAIT_L(0);
    BAR;
    const int xg = (__builtin_amdgcn_readfirstlane(*(const volatile LAS unsigned*)(arr + 2)) - 1) & 7;
    const int src = (p.srcmap >> (xg * 4)) & (NRANKS - 1);
    const int mt = src * 2 + (xi & 1);
    const auto rX = RSRC(p.ws[src] + WS_X16 + (call & 1u) * X16_BYTES, X16_BYTES);
    const int wr = wid >> 2, wc = wid & 3;
    const int tv = tid * 16;
    const int xs0 = (mt & 1) * 2 * UNIT, ws0 = WS_W16 + nt * 2 * UNIT;
    const int ldsw = wid * 1024;
    const int aoff = wr * 8192 + lane * 16, boff = wc * 4096 + lane * 16;
    const int lj = ((xg >> 1) * 2 + (xi & 1)) * 2 + (xg & 1);
#define JGS(ct, j) ((j) * 8 * WROW + (ct) * 1024 + wr * WPAIR + wc * 256)
#define JWS(ct, j) (WS_W16 + ((j) >> 3) * WKT_BYTES + (ct) * 2 * UNIT + ((((j) >> 2) & 1) << 10) + (((j) & 3) << 8) + wr * 2 * WKT_BYTES + wc * 8192)
#define JFL(ct, j) (WS_WFLAG + ((ct) * NQ * 16 + (j)) * 4)
    int qgs = JGS(nt, lj), qws = JWS(nt, lj), qfl = JFL(nt, lj);
    int nfl = 1;
    const int qdv = ((lane >> 4) << 11) + ((lane & 15) << 4);
    const int qrd = SCR + wid * 2048 + (lane >> 5) * 1024 + (lane & 31) * 4;
    const int pl = (lane >= 24) ? (lane & 15) : lane;
    const int pthr = (pl < 16) ? 0 : 2 * (pl - 16) - 1;
    const int pinc = (pl < 16) ? 64 : 0;
    int poff = (pl < 16) ? WS_WFLAG + (nt * NQ + 1) * 64 + pl * 4 : WS_MBOX + (xi * 8 + xg) * 128 + (pl - 16) * 4;
#define SA(b, h) (((b) * 2 + (h)) * 16384)
#define SB(b, h) ((4 + (b) * 2 + (h)) * 16384)
#define STX(b, h, kt) do { BLDS(rX, SA(b, h) + ldsw, 16, tv, xs0 + (kt) * XKT_BYTES + (h) * UNIT, 0); \
                           BLDS(rX, SA(b, h) + ldsw + 8192, 16, tv, xs0 + (kt) * XKT_BYTES + (h) * UNIT + 8192, 0); } while (0)
#define STW(b, h, kt) do { BLDS(rWS, SB(b, h) + ldsw, 16, tv, ws0 + (kt) * WKT_BYTES + (h) * UNIT, 0); \
                           BLDS(rWS, SB(b, h) + ldsw + 8192, 16, tv, ws0 + (kt) * WKT_BYTES + (h) * UNIT + 8192, 0); } while (0)
#define LDA(dst, b, h) do { _Pragma("unroll") for (int m = 0; m < 4; ++m) _Pragma("unroll") for (int k = 0; k < 2; ++k) \
        dst[m][k] = *(const LAS h8*)(lds + SA(b, h) + aoff + m * 2048 + k * 1024); } while (0)
#define LDB(dst, b, h) do { _Pragma("unroll") for (int n = 0; n < 2; ++n) _Pragma("unroll") for (int k = 0; k < 2; ++k) \
        dst[n][k] = *(const LAS h8*)(lds + SB(b, h) + boff + n * 2048 + k * 1024); } while (0)
#define MMH(ai, h) asm volatile( \
            "v_mfma_f32_16x16x32_f16 %0, %8, %16, %0\n\t" \
            "v_mfma_f32_16x16x32_f16 %0, %9, %17, %0\n\t" \
            "v_mfma_f32_16x16x32_f16 %1, %11, %17, %1\n\t" \
            "v_mfma_f32_16x16x32_f16 %1, %10, %16, %1\n\t" \
            "v_mfma_f32_16x16x32_f16 %2, %12, %16, %2\n\t" \
            "v_mfma_f32_16x16x32_f16 %2, %13, %17, %2\n\t" \
            "v_mfma_f32_16x16x32_f16 %3, %15, %17, %3\n\t" \
            "v_mfma_f32_16x16x32_f16 %3, %14, %16, %3\n\t" \
            "v_mfma_f32_16x16x32_f16 %7, %14, %18, %7\n\t" \
            "v_mfma_f32_16x16x32_f16 %7, %15, %19, %7\n\t" \
            "v_mfma_f32_16x16x32_f16 %6, %13, %19, %6\n\t" \
            "v_mfma_f32_16x16x32_f16 %6, %12, %18, %6\n\t" \
            "v_mfma_f32_16x16x32_f16 %5, %10, %18, %5\n\t" \
            "v_mfma_f32_16x16x32_f16 %5, %11, %19, %5\n\t" \
            "v_mfma_f32_16x16x32_f16 %4, %9, %19, %4\n\t" \
            "v_mfma_f32_16x16x32_f16 %4, %8, %18, %4\n\t" \
            : "+v"(acc[ai][0][2*(h)][0]), "+v"(acc[ai][0][2*(h)][1]), "+v"(acc[ai][1][2*(h)][0]), "+v"(acc[ai][1][2*(h)][1]), \
              "+v"(acc[ai][0][2*(h)+1][0]), "+v"(acc[ai][0][2*(h)+1][1]), "+v"(acc[ai][1][2*(h)+1][0]), "+v"(acc[ai][1][2*(h)+1][1]) \
            : "v"(B0[0][0]), "v"(B0[0][1]), "v"(B0[1][0]), "v"(B0[1][1]), "v"(B1[0][0]), "v"(B1[0][1]), "v"(B1[1][0]), "v"(B1[1][1]), \
              "v"(At[2*(h)][0]), "v"(At[2*(h)][1]), "v"(At[2*(h)+1][0]), "v"(At[2*(h)+1][1]))
#define COMPUTE(ai) do { BAR; SCHED; __builtin_amdgcn_s_setprio(1); MMH(ai, 0); MMH(ai, 1); __builtin_amdgcn_s_setprio(0); BAR; SCHED; } while (0)
#define SEGA(b, tn) do { STX((b) ^ 1, 1, tn); LDB(B0, b, 0); LDA(At, b, 0); LDB(B1, b, 1); } while (0)
#define SEGB(b, tn) do { STX(b, 0, tn); LDA(At, b, 1); STW(b, 0, tn); STW(b, 1, tn); } while (0)
#define CHECKQ(q) do { asm volatile("" : "+v"(pv)); \
        if (__builtin_expect(!__all((int)(pv >= call) | (int)(pthr > (q))), 0)) \
            if (wid == 0) spin_flags((const unsigned*)(myws + poff), pthr > (q) ? 0u : call, call, tmo); } while (0)

    f32x4 acc[2][2][4][2];
#pragma unroll
    for (int a = 0; a < 2; ++a)
#pragma unroll
        for (int b = 0; b < 2; ++b)
#pragma unroll
            for (int m = 0; m < 4; ++m)
#pragma unroll
                for (int n = 0; n < 2; ++n) acc[a][b][m][n] = (f32x4){0.f, 0.f, 0.f, 0.f};
    h8 At[4][2], B0[2][2], B1[2][2];

    STX(0, 0, 0); STX(0, 1, 0); STW(0, 0, 0); STW(0, 1, 0);
    STX(1, 0, 1); STW(1, 0, 1); STW(1, 1, 1);
    WAIT_V(6); BAR;
    if (wid == 2 && lane == 0) asm volatile("global_store_dword %0, %1, off sc1" :: "v"(wflag_row + 16 + kg), "v"(call) : "memory");
    if (wr == 1) BAR;
#define GJOB(gs) do { BLDS(rW, SCR + 2 * ldsw, 16, gvoff, gs, 2); BLDS(rW, SCR + 2 * ldsw + 1024, 16, gvoff, (gs) + 128, 2); } while (0)
#define CREAD const LAS float* c0 = (const LAS float*)(lds + qrd); const LAS float* c1 = (const LAS float*)(lds + (qrd ^ 64) + 512); \
        const float f0 = c0[0], f1 = c0[32], f2 = c0[64], f3 = c0[96], f4 = c1[0], f5 = c1[32], f6 = c1[64], f7 = c1[96]
#define CSTORE(ws) do { SCHED; const h8 hv = cvt8((f32x4){f0, f1, f2, f3}, (f32x4){f4, f5, f6, f7}); \
        __builtin_amdgcn_raw_buffer_store_b128(__builtin_bit_cast(u32x4, hv), rWS, qdv, ws, 16); } while (0)
#define TRIP(J1, J2) do { \
        if (J1) GJOB(qgs + (q + 2) * (2 * WPAIR)); \
        SEGA(0, t + 1); WAIT_V(8 + 2 * J1 + J2); WAIT_L(0); COMPUTE(0); \
        SEGB(0, t + 2); \
        if (wid == 0) BLDS(rWS, POLL, 4, poff, 0, 17);            \
        WAIT_VX(8 + 2 * J1, wid, 0); WAIT_L(0); COMPUTE(1); \
        SEGA(1, t + 2); WAIT_VX(8, wid, 0); WAIT_L(0); COMPUTE(0); \
        if (J1) { CREAD; SEGB(1, t + 3); CSTORE(qws + (q + 2) * (4 * WKT_BYTES)); } else SEGB(1, t + 3); \
        WAIT_V(8 + J1); WAIT_L(0); COMPUTE(1); \
        if (J2) GJOB(qgs + 8 * WROW + (q + 3) * (2 * WPAIR)); \
        unsigned pv = *(const volatile LAS unsigned*)(lds + POLL + lane * 4); \
        SEGA(0, t + 3); WAIT_V(8 + J1 + 2 * J2); WAIT_L(0); \
        CHECKQ(q); \
        poff += pinc; \
        COMPUTE(0); \
        SEGB(0, t + 4); WAIT_V(8 + 2 * J2); WAIT_L(0); COMPUTE(1); \
        if (J1) { if (wid == 1) { if (lane < nfl) __builtin_amdgcn_raw_buffer_store_b32(call, rWS, lane * 4, qfl + (q + 2) * 64, 16); } } \
        SEGA(1, t + 4); if (J1) WAIT_VX(8, wid, 1); else WAIT_V(8); \
        WAIT_L(0); COMPUTE(0); \
        if (J2) { CREAD; SEGB(1, t + 5); CSTORE(qws + 256 + (q + 3) * (4 * WKT_BYTES)); } else SEGB(1, t + 5); \
        if (J1) WAIT_VX(8 + J2, wid, 1); else WAIT_V(8); \
        WAIT_L(0); COMPUTE(1); } while (0)
    {
        const bool two = !(xg & 1) && nt != 0, helper = src == p.rank && xi >= 16;
        const int n1 = two ? NQ - 3 : 0;
        const int n2 = (two || helper) ? NQ - 2 : 1;
        int q = 0;
#pragma nounroll
        for (; q < n1; ++q) { const int t = q * 4; TRIP(1, 1); nfl = 2; }
#pragma nounroll
        for (; q < n2; ++q) { const int t = q * 4; TRIP(1, 0); qgs = JGS(0, xi - 16); qws = JWS(0, xi - 16); qfl = JFL(0, xi - 16); }
#pragma nounroll
        for (; q < NQ - 2; ++q) { const int t = q * 4; TRIP(0, 0); }
    }
#pragma nounroll
    for (int it = NIT - 4; it < NIT - 1; ++it) {
        const int t = it * 2;
        const int sel = __builtin_amdgcn_readfirstlane((it == NIT - 4) ? wid : 1);
        unsigned pv = *(const volatile LAS unsigned*)(lds + POLL + lane * 4);
        SEGA(0, t + 1); WAIT_V(8); WAIT_L(0);
        if (it == NIT - 3) CHECKQ(NQ - 2);
        COMPUTE(0);
        SEGB(0, t + 2);
        if (sel == 0) BLDS(rWS, POLL, 4, poff, 0, 17);
        WAIT_VX(8, sel, 0); WAIT_L(0); COMPUTE(1);
        SEGA(1, t + 2); WAIT_VX(8, sel, 0); WAIT_L(0); COMPUTE(0);
        SEGB(1, t + 3); WAIT_V(8); WAIT_L(0); COMPUTE(1);
    }
    int ln;
    asm volatile("v_mbcnt_lo_u32_b32 %0, -1, 0\n\tv_mbcnt_hi_u32_b32 %0, -1, %0" : "=v"(ln));
    const int fr = ln & 15, fq = ln >> 4;
    float* obase = p.out + (size_t)(mt * 256 + wr * 64 + fr) * NLOC + nt * 256 + wc * 32 + fq * 4;
    const float qnan = __builtin_nanf("");
    unsigned tmov;
#define OUTQ(ai, bj) do { _Pragma("unroll") for (int m = 0; m < 4; ++m) _Pragma("unroll") for (int n = 0; n < 2; ++n) { \
        f32x4 v = acc[ai][bj][m][n]; \
        v[0] = fmaxf(v[0], 0.f); v[1] = fmaxf(v[1], 0.f); v[2] = fmaxf(v[2], 0.f); v[3] = fmaxf(v[3], 0.f); \
        if (bad) v = (f32x4){qnan, qnan, qnan, qnan}; \
        float* dst = obase + (size_t)((ai) * 128 + m * 16) * NLOC + (bj) * 128 + n * 16; \
        asm volatile("global_store_dwordx4 %0, %1, off sc1\n\ts_nop 1" :: "v"(dst), "v"(v) : "memory"); } } while (0)
    {
        SEGA(0, NKT - 1); WAIT_V(8); WAIT_L(0); COMPUTE(0);
        LDA(At, 0, 1); WAIT_V(2); WAIT_L(0); COMPUTE(1);
        LDB(B0, 1, 0); LDA(At, 1, 0); LDB(B1, 1, 1);
        asm volatile("global_load_dword %0, %1, off sc1\n\ts_waitcnt vmcnt(0)" : "=&v"(tmov) : "v"(tmo) : "memory");
        WAIT_L(0);
        COMPUTE(0);
        const bool bad = __builtin_amdgcn_readfirstlane(tmov) == call;
        asm volatile("s_nop 15" ::: "memory"); SCHED;
        LDA(At, 1, 1);
        OUTQ(0, 0); OUTQ(0, 1);
        WAIT_L(0);
        BAR; SCHED; __builtin_amdgcn_s_setprio(1); MMH(1, 0); MMH(1, 1); __builtin_amdgcn_s_setprio(0);
        asm volatile("s_nop 15\n\ts_nop 15" ::: "memory"); SCHED;
        OUTQ(1, 0); OUTQ(1, 1);
    }
    if (ln == 0) asm volatile("global_store_dword %0, %1, off sc1" :: "v"(ctr), "v"(call) : "memory");
    WAIT_V(0);
    if (wr == 0) BAR;
}

}

extern "C" void kernel_launch(int rank, int nranks, ncclComm_t comm,
                              hipStream_t stream,
                              void* const* d_in, const int* in_sizes, int n_in,
                              void* d_out, int out_size,
                              const int* in_dtypes, int out_dtype,
                              void* const* d_out_all,
                              void* const* d_ws_all, size_t ws_bytes) {
    if (nranks != NRANKS || (unsigned)rank >= (unsigned)NRANKS || ws_bytes < WS_NEEDED + (8u << 20) || n_in < 2 ||
        in_sizes[0] != MLOC * K_ || in_sizes[1] != K_ * NFULL || out_size != M_ * NLOC) {
        fprintf(stderr, "kernel_launch: unexpected problem shape\n");
        return;
    }
    HIP_CHECK(hipFuncSetAttribute((const void*)fused_kernel, hipFuncAttributeMaxDynamicSharedMemorySize, LDS_BYTES));
    Params p;
    p.x = (const float*)d_in[0];
    p.w = (const float*)d_in[1];
    p.out = (float*)d_out;
    const size_t wsoff = (ws_bytes - WS_NEEDED - (4u << 20)) & ~(size_t)((2u << 20) - 1);
    for (int q = 0; q < NRANKS; ++q) p.ws[q] = (char*)d_ws_all[q] + wsoff;
    p.rank = rank;
    {
        const unsigned half_mask = 0x4Du;
        p.srcmap = p.grpmap = 0;
        for (int r = 0; r < NRANKS; ++r) {
            const int mine = (half_mask >> r) & 1;
            int g = 0;
            for (int pass = 0; pass < 2; ++pass)
                for (int d = 0; d < NRANKS; ++d) {
                    const int q = (r + d) & 7;
                    if (((int)((half_mask >> q) & 1) == mine) != (pass == 0)) continue;
                    const int gg = (g == 0) ? SELFG : (g == SELFG ? 0 : g);
                    if (r == rank) p.srcmap |= (unsigned)q << (4 * gg);
                    if (q == rank) p.grpmap |= (unsigned)gg << (4 * r);
                    ++g;
                }
        }
    }
    hipLaunchKernelGGL(fused_kernel, dim3(256), dim3(512), LDS_BYTES, stream, p);
    HIP_CHECK(hipGetLastError());
}
```

```cpp
#include <hip/hip_runtime.h>
#include <rccl/rccl.h>
#include <cstdio>

#define HIP_CHECK(expr) do { hipError_t _e = (expr); if (_e != hipSuccess) { \
    fprintf(stderr, "HIP error %s at %s:%d\n", hipGetErrorString(_e), __FILE__, __LINE__); \
    return; } } while (0)

namespace {

constexpr int SELFG = 1;
static_assert(SELFG & 1, "the helpers of column tile 0 must not run two jobs of their own");

typedef _Float16 h8 __attribute__((ext_vector_type(8)));
typedef _Float16 h4 __attribute__((ext_vector_type(4)));
typedef float f32x4 __attribute__((ext_vector_type(4)));
typedef unsigned u32x4 __attribute__((ext_vector_type(4)));
#define LAS __attribute__((address_space(3)))

constexpr int NRANKS = 8;
constexpr int M_ = 4096, K_ = 4096, NLOC = 4096, NFULL = 32768, MLOC = 512;
constexpr int NKT = K_ / 64;
constexpr int NIT = NKT / 2;
constexpr int NQ = NKT / 4;
constexpr int UNIT = 16384;
constexpr int WKT_BYTES = 32 * UNIT;
constexpr int XKT_BYTES = 4 * UNIT;
constexpr int X16_BYTES = NKT * XKT_BYTES;
constexpr int W16_BYTES = NKT * WKT_BYTES;
constexpr int WROW = NFULL * 4;
constexpr int WPAIR = 128 * WROW;
constexpr int WS_TMO = 0;
constexpr int WS_CTR = 4096;
constexpr int WS_XCNT = WS_CTR + 256 * 128;
constexpr int WS_MBOX = WS_XCNT + 24 * 128;
constexpr int WS_CLAIM = WS_MBOX + 256 * 128;
constexpr int WS_WFLAG = WS_CLAIM + 32 * 128;
constexpr int WS_X16 = 1 << 20;
constexpr int WS_W16 = WS_X16 + 2 * X16_BYTES;
constexpr size_t WS_NEEDED = (size_t)WS_W16 + W16_BYTES;
static_assert(WS_WFLAG + 16 * NQ * 16 * 4 <= WS_X16, "control area");
constexpr unsigned long long DEADLINE_TICKS = 100000000ull;
constexpr int SCR = 8 * 16384;
constexpr int POLL = SCR + 16384;
constexpr int ARR = POLL + 256;
constexpr int LDS_BYTES = ARR + 16;
constexpr int BULK_DELAY = 104;

struct Params {
    const float* x;
    const float* w;
    float* out;
    char* ws[NRANKS];
    int rank;
    unsigned srcmap;
    unsigned grpmap;
};

__device__ __forceinline__ h8 cvt8(f32x4 a, f32x4 b) {
    h8 r;
    r[0] = (_Float16)a[0]; r[1] = (_Float16)a[1]; r[2] = (_Float16)a[2]; r[3] = (_Float16)a[3];
    r[4] = (_Float16)b[0]; r[5] = (_Float16)b[1]; r[6] = (_Float16)b[2]; r[7] = (_Float16)b[3];
    return r;
}

#define WAIT_V(n) asm volatile("s_waitcnt vmcnt(%0)" :: "n"(n) : "memory")
#define WAIT_L(n) do { asm volatile("" ::: "memory"); __builtin_amdgcn_s_waitcnt(0xC07F); asm volatile("" ::: "memory"); } while (0)
#define WAIT_VX(n, sel, w) asm volatile("s_waitcnt vmcnt(%0)\n\ts_cmp_eq_u32 %2, %3\n\ts_cbranch_scc1 1f\n\ts_waitcnt vmcnt(%1)\n1:" \
        :: "n"((n) + 1), "n"(n), "s"(sel), "n"(w) : "memory", "scc")
#define BAR __builtin_amdgcn_s_barrier()
#define SCHED __builtin_amdgcn_sched_barrier(0)
#define RSRC(ptr, bytes) __builtin_amdgcn_make_buffer_rsrc((void*)(ptr), 0, (bytes), 0x00020000)
#define BLDS(rsrc, ldsoff, size, voff, soff, aux) \
        __builtin_amdgcn_raw_ptr_buffer_load_lds(rsrc, (LAS void*)(lds + (ldsoff)), size, voff, soff, 0, aux)

__device__ __forceinline__ void spin_flags(const unsigned* lane_ptr, unsigned need, unsigned call, unsigned* tmo) {
    const unsigned long long t0 = __builtin_amdgcn_s_memrealtime();
    for (unsigned n = 1;; ++n) {
        unsigned v;
        asm volatile("global_load_dword %0, %1, off sc0 sc1\n\ts_waitcnt vmcnt(0)" : "=&v"(v) : "v"(lane_ptr) : "memory");
        if (__all(v >= need)) return;
        if ((n & 15u) == 0u) {
            if (__builtin_amdgcn_readfirstlane(__hip_atomic_load(tmo, __ATOMIC_RELAXED, __HIP_MEMORY_SCOPE_AGENT)) == call
                || __builtin_amdgcn_s_memrealtime() - t0 > DEADLINE_TICKS) {
                __hip_atomic_store(tmo, call, __ATOMIC_RELAXED, __HIP_MEMORY_SCOPE_AGENT);
                return;
            }
        }
        __builtin_amdgcn_s_sleep(1);
    }
}

__global__ __launch_bounds__(512, 2) void fused_kernel(Params p) {
    extern __shared__ __attribute__((aligned(16))) unsigned char shm[];
    LAS unsigned char* lds = (LAS unsigned char*)shm;
    const int tid = threadIdx.x;
    const int wid = __builtin_amdgcn_readfirstlane(tid >> 6), lane = tid & 63;
    char* myws = p.ws[p.rank];
    const int bid = blockIdx.x;
    const int xi = bid >> 3, nt = xi >> 1;
    const int kg = (bid & 7) * 2 + (xi & 1);
    unsigned* tmo = (unsigned*)(myws + WS_TMO);
    const auto rWS = RSRC(myws, (int)WS_NEEDED);
    const auto rW = RSRC(p.w + p.rank * NLOC, (K_ - 1) * WROW + NLOC * 4);
    unsigned* ctr = (unsigned*)(myws + WS_CTR + bid * 128 + wid * 4);
    unsigned callv, call;
    const int gvoff = (lane >> 3) * WROW + (((lane & 7) ^ ((lane >> 5) << 2)) << 4);
    const int gsoff = kg * 8 * WROW + nt * 1024;
    const int wsoff = WS_W16 + (kg >> 3) * WKT_BYTES + nt * 2 * UNIT + (((kg >> 2) & 1) << 10) + ((kg & 3) << 8);
    unsigned* wflag_row = (unsigned*)(myws + WS_WFLAG) + nt * NQ * 16;
    LAS unsigned* arr = (LAS unsigned*)(lds + ARR);
    if (tid < 3) arr[tid] = 0u;
    WAIT_L(0); BAR;
    if (wid == 2) {
        const int u = bid >> 4, blk = bid & 15;
        const float* xsrc = p.x + (size_t)((u & 3) * 128 + (blk >> 1) * 16 + (lane & 15)) * K_ + (u >> 2) * 64 + (blk & 1) * 32 + (lane >> 4) * 8;
        f32x4 a0, a1;
        asm volatile("global_load_dwordx4 %0, %3, off nt\n\tglobal_load_dwordx4 %1, %3, off offset:16 nt\n\t"
                     "s_load_dword %2, %4, 0x0\n\ts_waitcnt vmcnt(0) lgkmcnt(0)"
                     : "=&v"(a0), "=&v"(a1), "=&s"(callv) : "v"(xsrc), "s"(ctr) : "memory");
        call = callv + 1u;
        char* xd = myws + WS_X16 + (call & 1u) * X16_BYTES + u * UNIT + blk * 1024 + lane * 16;
        const h8 v0 = cvt8(a0, a1);
        asm volatile("global_store_dwordx4 %0, %1, off sc0 sc1\n\ts_waitcnt vmcnt(0)" :: "v"(xd), "v"(v0) : "memory");
        unsigned cnt = 0;
        if (lane == 0) cnt = __hip_atomic_fetch_add((unsigned*)(myws + WS_XCNT + (8 + u) * 128), 1u, __ATOMIC_RELAXED, __HIP_MEMORY_SCOPE_AGENT);
        if (__builtin_amdgcn_readfirstlane(cnt) + 1u == 16u * call) {
#pragma unroll
            for (int i = 0; i < 2; ++i) {
                char* w0 = p.ws[4 * i] + (((p.grpmap >> (16 * i)) & 7u) << 7);
                char* w1 = p.ws[4 * i + 1] + (((p.grpmap >> (16 * i + 4)) & 7u) << 7);
                char* w2 = p.ws[4 * i + 2] + (((p.grpmap >> (16 * i + 8)) & 7u) << 7);
                char* w3 = p.ws[4 * i + 3] + (((p.grpmap >> (16 * i + 12)) & 7u) << 7);
                char* f = (lane < 32 ? (lane < 16 ? w0 : w1) : (lane < 48 ? w2 : w3)) + WS_MBOX
                          + ((lane & 15) * 2 + ((u >> 1) & 1)) * 1024 + (24 + (u >> 2) * 2 + (u & 1)) * 4;
                asm volatile("global_store_dword %0, %1, off sc0 sc1" :: "v"(f), "v"(call) : "memory");
            }
        }
    } else if (wid == 3) {
        asm volatile("s_load_dword %0, %1, 0x0\n\ts_waitcnt lgkmcnt(0)" : "=&s"(callv) : "s"(ctr) : "memory");
        call = callv + 1u;
        unsigned g = (unsigned)__builtin_amdgcn_s_getreg(20 | (3 << 11)) & 7u;
        unsigned* cl = (unsigned*)(myws + WS_CLAIM + xi * 128);
        for (int i = 0; i < 8; ++i) {
            unsigned old = 0;
            if (lane == 0) old = __hip_atomic_fetch_max(cl + g, call, __ATOMIC_RELAXED, __HIP_MEMORY_SCOPE_AGENT);
            if (__builtin_amdgcn_readfirstlane(old) < call) break;
            g = (g + 1u) & 7u;
        }
        if (lane == 0) *(volatile LAS unsigned*)(arr + 2) = g + 1u;
        const unsigned* mbox = (const unsigned*)(myws + WS_MBOX + (xi * 8 + g) * 128);
        __builtin_amdgcn_s_sleep(32);
        spin_flags(mbox + (lane & 31), (lane & 31) >= 8 ? call : 0u, call, tmo);
    } else if (wid < 2) {
        const int v = bid * 2 + wid + 32;
        const int shard = v >> 6, xrb = shard ? (v >> 1) & 31 : v - 32, xkt = shard ? shard * 8 + (v & 1) * 4 : 4;
        if (bid >= 240) {
            asm volatile("s_load_dword %0, %1, 0x0\n\ts_waitcnt lgkmcnt(0)" : "=&s"(callv) : "s"(ctr) : "memory");
            call = callv + 1u;
        } else {
            __builtin_amdgcn_s_sleep(BULK_DELAY);
            const float* xs = p.x + (size_t)(xrb * 16 + (lane & 15)) * K_ + xkt * 64 + (lane >> 4) * 8;
            f32x4 a[16];
#define XL4(i, o) asm volatile("global_load_dwordx4 %0, %4, off offset:%5 nt\n\tglobal_load_dwordx4 %1, %4, off offset:%6 nt\n\t" \
                     "global_load_dwordx4 %2, %4, off offset:%7 nt\n\tglobal_load_dwordx4 %3, %4, off offset:%8 nt" \
                     : "=&v"(a[i]), "=&v"(a[i + 1]), "=&v"(a[i + 2]), "=&v"(a[i + 3]) : "v"(xs), "n"(o), "n"((o) + 16), "n"((o) + 128), "n"((o) + 144) : "memory")
            XL4(0, 0); XL4(4, 256); XL4(8, 512); XL4(12, 768);
            asm volatile("s_load_dword %16, %17, 0x0\n\ts_waitcnt vmcnt(0) lgkmcnt(0)"
                         : "+v"(a[0]), "+v"(a[1]), "+v"(a[2]), "+v"(a[3]), "+v"(a[4]), "+v"(a[5]), "+v"(a[6]), "+v"(a[7]),
                           "+v"(a[8]), "+v"(a[9]), "+v"(a[10]), "+v"(a[11]), "+v"(a[12]), "+v"(a[13]), "+v"(a[14]), "+v"(a[15]), "=&s"(callv)
                         : "s"(ctr) : "memory");
            call = callv + 1u;
            const int xo = WS_X16 + (call & 1u) * X16_BYTES + xkt * XKT_BYTES + xrb * 2048;
#pragma unroll
            for (int i = 0; i < 8; ++i)
                __builtin_amdgcn_raw_buffer_store_b128(__builtin_bit_cast(u32x4, cvt8(a[2 * i], a[2 * i + 1])), rWS, lane * 16, xo + (i >> 1) * XKT_BYTES + (i & 1) * 1024, 17);
            WAIT_V(0);
            unsigned old = 0;
            if (lane == 0) old = __hip_atomic_fetch_add(arr, 1u, __ATOMIC_RELAXED, __HIP_MEMORY_SCOPE_WORKGROUP);
            if (__builtin_amdgcn_readfirstlane(old) == 1u) {
                unsigned cnt = 0;
                if (lane == 0) cnt = __hip_atomic_fetch_add((unsigned*)(myws + WS_XCNT + shard * 128), 1u, __ATOMIC_RELAXED, __HIP_MEMORY_SCOPE_AGENT);
                if (__builtin_amdgcn_readfirstlane(cnt) + 1u == (shard ? 32u : 16u) * call) {
#pragma unroll
                    for (int i = 0; i < 4; ++i) {
                        const int q0 = 2 * i, q1 = 2 * i + 1;
                        char* w0 = p.ws[q0] + (((p.grpmap >> (4 * q0)) & 7u) << 7);
                        char* w1 = p.ws[q1] + (((p.grpmap >> (4 * q1)) & 7u) << 7);
                        char* f = (lane < 32 ? w0 : w1) + WS_MBOX + (lane & 31) * 1024 + shard * 4;
                        asm volatile("global_store_dword %0, %1, off sc0 sc1" :: "v"(f), "v"(call) : "memory");
                    }
                }
            }
        }
    } else {
        const int j = wid - 4;
        if (j < 2) __builtin_amdgcn_s_sleep(4); else __builtin_amdgcn_s_sleep(BULK_DELAY);
#pragma unroll
        for (int c = 0; c < 8; ++c) BLDS(rW, (j * 8 + c) * 1024, 16, gvoff, gsoff + j * WPAIR + c * 128, 2);
        asm volatile("s_load_dword %0, %1, 0x0\n\ts_waitcnt lgkmcnt(0)" : "=&s"(callv) : "s"(ctr) : "memory");
        call = callv + 1u;
        const int rd = j * 8192 + (lane >> 5) * 1024 + (lane & 31) * 4;
        const int dv = ((lane >> 4) << 11) + ((lane & 15) << 4);
#pragma unroll
        for (int c = 0; c < 4; ++c) {
            if (c == 0) WAIT_V(6); else if (c == 1) WAIT_V(5); else if (c == 2) WAIT_V(4); else WAIT_V(3);
            const LAS float* c0 = (const LAS float*)(lds + rd + c * 2048);
            const LAS float* c1 = (const LAS float*)(lds + (rd ^ 64) + 512 + c * 2048);
            const h8 hv = cvt8((f32x4){c0[0], c0[32], c0[64], c0[96]}, (f32x4){c1[0], c1[32], c1[64], c1[96]});
            __builtin_amdgcn_raw_buffer_store_b128(__builtin_bit_cast(u32x4, hv), rWS, dv, wsoff + j * 2 * WKT_BYTES + c * 8192, 16);
        }
        if (j < 2) {
            WAIT_V(0);
            unsigned old = 0;
            if (lane == 0) old = __hip_atomic_fetch_add(arr + 1, 1u, __ATOMIC_RELAXED, __HIP_MEMORY_SCOPE_WORKGROUP);
            if (__builtin_amdgcn_readfirstlane(old) == 1u && lane < 16) {
                char* f = myws + WS_MBOX + (nt * 16 + lane) * 128 + (8 + kg) * 4;
                asm volatile("global_store_dword %0, %1, off sc1" :: "v"(f), "v"(call) : "memory");
            }
        }
    }
    WAIT_L(0);
    BAR;
    call = __builtin_amdgcn_readfirstlane(call);
    const int xg = (__builtin_amdgcn_readfirstlane(*(const volatile LAS unsigned*)(arr + 2)) - 1) & 7;
    const int src = (p.srcmap >> (xg * 4)) & (NRANKS - 1);
    const int mt = src * 2 + (xi & 1);
    const auto rX = RSRC(p.ws[src] + WS_X16 + (call & 1u) * X16_BYTES, X16_BYTES);
    const int wr = wid >> 2, wc = wid & 3;
    const int tv = tid * 16;
    const int xs0 = (mt & 1) * 2 * UNIT, ws0 = WS_W16 + nt * 2 * UNIT;
    const int ldsw = wid * 1024;
    const int aoff = wr * 8192 + lane * 16, boff = wc * 4096 + lane * 16;
    const int lj = ((xg >> 1) * 2 + (xi & 1)) * 2 + (xg & 1);
#define JGS(ct, j) ((j) * 8 * WROW + (ct) * 1024 + wr * WPAIR + wc * 256)
#define JWS(ct, j) (WS_W16 + ((j) >> 3) * WKT_BYTES + (ct) * 2 * UNIT + ((((j) >> 2) & 1) << 10) + (((j) & 3) << 8) + wr * 2 * WKT_BYTES + wc * 8192)
#define JFL(ct, j) (WS_WFLAG + ((ct) * NQ * 16 + (j)) * 4)
    int qgs = JGS(nt, lj), qws = JWS(nt, lj), qfl = JFL(nt, lj);
    int nfl = 1;
    const int qdv = ((lane >> 4) << 11) + ((lane & 15) << 4);
    const int qrd = SCR + wid * 2048 + (lane >> 5) * 1024 + (lane & 31) * 4;
    const int pl = (lane >= 24) ? (lane & 15) : lane;
    const int pthr = (pl < 16) ? 0 : 2 * (pl - 16) - 1;
    const int pinc = (pl < 16) ? 64 : 0;
    int poff = (pl < 16) ? WS_WFLAG + (nt * NQ + 1) * 64 + pl * 4 : WS_MBOX + (xi * 8 + xg) * 128 + (pl - 16) * 4;
#define SA(b, h) (((b) * 2 + (h)) * 16384)
#define SB(b, h) ((4 + (b) * 2 + (h)) * 16384)
#define STX(b, h, kt) do { BLDS(rX, SA(b, h) + ldsw, 16, tv, xs0 + (kt) * XKT_BYTES + (h) * UNIT, 0); \
                           BLDS(rX, SA(b, h) + ldsw + 8192, 16, tv, xs0 + (kt) * XKT_BYTES + (h) * UNIT + 8192, 0); } while (0)
#define STW(b, h, kt) do { BLDS(rWS, SB(b, h) + ldsw, 16, tv, ws0 + (kt) * WKT_BYTES + (h) * UNIT, 0); \
                           BLDS(rWS, SB(b, h) + ldsw + 8192, 16, tv, ws0 + (kt) * WKT_BYTES + (h) * UNIT + 8192, 0); } while (0)
#define LDA(dst, b, h) do { _Pragma("unroll") for (int m = 0; m < 4; ++m) _Pragma("unroll") for (int k = 0; k < 2; ++k) \
        dst[m][k] = *(const LAS h8*)(lds + SA(b, h) + aoff + m * 2048 + k * 1024); } while (0)
#define LDB(dst, b, h) do { _Pragma("unroll") for (int n = 0; n < 2; ++n) _Pragma("unroll") for (int k = 0; k < 2; ++k) \
        dst[n][k] = *(const LAS h8*)(lds + SB(b, h) + boff + n * 2048 + k * 1024); } while (0)
#define MMH(ai, h) asm volatile( \
            "v_mfma_f32_16x16x32_f16 %0, %8, %16, %0\n\t" \
            "v_mfma_f32_16x16x32_f16 %0, %9, %17, %0\n\t" \
            "v_mfma_f32_16x16x32_f16 %1, %11, %17, %1\n\t" \
            "v_mfma_f32_16x16x32_f16 %1, %10, %16, %1\n\t" \
            "v_mfma_f32_16x16x32_f16 %2, %12, %16, %2\n\t" \
            "v_mfma_f32_16x16x32_f16 %2, %13, %17, %2\n\t" \
            "v_mfma_f32_16x16x32_f16 %3, %15, %17, %3\n\t" \
            "v_mfma_f32_16x16x32_f16 %3, %14, %16, %3\n\t" \
            "v_mfma_f32_16x16x32_f16 %7, %14, %18, %7\n\t" \
            "v_mfma_f32_16x16x32_f16 %7, %15, %19, %7\n\t" \
            "v_mfma_f32_16x16x32_f16 %6, %13, %19, %6\n\t" \
            "v_mfma_f32_16x16x32_f16 %6, %12, %18, %6\n\t" \
            "v_mfma_f32_16x16x32_f16 %5, %10, %18, %5\n\t" \
            "v_mfma_f32_16x16x32_f16 %5, %11, %19, %5\n\t" \
            "v_mfma_f32_16x16x32_f16 %4, %9, %19, %4\n\t" \
            "v_mfma_f32_16x16x32_f16 %4, %8, %18, %4\n\t" \
            : "+v"(acc[ai][0][2*(h)][0]), "+v"(acc[ai][0][2*(h)][1]), "+v"(acc[ai][1][2*(h)][0]), "+v"(acc[ai][1][2*(h)][1]), \
              "+v"(acc[ai][0][2*(h)+1][0]), "+v"(acc[ai][0][2*(h)+1][1]), "+v"(acc[ai][1][2*(h)+1][0]), "+v"(acc[ai][1][2*(h)+1][1]) \
            : "v"(B0[0][0]), "v"(B0[0][1]), "v"(B0[1][0]), "v"(B0[1][1]), "v"(B1[0][0]), "v"(B1[0][1]), "v"(B1[1][0]), "v"(B1[1][1]), \
              "v"(At[2*(h)][0]), "v"(At[2*(h)][1]), "v"(At[2*(h)+1][0]), "v"(At[2*(h)+1][1]))
#define COMPUTE(ai) do { BAR; SCHED; __builtin_amdgcn_s_setprio(1); MMH(ai, 0); MMH(ai, 1); __builtin_amdgcn_s_setprio(0); BAR; SCHED; } while (0)
#define SEGA(b, tn) do { STX((b) ^ 1, 1, tn); LDB(B0, b, 0); LDA(At, b, 0); LDB(B1, b, 1); } while (0)
#define SEGB(b, tn) do { STX(b, 0, tn); LDA(At, b, 1); STW(b, 0, tn); STW(b, 1, tn); } while (0)
#define CHECKQ(q) do { asm volatile("" : "+v"(pv)); \
        if (__builtin_expect(!__all((int)(pv >= call) | (int)(pthr > (q))), 0)) \
            if (wid == 0) spin_flags((const unsigned*)(myws + poff), pthr > (q) ? 0u : call, call, tmo); } while (0)

    f32x4 acc[2][2][4][2];
#pragma unroll
    for (int a = 0; a < 2; ++a)
#pragma unroll
        for (int b = 0; b < 2; ++b)
#pragma unroll
            for (int m = 0; m < 4; ++m)
#pragma unroll
                for (int n = 0; n < 2; ++n) acc[a][b][m][n] = (f32x4){0.f, 0.f, 0.f, 0.f};
    h8 At[4][2], B0[2][2], B1[2][2];

    STX(0, 0, 0); STX(0, 1, 0); STW(0, 0, 0); STW(0, 1, 0);
    STX(1, 0, 1); STW(1, 0, 1); STW(1, 1, 1);
    WAIT_V(6); BAR;
    if (wid == 2 && lane == 0) asm volatile("global_store_dword %0, %1, off sc1" :: "v"(wflag_row + 16 + kg), "v"(call) : "memory");
    if (wr == 1) BAR;
#define GJOB(gs) do { BLDS(rW, SCR + 2 * ldsw, 16, gvoff, gs, 2); BLDS(rW, SCR + 2 * ldsw + 1024, 16, gvoff, (gs) + 128, 2); } while (0)
#define CREAD const LAS float* c0 = (const LAS float*)(lds + qrd); const LAS float* c1 = (const LAS float*)(lds + (qrd ^ 64) + 512); \
        const float f0 = c0[0], f1 = c0[32], f2 = c0[64], f3 = c0[96], f4 = c1[0], f5 = c1[32], f6 = c1[64], f7 = c1[96]
#define CSTORE(ws) do { SCHED; const h8 hv = cvt8((f32x4){f0, f1, f2, f3}, (f32x4){f4, f5, f6, f7}); \
        __builtin_amdgcn_raw_buffer_store_b128(__builtin_bit_cast(u32x4, hv), rWS, qdv, ws, 16); } while (0)
#define TRIP(J1, J2) do { \
        if (J1) GJOB(qgs + (q + 2) * (2 * WPAIR)); \
        SEGA(0, t + 1); WAIT_V(8 + 2 * J1 + J2); WAIT_L(0); COMPUTE(0); \
        SEGB(0, t + 2); \
        if (wid == 0) BLDS(rWS, POLL, 4, poff, 0, 17);            \
        WAIT_VX(8 + 2 * J1, wid, 0); WAIT_L(0); COMPUTE(1); \
        SEGA(1, t + 2); WAIT_VX(8, wid, 0); WAIT_L(0); COMPUTE(0); \
        if (J1) { CREAD; SEGB(1, t + 3); CSTORE(qws + (q + 2) * (4 * WKT_BYTES)); } else SEGB(1, t + 3); \
        WAIT_V(8 + J1); WAIT_L(0); COMPUTE(1); \
        if (J2) GJOB(qgs + 8 * WROW + (q + 3) * (2 * WPAIR)); \
        unsigned pv = *(const volatile LAS unsigned*)(lds + POLL + lane * 4); \
        SEGA(0, t + 3); WAIT_V(8 + J1 + 2 * J2); WAIT_L(0); \
        CHECKQ(q); \
        poff += pinc; \
        COMPUTE(0); \
        SEGB(0, t + 4); WAIT_V(8 + 2 * J2); WAIT_L(0); COMPUTE(1); \
        if (J1) { if (wid == 1) { if (lane < nfl) __builtin_amdgcn_raw_buffer_store_b32(call, rWS, lane * 4, qfl + (q + 2) * 64, 16); } } \
        SEGA(1, t + 4); if (J1) WAIT_VX(8, wid, 1); else WAIT_V(8); \
        WAIT_L(0); COMPUTE(0); \
        if (J2) { CREAD; SEGB(1, t + 5); CSTORE(qws + 256 + (q + 3) * (4 * WKT_BYTES)); } else SEGB(1, t + 5); \
        if (J1) WAIT_VX(8 + J2, wid, 1); else WAIT_V(8); \
        WAIT_L(0); COMPUTE(1); } while (0)
    {
        const bool two = !(xg & 1) && nt != 0, helper = src == p.rank && xi >= 16;
        const int n1 = two ? NQ - 3 : 0;
        const int n2 = (two || helper) ? NQ - 2 : 1;
        int q = 0;
#pragma nounroll
        for (; q < n1; ++q) { const int t = q * 4; TRIP(1, 1); nfl = 2; }
#pragma nounroll
        for (; q < n2; ++q) { const int t = q * 4; TRIP(1, 0); qgs = JGS(0, xi - 16); qws = JWS(0, xi - 16); qfl = JFL(0, xi - 16); }
#pragma nounroll
        for (; q < NQ - 2; ++q) { const int t = q * 4; TRIP(0, 0); }
    }
#pragma nounroll
    for (int it = NIT - 4; it < NIT - 1; ++it) {
        const int t = it * 2;
        const int sel = __builtin_amdgcn_readfirstlane((it == NIT - 4) ? wid : 1);
        unsigned pv = *(const volatile LAS unsigned*)(lds + POLL + lane * 4);
        SEGA(0, t + 1); WAIT_V(8); WAIT_L(0);
        if (it == NIT - 3) CHECKQ(NQ - 2);
        COMPUTE(0);
        SEGB(0, t + 2);
        if (sel == 0) BLDS(rWS, POLL, 4, poff, 0, 17);
        WAIT_VX(8, sel, 0); WAIT_L(0); COMPUTE(1);
        SEGA(1, t + 2); WAIT_VX(8, sel, 0); WAIT_L(0); COMPUTE(0);
        SEGB(1, t + 3); WAIT_V(8); WAIT_L(0); COMPUTE(1);
    }
    int ln;
    asm volatile("v_mbcnt_lo_u32_b32 %0, -1, 0\n\tv_mbcnt_hi_u32_b32 %0, -1, %0" : "=v"(ln));
    const int fr = ln & 15, fq = ln >> 4;
    float* obase = p.out + (size_t)(mt * 256 + wr * 64 + fr) * NLOC + nt * 256 + wc * 32 + fq * 4;
    const float qnan = __builtin_nanf("");
    unsigned tmov;
#define OUTQ(ai, bj) do { _Pragma("unroll") for (int m = 0; m < 4; ++m) _Pragma("unroll") for (int n = 0; n < 2; ++n) { \
        f32x4 v = acc[ai][bj][m][n]; \
        v[0] = fmaxf(v[0], 0.f); v[1] = fmaxf(v[1], 0.f); v[2] = fmaxf(v[2], 0.f); v[3] = fmaxf(v[3], 0.f); \
        if (bad) v = (f32x4){qnan, qnan, qnan, qnan}; \
        float* dst = obase + (size_t)((ai) * 128 + m * 16) * NLOC + (bj) * 128 + n * 16; \
        asm volatile("global_store_dwordx4 %0, %1, off sc1\n\ts_nop 1" :: "v"(dst), "v"(v) : "memory"); } } while (0)
    {
        SEGA(0, NKT - 1); WAIT_V(8); WAIT_L(0); COMPUTE(0);
        LDA(At, 0, 1); WAIT_V(2); WAIT_L(0); COMPUTE(1);
        LDB(B0, 1, 0); LDA(At, 1, 0); LDB(B1, 1, 1);
        asm volatile("global_load_dword %0, %1, off sc1\n\ts_waitcnt vmcnt(0)" : "=&v"(tmov) : "v"(tmo) : "memory");
        WAIT_L(0);
        COMPUTE(0);
        const bool bad = __builtin_amdgcn_readfirstlane(tmov) == call;
        asm volatile("s_nop 15" ::: "memory"); SCHED;
        LDA(At, 1, 1);
        OUTQ(0, 0); OUTQ(0, 1);
        WAIT_L(0);
        BAR; SCHED; __builtin_amdgcn_s_setprio(1); MMH(1, 0); MMH(1, 1); __builtin_amdgcn_s_setprio(0);
        asm volatile("s_nop 15\n\ts_nop 15" ::: "memory"); SCHED;
        OUTQ(1, 0); OUTQ(1, 1);
    }
    if (ln == 0) asm volatile("global_store_dword %0, %1, off sc1" :: "v"(ctr), "v"(call) : "memory");
    WAIT_V(0);
    if (wr == 0) BAR;
}

}

extern "C" void kernel_launch(int rank, int nranks, ncclComm_t comm,
                              hipStream_t stream,
                              void* const* d_in, const int* in_sizes, int n_in,
                              void* d_out, int out_size,
                              const int* in_dtypes, int out_dtype,
                              void* const* d_out_all,
                              void* const* d_ws_all, size_t ws_bytes) {
    if (nranks != NRANKS || (unsigned)rank >= (unsigned)NRANKS || ws_bytes < WS_NEEDED + (8u << 20) || n_in < 2 ||
        in_sizes[0] != MLOC * K_ || in_sizes[1] != K_ * NFULL || out_size != M_ * NLOC) {
        fprintf(stderr, "kernel_launch: unexpected problem shape\n");
        return;
    }
    HIP_CHECK(hipFuncSetAttribute((const void*)fused_kernel, hipFuncAttributeMaxDynamicSharedMemorySize, LDS_BYTES));
    Params p;
    p.x = (const float*)d_in[0];
    p.w = (const float*)d_in[1];
    p.out = (float*)d_out;
    const size_t wsoff = (ws_bytes - WS_NEEDED - (4u << 20)) & ~(size_t)((2u << 20) - 1);
    for (int q = 0; q < NRANKS; ++q) p.ws[q] = (char*)d_ws_all[q] + wsoff;
    p.rank = rank;
    {
        const unsigned half_mask = 0x4Du;
        p.srcmap = p.grpmap = 0;
        for (int r = 0; r < NRANKS; ++r) {
            const int mine = (half_mask >> r) & 1;
            int g = 0;
            for (int pass = 0; pass < 2; ++pass)
                for (int d = 0; d < NRANKS; ++d) {
                    const int q = (r + d) & 7;
                    if (((int)((half_mask >> q) & 1) == mine) != (pass == 0)) continue;
                    const int gg = (g == 0) ? SELFG : (g == SELFG ? 0 : g);
                    if (r == rank) p.srcmap |= (unsigned)q << (4 * gg);
                    if (q == rank) p.grpmap |= (unsigned)gg << (4 * r);
                    ++g;
                }
        }
    }
    hipLaunchKernelGGL(fused_kernel, dim3(256), dim3(512), LDS_BYTES, stream, p);
    HIP_CHECK(hipGetLastError());
}
```
